```python
import numpy as np

kernel_name = "gpu_gemm_rs_m4096_k4096_n16384_e4m3e5m2_dqrelu"


ABSMAX_RTOL = 0.05

COMPUTE_DTYPE = "float32"
EPILOGUE = "dequant_relu"
IN_QUANT = None
OUT_QUANT = None
IN_FP8 = "e4m3e5m2"
IN_DTYPES = {"x": "e4m3", "w_mat": "e5m2",
             "scale_x": "float32", "scale_w": "float32"}
WORLD_SIZE = 8

def _snap_fp8(x, mant_bits, max_abs, min_normal):
    a = np.abs(x.astype(np.float64))
    _m, _e = np.frexp(a)
    q = np.where(a >= min_normal, np.ldexp(1.0, _e - 1 - mant_bits),
                 min_normal * 2.0 ** -mant_bits)
    snapped = np.minimum(np.round(a / q) * q, max_abs)
    return (np.sign(x) * snapped).astype(np.float32)

def setup_inputs():

    x = _snap_fp8(np.random.randn(4096, 4096).astype(np.float32), 3, 448.0, 2.0 ** -6)
    w_mat = _snap_fp8(np.random.randn(4096, 16384).astype(np.float32), 2, 57344.0, 2.0 ** -14)
    scale_x = (0.02 * (1.0 + np.random.rand(1))).astype(np.float32)
    scale_w = (0.02 * (1.0 + np.random.rand(1))).astype(np.float32)
    return {"x": x, "w_mat": w_mat, "scale_x": scale_x, "scale_w": scale_w}

def reference(x, w_mat, scale_x, scale_w):
    acc = (x @ w_mat).astype(np.float32)
    y = (acc * (scale_x * scale_w)).astype(np.float32)
    return np.maximum(y, 0.0).astype(np.float32)
```

```cpp
#include <hip/hip_runtime.h>
#include <rccl/rccl.h>
#include <cstdio>
#include <cstdint>

#define HIP_CHECK(expr) do { hipError_t _e = (expr); if (_e != hipSuccess) { \
    fprintf(stderr, "HIP error %s at %s:%d\n", hipGetErrorString(_e), __FILE__, __LINE__); \
    return; } } while (0)

namespace {

constexpr int WORLD = 8;
constexpr int GM = 4096, GK = 4096, GN = 16384;
constexpr int KS = GK / WORLD;
constexpr int NS = GN / WORLD;
constexpr int MB = GM / WORLD;
constexpr int BM = 128, BN = 256, BK = 128;
constexpr int NKB = GK / BK;
constexpr int A_STAGE = BM * BK;
constexpr int B_STAGE = BN * BK;
constexpr int STAGE = A_STAGE + B_STAGE;
constexpr int NSTAGE = 3;
constexpr int NTHREADS = 512;
constexpr int NGRID = 256;
#define NPUSH_V 8
#define EARLY_REL 17
#define NTOK_V 3
constexpr int NPUSH = NPUSH_V;
constexpr unsigned NTOK = NTOK_V;
constexpr int NPIECE = 48;

constexpr size_t WS_CALL = 0;
constexpr size_t WS_EXIT = 128;
constexpr size_t WS_TMO = 256;
constexpr size_t WS_QTICK = 1024;
constexpr size_t WS_QDONE = 2048;
constexpr size_t WS_PDONE = 3072;
constexpr size_t WS_XFLAG = 4096;
constexpr size_t WS_WFLAG = 8192;
constexpr size_t WS_OFLAG = 16384;
constexpr size_t WS_DBG = 65536;
constexpr size_t WS_STAGE = 1u << 20;
constexpr size_t XS_BYTES = (size_t)GM * GK;
constexpr size_t WT_BYTES = (size_t)GK * NS;
constexpr size_t PAR_BYTES = XS_BYTES + WT_BYTES;
constexpr size_t WS_NEED = WS_STAGE + 2 * PAR_BYTES;
constexpr size_t OUT_BYTES = (size_t)MB * GN * 4;

constexpr unsigned long long DEADLINE_TICKS = 50000000ull;

typedef int v4i __attribute__((ext_vector_type(4)));
typedef unsigned v4u __attribute__((ext_vector_type(4)));
typedef int v8i __attribute__((ext_vector_type(8)));
typedef float v4f __attribute__((ext_vector_type(4)));

struct Params {
  const uint8_t* x;
  const uint8_t* w;
  const float* sx;
  const float* sw;
  float* out[WORLD];
  uint8_t* ws[WORLD];
  int rank;
};

#define AUX_SYS 17

__device__ __forceinline__ __amdgpu_buffer_rsrc_t make_rsrc(void* p, unsigned bytes) {
  return __builtin_amdgcn_make_buffer_rsrc(p, 0, bytes, 0x00020000);
}
__device__ __forceinline__ void glds16(const uint8_t* g, uint8_t* l) {
  __builtin_amdgcn_global_load_lds((const __attribute__((address_space(1))) void*)g,
                                   (__attribute__((address_space(3))) void*)l, 16, 0, 0);
}
__device__ __forceinline__ unsigned ld_sys(const unsigned* p) {
  return __hip_atomic_load(p, __ATOMIC_RELAXED, __HIP_MEMORY_SCOPE_SYSTEM);
}
__device__ __forceinline__ unsigned ld_agent(const unsigned* p) {
  return __hip_atomic_load(p, __ATOMIC_RELAXED, __HIP_MEMORY_SCOPE_AGENT);
}
__device__ __forceinline__ void st_sys(unsigned* p, unsigned v) {
  __hip_atomic_store(p, v, __ATOMIC_RELAXED, __HIP_MEMORY_SCOPE_SYSTEM);
}
__device__ __forceinline__ void st_agent(unsigned* p, unsigned v) {
  __hip_atomic_store(p, v, __ATOMIC_RELAXED, __HIP_MEMORY_SCOPE_AGENT);
}

__device__ __forceinline__ void wait_flags(const unsigned* flag, unsigned call, unsigned* tmo) {
  const unsigned long long t0 = __builtin_amdgcn_s_memrealtime();
  for (;;) {
    unsigned v = call;
    if (flag) v = ld_sys(flag);
    if (__all(v >= call)) break;
    if (ld_agent(tmo) == call || __builtin_amdgcn_s_memrealtime() - t0 > DEADLINE_TICKS) {
      st_agent(tmo, call);
      break;
    }
    __builtin_amdgcn_s_sleep(2);
  }
  __builtin_amdgcn_fence(__ATOMIC_ACQUIRE, "");
  asm volatile("s_waitcnt vmcnt(0)" ::: "memory");
}

#define STAMP(slot) do {} while (0)

__global__ void __launch_bounds__(NTHREADS) gemm_rs_kernel(Params p) {
  __shared__ __attribute__((aligned(16))) uint8_t smem[NSTAGE * STAGE];

  const int tid = threadIdx.x;
  const int lane = tid & 63;
  const int wave = __builtin_amdgcn_readfirstlane(tid >> 6);
  const unsigned g = blockIdx.x & 7u;
  const int idx_in = blockIdx.x >> 3;
  const int rank = p.rank;
  uint8_t* const myws = p.ws[rank];
  unsigned* const tmo = (unsigned*)(myws + WS_TMO);
  const unsigned call = ld_agent((const unsigned*)(myws + WS_CALL)) + 1u;
  const size_t stage_off = WS_STAGE + (size_t)(call & 1u) * PAR_BYTES;
  STAMP(0);

  if (idx_in < NPUSH) {
    uint8_t* const dws = p.ws[g];
    const __amdgpu_buffer_rsrc_t rs = make_rsrc(dws + stage_off, (unsigned)PAR_BYTES);
    const int c16 = lane & 15, gq = lane >> 4;
    for (int q = idx_in; q < NPIECE; q += NPUSH) {
      const bool is_w = (q < 8) || (q >= 16 && q < 24);
      unsigned* flag;
      if (is_w) {
        const int qq = q < 8 ? q : q - 8;
        const int nt = qq >> 1, h = qq & 1;
        const int wn = wave & 3, kbl = 2 * h + (wave >> 2);
        const uint8_t* src = p.w + (size_t)(kbl * 128 + gq * 16) * GN + (size_t)g * NS + nt * BN + wn * 64 + c16 * 4;
        const unsigned dst = (unsigned)XS_BYTES + (unsigned)nt * (NKB * B_STAGE) + (unsigned)(rank * 4 + kbl) * B_STAGE +
                             (unsigned)wn * 4 * 2048 + lane * 16;
#pragma unroll
        for (int qh = 0; qh < 2; ++qh) {
          unsigned v[16];
#pragma unroll
          for (int t = 0; t < 16; ++t) v[t] = *(const unsigned*)(src + (size_t)(qh * 64 + t) * GN);
          v4u o[4];
#pragma unroll
          for (int u = 0; u < 4; ++u) {
            const unsigned t0 = __builtin_amdgcn_perm(v[4 * u + 1], v[4 * u + 0], 0x05010400u);
            const unsigned t1 = __builtin_amdgcn_perm(v[4 * u + 1], v[4 * u + 0], 0x07030602u);
            const unsigned t2 = __builtin_amdgcn_perm(v[4 * u + 3], v[4 * u + 2], 0x05010400u);
            const unsigned t3 = __builtin_amdgcn_perm(v[4 * u + 3], v[4 * u + 2], 0x07030602u);
            o[0][u] = __builtin_amdgcn_perm(t2, t0, 0x05040100u);
            o[1][u] = __builtin_amdgcn_perm(t2, t0, 0x07060302u);
            o[2][u] = __builtin_amdgcn_perm(t3, t1, 0x05040100u);
            o[3][u] = __builtin_amdgcn_perm(t3, t1, 0x07060302u);
          }
#pragma unroll
          for (int j = 0; j < 4; ++j)
            __builtin_amdgcn_raw_buffer_store_b128(o[j], rs, dst + j * 2048 + qh * 1024, 0, AUX_SYS);
        }
        flag = (unsigned*)(dws + WS_WFLAG + (size_t)qq * 128) + rank;
      } else {
        const int qq = q < 16 ? q - 8 : q - 16;
        const int tm = (qq & 7) * 4 + (qq >> 3);
        const uint8_t* src = p.x + (size_t)(tm * BM + wave * 16 + c16) * KS + gq * 16;
        const unsigned dst = (unsigned)tm * (NKB * A_STAGE) + (unsigned)(rank * 4) * A_STAGE + (unsigned)wave * 2048 + lane * 16;
        v4u v[8];
#pragma unroll
        for (int u = 0; u < 8; ++u) v[u] = *(const v4u*)(src + u * 64);
#pragma unroll
        for (int u = 0; u < 8; ++u)
          __builtin_amdgcn_raw_buffer_store_b128(v[u], rs, dst + (u >> 1) * A_STAGE + (u & 1) * 1024, 0, AUX_SYS);
        flag = (unsigned*)(dws + WS_XFLAG + (size_t)tm * 128) + rank;
      }
      asm volatile("s_waitcnt vmcnt(0)" ::: "memory");
      __builtin_amdgcn_s_barrier();
      if (tid == 0) st_sys(flag, call);
    }
    if (tid == 0) __hip_atomic_fetch_add((unsigned*)(myws + WS_PDONE + g * 128), 1u, __ATOMIC_RELAXED, __HIP_MEMORY_SCOPE_AGENT);
  }
  STAMP(1);

  const int idx = (idx_in - NPUSH) & 31;
  const int mt = idx >> 3, nt = idx & 7;
  const int tm = (int)g * 4 + mt;
  if (wave == 0) {
    const unsigned* f = nullptr;
    if (lane < 8) f = (const unsigned*)(myws + WS_XFLAG + (size_t)tm * 128) + lane;
    else if (lane < 24) f = (const unsigned*)(myws + WS_WFLAG + (size_t)(nt * 2 + ((lane - 8) >> 3)) * 128) + (lane & 7);
    wait_flags(f, call, tmo);
  }
  __syncthreads();
  STAMP(2);

  const int wm = wave >> 2, wn = wave & 3;
  v4f acc[4][4];
#pragma unroll
  for (int i = 0; i < 4; ++i)
#pragma unroll
    for (int j = 0; j < 4; ++j) acc[i][j] = v4f{0.f, 0.f, 0.f, 0.f};

  {
    const uint8_t* ga = myws + stage_off + (size_t)tm * (NKB * A_STAGE) + wave * 2048 + lane * 16;
    const uint8_t* gb = myws + stage_off + XS_BYTES + (size_t)nt * (NKB * B_STAGE) + wave * 4096 + lane * 16;
    auto issue = [&](int kb, int boff) {
      uint8_t* la = smem + boff + wave * 2048;
      uint8_t* lb = smem + boff + A_STAGE + wave * 4096;
      const uint8_t* a = ga + (size_t)kb * A_STAGE;
      const uint8_t* b = gb + (size_t)kb * B_STAGE;
      glds16(a, la);
      glds16(a + 1024, la + 1024);
      glds16(b, lb);
      glds16(b + 1024, lb + 1024);
      glds16(b + 2048, lb + 2048);
      glds16(b + 3072, lb + 3072);
    };
    issue(0, 0);
    issue(1, STAGE);
    int boff = 0;
    const int a_rd = wm * 4 * 2048 + lane * 16;
    const int b_rd = A_STAGE + wn * 4 * 2048 + lane * 16;
    for (int kt = 0; kt < NKB; ++kt) {
      if (kt + 1 < NKB) asm volatile("s_waitcnt vmcnt(6)" ::: "memory");
      else asm volatile("s_waitcnt vmcnt(0)" ::: "memory");
      __builtin_amdgcn_s_barrier();
      asm volatile("" ::: "memory");
      if (kt + 2 < NKB) {
        int nb = boff + 2 * STAGE;
        if (nb >= NSTAGE * STAGE) nb -= NSTAGE * STAGE;
        issue(kt + 2, nb);
      }
      const uint8_t* sa = smem + boff + a_rd;
      const uint8_t* sb = smem + boff + b_rd;
      v8i af[4], bf[4];
#pragma unroll
      for (int i = 0; i < 4; ++i) {
        const v4i lo = *(const v4i*)(sa + i * 2048), hi = *(const v4i*)(sa + i * 2048 + 1024);
        af[i] = __builtin_shufflevector(lo, hi, 0, 1, 2, 3, 4, 5, 6, 7);
      }
#pragma unroll
      for (int j = 0; j < 4; ++j) {
        const v4i lo = *(const v4i*)(sb + j * 2048), hi = *(const v4i*)(sb + j * 2048 + 1024);
        bf[j] = __builtin_shufflevector(lo, hi, 0, 1, 2, 3, 4, 5, 6, 7);
      }
#pragma unroll
      for (int i = 0; i < 4; ++i)
#pragma unroll
        for (int j = 0; j < 4; ++j)
          acc[i][j] = __builtin_amdgcn_mfma_scale_f32_16x16x128_f8f6f4(af[i], bf[j], acc[i][j], 0  , 1  , 0, 0, 0, 0);
      asm volatile("s_waitcnt lgkmcnt(0)" ::: "memory");
      boff += STAGE;
      if (boff >= NSTAGE * STAGE) boff = 0;
    }
  }
  STAMP(3);

  if (g != (unsigned)rank) {
    if (tid == 0) {
      const unsigned t = __hip_atomic_fetch_add((unsigned*)(myws + WS_QTICK + g * 128), 1u, __ATOMIC_RELAXED, __HIP_MEMORY_SCOPE_AGENT);
      const unsigned* pd = (const unsigned*)(myws + WS_PDONE + g * 128);
      const unsigned* qd = (const unsigned*)(myws + WS_QDONE + g * 128);
      const unsigned long long t0 = __builtin_amdgcn_s_memrealtime();
      while (ld_agent(pd) < call * NPUSH || (int)(ld_agent(qd) + NTOK - t) <= 0) {
        if (ld_agent(tmo) == call || __builtin_amdgcn_s_memrealtime() - t0 > DEADLINE_TICKS) {
          st_agent(tmo, call);
          break;
        }
        __builtin_amdgcn_s_sleep(8);
      }
    }
    __syncthreads();
  }
  STAMP(6);

  {
    const bool bad = ld_agent(tmo) == call;
    const float s = p.sx[0] * p.sw[0];
    const __amdgpu_buffer_rsrc_t ro = make_rsrc(p.out[g], (unsigned)OUT_BYTES);
    const int c16 = lane & 15, gq = lane >> 4;
    const unsigned off0 = ((unsigned)(mt * BM + wm * 64 + gq * 4) * GN + (unsigned)(rank * NS + nt * BN + wn * 64 + c16 * 4)) * 4u;
#pragma unroll
    for (int i = 0; i < 4; ++i)
#pragma unroll
      for (int r = 0; r < 4; ++r) {
        v4f v;
#pragma unroll
        for (int j = 0; j < 4; ++j) {
          const float y = acc[i][j][r] * s;
          v[j] = (y < 0.f) ? 0.f : y;
          if (bad) v[j] = __builtin_nanf("");
        }
        __builtin_amdgcn_raw_buffer_store_b128(__builtin_bit_cast(v4u, v), ro, off0 + (unsigned)(i * 16 + r) * (GN * 4), 0, AUX_SYS);
      }
    asm volatile("s_waitcnt vmcnt(%0)" :: "n"(EARLY_REL - 1) : "memory");
    __builtin_amdgcn_s_barrier();
    if (tid == 0 && g != (unsigned)rank)
      __hip_atomic_fetch_add((unsigned*)(myws + WS_QDONE + g * 128), 1u, __ATOMIC_RELAXED, __HIP_MEMORY_SCOPE_AGENT);
    asm volatile("s_waitcnt vmcnt(0)" ::: "memory");
    __builtin_amdgcn_s_barrier();
    STAMP(4);
    if (tid == 0 && g != (unsigned)rank) {
      st_sys((unsigned*)(p.ws[g] + WS_OFLAG + ((size_t)rank * 32 + idx_in) * 128), call);
    }
  }

  if (wave == 0) {
    if (g != (unsigned)rank) {
      const unsigned* f = lane == 0 ? (const unsigned*)(myws + WS_OFLAG + ((size_t)g * 32 + idx_in) * 128) : nullptr;
      wait_flags(f, call, tmo);
      if (lane == 0 && ld_agent(tmo) == call)
        p.out[rank][(size_t)(mt * BM) * GN + (size_t)g * NS + nt * BN] = __builtin_nanf("");
    }
    STAMP(5);
    if (lane == 0) {
      const unsigned t = __hip_atomic_fetch_add((unsigned*)(myws + WS_EXIT), 1u, __ATOMIC_RELAXED, __HIP_MEMORY_SCOPE_AGENT);
      if ((t + 1u) % NGRID == 0u) st_agent((unsigned*)(myws + WS_CALL), call);
    }
  }
}

}

extern "C" void kernel_launch(int rank, int nranks, ncclComm_t comm,
                              hipStream_t stream,
                              void* const* d_in, const int* in_sizes, int n_in,
                              void* d_out, int out_size,
                              const int* in_dtypes, int out_dtype,
                              void* const* d_out_all,
                              void* const* d_ws_all, size_t ws_bytes) {
  (void)comm; (void)in_dtypes; (void)out_dtype; (void)d_out;
  if (nranks != WORLD || n_in != 4 || ws_bytes < WS_NEED || (unsigned)rank >= (unsigned)WORLD ||
      in_sizes[0] != GM * KS || in_sizes[1] != KS * GN || out_size != MB * GN) {
    fprintf(stderr, "kernel_launch: unexpected problem shape\n");
    return;
  }
  Params p;
  p.x = (const uint8_t*)d_in[0];
  p.w = (const uint8_t*)d_in[1];
  p.sx = (const float*)d_in[2];
  p.sw = (const float*)d_in[3];
  for (int r = 0; r < WORLD; ++r) {
    p.out[r] = (float*)d_out_all[r];
    p.ws[r] = (uint8_t*)d_ws_all[r];
  }
  p.rank = rank;
  gemm_rs_kernel<<<dim3(NGRID), dim3(NTHREADS), 0, stream>>>(p);
  HIP_CHECK(hipGetLastError());
}
```

```cpp
#include <hip/hip_runtime.h>
#include <rccl/rccl.h>
#include <cstdio>
#include <cstdint>
#include <type_traits>

#define HIP_CHECK(expr) do { hipError_t _e = (expr); if (_e != hipSuccess) { \
    fprintf(stderr, "HIP error %s at %s:%d\n", hipGetErrorString(_e), __FILE__, __LINE__); \
    return; } } while (0)

namespace {

constexpr int WORLD = 8;
constexpr int GM = 4096, GK = 4096, GN = 16384;
constexpr int KS = GK / WORLD;
constexpr int NS = GN / WORLD;
constexpr int MB = GM / WORLD;
constexpr int BM = 128, BN = 256, BK = 128;
constexpr int NKB = GK / BK;
constexpr int A_STAGE = BM * BK;
constexpr int B_STAGE = BN * BK;
constexpr int STAGE = A_STAGE + B_STAGE;
constexpr int NSTAGE = 3;
constexpr int NTHREADS = 512;
constexpr int NGRID = 256;
#define REL_AT 4
#define DEC_AUX 17
#define NTOK_V 2
constexpr int NPUSH = 8;
constexpr unsigned NTOK = NTOK_V;

constexpr size_t WS_CALL = 0;
constexpr size_t WS_EXIT = 128;
constexpr size_t WS_TMO = 256;
constexpr size_t WS_QTICK = 1024;
constexpr size_t WS_QDONE = 2048;
constexpr size_t WS_PDONE = 3072;
constexpr size_t WS_PQ = 12288;
constexpr size_t WS_XFLAG = 4096;
constexpr size_t WS_WFLAG = 8192;
constexpr size_t WS_OFLAG = 16384;
constexpr size_t WS_DBG = 65536;
constexpr size_t WS_STAGE = 1u << 20;
constexpr size_t XS_BYTES = (size_t)GM * GK;
constexpr size_t WT_BYTES = (size_t)GK * NS;
constexpr size_t PAR_BYTES = XS_BYTES + WT_BYTES;
constexpr unsigned PK_SLOT = 17408;
constexpr size_t WS_PK = WS_STAGE + 2 * PAR_BYTES;
constexpr size_t PK_BYTES = (size_t)WORLD * 32 * 8 * PK_SLOT;
constexpr size_t WS_NEED = WS_PK + PK_BYTES;
constexpr size_t OUT_BYTES = (size_t)MB * GN * 4;

constexpr unsigned long long DEADLINE_TICKS = 50000000ull;

typedef int v4i __attribute__((ext_vector_type(4)));
typedef unsigned v4u __attribute__((ext_vector_type(4)));
typedef int v8i __attribute__((ext_vector_type(8)));
typedef float v4f __attribute__((ext_vector_type(4)));

struct Params {
  const uint8_t* x;
  const uint8_t* w;
  const float* sx;
  const float* sw;
  float* out[WORLD];
  uint8_t* ws[WORLD];
  int rank;
};

template <int I, int N, class F>
__device__ __forceinline__ void static_for(F&& f) {
  if constexpr (I < N) {
    f(std::integral_constant<int, I>{});
    static_for<I + 1, N>(f);
  }
}

#define AUX_SYS 17

__device__ __forceinline__ __amdgpu_buffer_rsrc_t make_rsrc(void* p, unsigned bytes) {
  return __builtin_amdgcn_make_buffer_rsrc(p, 0, bytes, 0x00020000);
}
#define bufld16(rs, l, voff, soff, aux) \
  __builtin_amdgcn_raw_ptr_buffer_load_lds(rs, (__attribute__((address_space(3))) void*)(l), 16, voff, soff, 0, aux)
__device__ __forceinline__ unsigned ld_sys(const unsigned* p) {
  return __hip_atomic_load(p, __ATOMIC_RELAXED, __HIP_MEMORY_SCOPE_SYSTEM);
}
__device__ __forceinline__ unsigned ld_agent(const unsigned* p) {
  return __hip_atomic_load(p, __ATOMIC_RELAXED, __HIP_MEMORY_SCOPE_AGENT);
}
__device__ __forceinline__ void st_sys(unsigned* p, unsigned v) {
  __hip_atomic_store(p, v, __ATOMIC_RELAXED, __HIP_MEMORY_SCOPE_SYSTEM);
}
__device__ __forceinline__ void st_agent(unsigned* p, unsigned v) {
  __hip_atomic_store(p, v, __ATOMIC_RELAXED, __HIP_MEMORY_SCOPE_AGENT);
}

template <bool ACQ>
__device__ __forceinline__ void wait_flags(const unsigned* flag, unsigned call, unsigned* tmo) {
  const unsigned long long t0 = __builtin_amdgcn_s_memrealtime();
  for (;;) {
    unsigned v = call;
    if (flag) v = ld_sys(flag);
    if (__all(v >= call)) break;
    if (ld_agent(tmo) == call || __builtin_amdgcn_s_memrealtime() - t0 > DEADLINE_TICKS) {
      st_agent(tmo, call);
      break;
    }
    __builtin_amdgcn_s_sleep(2);
  }
  if (ACQ) __builtin_amdgcn_fence(__ATOMIC_ACQUIRE, "");
  asm volatile("s_waitcnt vmcnt(0)" ::: "memory");
}

#define STAMP(slot) do {} while (0)

__global__ void __launch_bounds__(NTHREADS) gemm_rs_kernel(Params p) {
  __shared__ __attribute__((aligned(16))) uint8_t smem[NSTAGE * STAGE + 64];

  const int tid = threadIdx.x;
  const int lane = tid & 63;
  const int wave = __builtin_amdgcn_readfirstlane(tid >> 6);
  const unsigned g = blockIdx.x & 7u;
  const int idx_in = blockIdx.x >> 3;
  const int rank = p.rank;
  uint8_t* const myws = p.ws[rank];
  unsigned* const tmo = (unsigned*)(myws + WS_TMO);
  const unsigned call_v = ld_agent((const unsigned*)(myws + WS_CALL));
  STAMP(0);

  const int c16 = lane & 15, gq = lane >> 4;

  if (idx_in < NPUSH) {
    uint8_t* const dws = p.ws[g];
    auto* const lds_next = (__attribute__((address_space(3))) unsigned*)smem;
    const int wn = wave & 3;
    const uint8_t* const wsrc = p.w + (size_t)((wave >> 2) * 128 + (idx_in & 1) * 256 + gq * 16) * GN + (size_t)g * NS + (idx_in >> 1) * BN + wn * 64 + c16 * 4;
    const unsigned wdst = (unsigned)XS_BYTES + (unsigned)(idx_in >> 1) * (NKB * B_STAGE) + (unsigned)(rank * 4 + (idx_in & 1) * 2 + (wave >> 2)) * B_STAGE +
                          (unsigned)wn * 4 * 2048 + lane * 16;
    const unsigned xsrc = (unsigned)(wave * 16 + c16) * KS + gq * 16;
    const unsigned xdst = (unsigned)(rank * 4) * A_STAGE + (unsigned)wave * 2048 + lane * 16;
    unsigned* const pq = (unsigned*)(myws + WS_PQ + g * 128);

    auto load_x = [&](v4u (&b)[8], int tmq) {
      const __amdgpu_buffer_rsrc_t rx = make_rsrc((void*)p.x, (unsigned)tmq < 32u ? (unsigned)(GM * KS) : 0u);
#pragma unroll
      for (int u = 0; u < 8; ++u) b[u] = __builtin_amdgcn_raw_buffer_load_b128(rx, xsrc, (tmq & 31) * (BM * KS) + u * 64, 0);
    };
    auto load_w = [&](unsigned (&b)[32], int half) {
#pragma unroll
      for (int t = 0; t < 32; ++t) b[t] = *(const unsigned*)(wsrc + half * (4 * BN) + (size_t)((t >> 4) * 64 + (t & 15)) * GN);
    };
    v4u xa[8], xb[8];
    unsigned wa[32], wb[32];
    load_x(xa, idx_in * 4);
    asm volatile("" ::: "memory");
    load_w(wa, 0);
    asm volatile("" ::: "memory");
    const unsigned call = __builtin_amdgcn_readfirstlane(call_v) + 1u;
    const __amdgpu_buffer_rsrc_t rs = make_rsrc(dws + WS_STAGE + (size_t)(call & 1u) * PAR_BYTES, (unsigned)PAR_BYTES);
    const unsigned pq_base = (call - 1u) * (24u + NPUSH);

    auto store_x = [&](const v4u (&b)[8], int tmq) {
      const unsigned dst = xdst + (unsigned)tmq * (NKB * A_STAGE);
#pragma unroll
      for (int u = 0; u < 8; ++u)
        __builtin_amdgcn_raw_buffer_store_b128(b[u], rs, dst + (u >> 1) * A_STAGE + (u & 1) * 1024, 0, AUX_SYS);
    };
    auto store_w = [&](const unsigned (&b)[32], int half) {
      const unsigned dst = wdst + (unsigned)half * (4u * NKB * B_STAGE);
#pragma unroll
      for (int qh = 0; qh < 2; ++qh) {
        v4u o[4];
#pragma unroll
        for (int u = 0; u < 4; ++u) {
          const unsigned* v = b + qh * 16 + 4 * u;
          const unsigned t0 = __builtin_amdgcn_perm(v[1], v[0], 0x05010400u);
          const unsigned t1 = __builtin_amdgcn_perm(v[1], v[0], 0x07030602u);
          const unsigned t2 = __builtin_amdgcn_perm(v[3], v[2], 0x05010400u);
          const unsigned t3 = __builtin_amdgcn_perm(v[3], v[2], 0x07030602u);
          o[0][u] = __builtin_amdgcn_perm(t2, t0, 0x05040100u);
          o[1][u] = __builtin_amdgcn_perm(t2, t0, 0x07060302u);
          o[2][u] = __builtin_amdgcn_perm(t3, t1, 0x05040100u);
          o[3][u] = __builtin_amdgcn_perm(t3, t1, 0x07060302u);
        }
#pragma unroll
        for (int j = 0; j < 4; ++j)
          __builtin_amdgcn_raw_buffer_store_b128(o[j], rs, dst + j * 2048 + qh * 1024, 0, AUX_SYS);
      }
    };
    auto xflag = [&](int tmq) { return (unsigned*)(dws + WS_XFLAG + (size_t)tmq * 128) + rank; };
    auto wflag = [&](int half) { return (unsigned*)(dws + WS_WFLAG + (size_t)(half * 8 + idx_in) * 128) + rank; };
#define PUSH_FENCE() asm volatile("" ::: "memory")
#define PUSH_PUBLISH(N, flagptr) do { \
      asm volatile("s_waitcnt vmcnt(" #N ")" ::: "memory"); \
      __builtin_amdgcn_s_barrier(); PUSH_FENCE(); \
      if (tid == 0) st_sys(flagptr, call); \
      PUSH_STAMP(flagptr); PUSH_FENCE(); } while (0)
#define PUSH_DRAW() do { if (tid == 0) { \
      const unsigned d = __hip_atomic_fetch_add(pq, 1u, __ATOMIC_RELAXED, __HIP_MEMORY_SCOPE_AGENT) - pq_base; \
      *lds_next = d < 24u ? (d & 7u) * 4u + 1u + (d >> 3) : 32u; } } while (0)
    auto ticket_done = [&]() -> int {
      asm volatile("s_waitcnt lgkmcnt(0)" ::: "memory");
      __builtin_amdgcn_s_barrier();
      PUSH_FENCE();
      return __builtin_amdgcn_readfirstlane(*lds_next);
    };
#define PUSH_STAMP(f) do {} while (0)
    store_x(xa, idx_in * 4); PUSH_FENCE();
    load_w(wb, 1); PUSH_FENCE();
    PUSH_PUBLISH(32, xflag(idx_in * 4));
    PUSH_DRAW();
    store_w(wa, 0); PUSH_FENCE();
    int ta = ticket_done();
    load_x(xa, ta); PUSH_FENCE();
    PUSH_PUBLISH(8, wflag(0));
    if (ta < 32) PUSH_DRAW();
    store_w(wb, 1); PUSH_FENCE();
    int tb = 32;
    if (ta < 32) tb = ticket_done();
    else if (tid == 0) __hip_atomic_fetch_add((unsigned*)(myws + WS_PDONE + g * 128), 1u, __ATOMIC_RELAXED, __HIP_MEMORY_SCOPE_AGENT);
    load_x(xb, tb); PUSH_FENCE();
    PUSH_PUBLISH(8, wflag(1));
    for (;;) {
      if (ta >= 32) break;
      {
        const int cur = ta;
        if (tb < 32) PUSH_DRAW();
        store_x(xa, cur); PUSH_FENCE();
        if (tb < 32) ta = ticket_done();
        else {
          ta = 32;
          if (tid == 0) __hip_atomic_fetch_add((unsigned*)(myws + WS_PDONE + g * 128), 1u, __ATOMIC_RELAXED, __HIP_MEMORY_SCOPE_AGENT);
        }
        load_x(xa, ta); PUSH_FENCE();
        PUSH_PUBLISH(8, xflag(cur));
      }
      if (tb >= 32) break;
      {
        const int cur = tb;
        if (ta < 32) PUSH_DRAW();
        store_x(xb, cur); PUSH_FENCE();
        if (ta < 32) tb = ticket_done();
        else {
          tb = 32;
          if (tid == 0) __hip_atomic_fetch_add((unsigned*)(myws + WS_PDONE + g * 128), 1u, __ATOMIC_RELAXED, __HIP_MEMORY_SCOPE_AGENT);
        }
        load_x(xb, tb); PUSH_FENCE();
        PUSH_PUBLISH(8, xflag(cur));
      }
    }
  }
  STAMP(1);
  const unsigned call = __builtin_amdgcn_readfirstlane(call_v) + 1u;
  const size_t stage_off = WS_STAGE + (size_t)(call & 1u) * PAR_BYTES;

  const int idx = (idx_in - NPUSH) & 31;
  const int mt = idx >> 3, nt = idx & 7;
  const int tm = (int)g * 4 + mt;
  auto* const lds_sel = (__attribute__((address_space(3))) int*)(smem + NSTAGE * STAGE);
  const unsigned* myflag = nullptr;
  if (lane < 8) myflag = (const unsigned*)(myws + WS_XFLAG + (size_t)tm * 128) + lane;
  else if (lane < 24) myflag = (const unsigned*)(myws + WS_WFLAG + (size_t)(nt * 2 + ((lane - 8) >> 3)) * 128) + (lane & 7);
  unsigned rem = 0xffu, ready = 0u;
  auto select = [&]() {
    const unsigned long long t0 = __builtin_amdgcn_s_memrealtime();
    while ((ready & rem) == 0u) {
      unsigned v = call;
      if (myflag) v = ld_sys(myflag);
      const unsigned m = (unsigned)__builtin_amdgcn_ballot_w64(v >= call);
      ready = m & (m >> 8) & (m >> 16) & 0xffu;
      if (ready & rem) break;
      if (ld_agent(tmo) == call || __builtin_amdgcn_s_memrealtime() - t0 > DEADLINE_TICKS) {
        if (lane == 0) st_agent(tmo, call);
        ready = 0xffu;
        break;
      }
      __builtin_amdgcn_s_sleep(4);
    }
    const unsigned cand = ready & rem;
    const int sel = (cand >> rank) & 1u ? rank : __builtin_ctz(cand);
    rem &= ~(1u << sel);
    if (lane == 0) *lds_sel = sel;
  };
  if (wave == 0) {
    select();
    __builtin_amdgcn_fence(__ATOMIC_ACQUIRE, "");
    asm volatile("s_waitcnt vmcnt(0) lgkmcnt(0)" ::: "memory");
  }
  __builtin_amdgcn_s_barrier();
  asm volatile("" ::: "memory");
  int src = __builtin_amdgcn_readfirstlane(*lds_sel);

  const int wm = wave >> 2, wn = wave & 3;
  v4f acc[4][4];
#pragma unroll
  for (int i = 0; i < 4; ++i)
#pragma unroll
    for (int j = 0; j < 4; ++j) acc[i][j] = v4f{0.f, 0.f, 0.f, 0.f};
  const float s = p.sx[0] * p.sw[0];
  unsigned ticket = 0;

  {
    uint8_t* const sbase = myws + stage_off;
    const unsigned va = (unsigned)tm * (NKB * A_STAGE) + wave * 2048 + lane * 16;
    const unsigned vb = (unsigned)XS_BYTES + (unsigned)nt * (NKB * B_STAGE) + wave * 4096 + lane * 16;
    auto issue1 = [&](unsigned nrec, int kb, int boff, int c) {
      const __amdgpu_buffer_rsrc_t rs = make_rsrc(sbase, nrec);
      if (c < 2) bufld16(rs, smem + boff + wave * 2048 + c * 1024, va, kb * A_STAGE + c * 1024, 0);
      else bufld16(rs, smem + boff + A_STAGE + wave * 4096 + (c - 2) * 1024, vb, kb * B_STAGE + (c - 2) * 1024, 0);
    };
#pragma unroll
    for (int c = 0; c < 6; ++c) issue1((unsigned)PAR_BYTES, src * 4, 0, c);
#pragma unroll
    for (int c = 0; c < 6; ++c) issue1((unsigned)PAR_BYTES, src * 4 + 1, STAGE, c);
    int boff = 0;
    const int a_rd = wm * 4 * 2048 + lane * 16;
    const int b_rd = A_STAGE + wn * 4 * 2048 + lane * 16;
    auto step = [&](unsigned nrec, int pkb) {
      int nb = boff + 2 * STAGE;
      if (nb >= NSTAGE * STAGE) nb -= NSTAGE * STAGE;
      const uint8_t* sa = smem + boff + a_rd;
      const uint8_t* sb = smem + boff + b_rd;
      v8i af[4], bf[4];
      auto rd = [&](const uint8_t* q) {
        const v4i lo = *(const v4i*)q, hi = *(const v4i*)(q + 1024);
        return __builtin_shufflevector(lo, hi, 0, 1, 2, 3, 4, 5, 6, 7);
      };
      af[0] = rd(sa);
#pragma unroll
      for (int j = 0; j < 4; ++j) bf[j] = rd(sb + j * 2048);
#pragma unroll
      for (int i = 1; i < 4; ++i) af[i] = rd(sa + i * 2048);
      __builtin_amdgcn_sched_barrier(0);
#pragma unroll
      for (int i = 0; i < 4; ++i) {
#pragma unroll
        for (int j = 0; j < 4; ++j) {
          acc[i][j] = __builtin_amdgcn_mfma_scale_f32_16x16x128_f8f6f4(af[i], bf[j], acc[i][j], 0  , 1  , 0, 0, 0, 0);
          if ((j & 1) && i < 3) {
            __builtin_amdgcn_sched_barrier(0);
            issue1(nrec, pkb, nb, i * 2 + (j >> 1));
            __builtin_amdgcn_sched_barrier(0);
          }
        }
      }
      __builtin_amdgcn_s_waitcnt(0xC07F);
      boff += STAGE;
      if (boff >= NSTAGE * STAGE) boff = 0;
    };
#define GEMM_SYNC() do { asm volatile("s_waitcnt vmcnt(6)" ::: "memory"); __builtin_amdgcn_s_barrier(); asm volatile("" ::: "memory"); } while (0)
    __builtin_amdgcn_s_waitcnt(0xC07F);
    for (int slot = 0; slot < WORLD; ++slot) {
      const bool more = slot + 1 < WORLD;
      const unsigned nrec = more ? (unsigned)PAR_BYTES : 0u;
      GEMM_SYNC();
      step((unsigned)PAR_BYTES, src * 4 + 2);
      GEMM_SYNC();
      step((unsigned)PAR_BYTES, src * 4 + 3);
      if (more && wave == 0) {
        select();
        if (slot == WORLD - 2) {
          STAMP(2);
          if (lane == 0 && g != (unsigned)rank)
            ticket = __hip_atomic_fetch_add((unsigned*)(myws + WS_QTICK + g * 128), 1u, __ATOMIC_RELAXED, __HIP_MEMORY_SCOPE_AGENT);
        }
        asm volatile("s_waitcnt lgkmcnt(0)" ::: "memory");
      }
      GEMM_SYNC();
      const int nsrc = more ? __builtin_amdgcn_readfirstlane(*lds_sel) : 0;
      __builtin_amdgcn_s_waitcnt(0xC07F);
      step(nrec, nsrc * 4);
      GEMM_SYNC();
      step(nrec, nsrc * 4 + 1);
      src = nsrc;
    }
  }
  const unsigned tmo_v = ld_agent(tmo);
  asm volatile("s_waitcnt vmcnt(1)" ::: "memory");
  STAMP(3);

  const __amdgpu_buffer_rsrc_t ro = make_rsrc(p.out[rank], (unsigned)OUT_BYTES);
  const unsigned row_off = (unsigned)(mt * BM + wm * 64 + gq * 4) * GN * 4u;
  const unsigned col_off = (unsigned)(nt * BN + wn * 64 + c16 * 4) * 4u;

  if (g == (unsigned)rank) {
    const bool bad = tmo_v == call;
#pragma unroll
    for (int i = 0; i < 4; ++i)
#pragma unroll
      for (int r = 0; r < 4; ++r) {
        v4f v;
#pragma unroll
        for (int j = 0; j < 4; ++j) {
          const float y = acc[i][j][r] * s;
          v[j] = (y <= 0.f) ? 0.f : y;
          if (bad) v[j] = __builtin_nanf("");
        }
        __builtin_amdgcn_raw_buffer_store_b128(__builtin_bit_cast(v4u, v), ro,
                                               row_off + col_off + (unsigned)(rank * NS * 4) + (unsigned)(i * 16 + r) * (GN * 4), 0, AUX_SYS);
      }
    STAMP(4);
  } else {
    asm volatile("s_waitcnt lgkmcnt(0)" ::: "memory");
    __builtin_amdgcn_s_barrier();
    asm volatile("" ::: "memory");
    uint8_t* const pk = smem + wave * PK_SLOT;
    const unsigned pk_lds = (unsigned)(unsigned long)((__attribute__((address_space(3))) uint8_t*)smem) + wave * PK_SLOT;
    unsigned nxt = pk_lds + 512u;
    unsigned bits[2] = {0u, 0u};
#pragma unroll
    for (int i = 0; i < 4; ++i)
#pragma unroll
      for (int r = 0; r < 4; ++r) {
        float y0, y1, y2, y3;
        unsigned t0, t1, t2, t3, n, b1, b2, b3;
        asm volatile(
            "v_mul_f32 %[y0], %[s], %[a0]\n\t"
            "v_mul_f32 %[y1], %[s], %[a1]\n\t"
            "v_mul_f32 %[y2], %[s], %[a2]\n\t"
            "v_mul_f32 %[y3], %[s], %[a3]\n\t"
            "v_cmp_nge_f32_e64 s[84:85], 0, %[y0]\n\t"
            "v_cmp_nge_f32_e64 s[86:87], 0, %[y1]\n\t"
            "v_cmp_nge_f32_e64 s[88:89], 0, %[y2]\n\t"
            "v_cmp_nge_f32_e64 s[90:91], 0, %[y3]\n\t"
            "s_bcnt1_i32_b64 %[n], s[84:85]\n\t"
            "s_lshl2_add_u32 %[b1], %[n], %[nx]\n\t"
            "s_bcnt1_i32_b64 %[n], s[86:87]\n\t"
            "s_lshl2_add_u32 %[b2], %[n], %[b1]\n\t"
            "s_bcnt1_i32_b64 %[n], s[88:89]\n\t"
            "s_lshl2_add_u32 %[b3], %[n], %[b2]\n\t"
            "v_mbcnt_lo_u32_b32 %[t0], s84, 0\n\t"
            "v_mbcnt_lo_u32_b32 %[t1], s86, 0\n\t"
            "v_mbcnt_lo_u32_b32 %[t2], s88, 0\n\t"
            "v_mbcnt_lo_u32_b32 %[t3], s90, 0\n\t"
            "v_mbcnt_hi_u32_b32 %[t0], s85, %[t0]\n\t"
            "v_mbcnt_hi_u32_b32 %[t1], s87, %[t1]\n\t"
            "v_mbcnt_hi_u32_b32 %[t2], s89, %[t2]\n\t"
            "v_mbcnt_hi_u32_b32 %[t3], s91, %[t3]\n\t"
            "v_lshl_add_u32 %[t0], %[t0], 2, %[nx]\n\t"
            "v_lshl_add_u32 %[t1], %[t1], 2, %[b1]\n\t"
            "v_lshl_add_u32 %[t2], %[t2], 2, %[b2]\n\t"
            "v_lshl_add_u32 %[t3], %[t3], 2, %[b3]\n\t"
            "s_bcnt1_i32_b64 %[n], s[90:91]\n\t"
            "s_lshl2_add_u32 %[nx], %[n], %[b3]\n\t"
            "v_addc_co_u32_e64 %[bt], vcc, %[bt], %[bt], s[84:85]\n\t"
            "v_addc_co_u32_e64 %[bt], vcc, %[bt], %[bt], s[86:87]\n\t"
            "v_addc_co_u32_e64 %[bt], vcc, %[bt], %[bt], s[88:89]\n\t"
            "v_addc_co_u32_e64 %[bt], vcc, %[bt], %[bt], s[90:91]\n\t"
            "s_mov_b64 exec, s[84:85]\n\t"
            "ds_write_b32 %[t0], %[y0]\n\t"
            "s_mov_b64 exec, s[86:87]\n\t"
            "ds_write_b32 %[t1], %[y1]\n\t"
            "s_mov_b64 exec, s[88:89]\n\t"
            "ds_write_b32 %[t2], %[y2]\n\t"
            "s_mov_b64 exec, s[90:91]\n\t"
            "ds_write_b32 %[t3], %[y3]\n\t"
            "s_mov_b64 exec, -1"
            : [y0] "=&v"(y0), [y1] "=&v"(y1), [y2] "=&v"(y2), [y3] "=&v"(y3), [t0] "=&v"(t0), [t1] "=&v"(t1), [t2] "=&v"(t2),
              [t3] "=&v"(t3), [n] "=&s"(n), [b1] "=&s"(b1), [b2] "=&s"(b2), [b3] "=&s"(b3), [nx] "+s"(nxt), [bt] "+v"(bits[i >> 1])
            : [s] "s"(s), [a0] "v"(acc[i][0][r]), [a1] "v"(acc[i][1][r]), [a2] "v"(acc[i][2][r]), [a3] "v"(acc[i][3][r])
            : "vcc", "scc", "s84", "s85", "s86", "s87", "s88", "s89", "s90", "s91", "memory");
      }
    unsigned base = nxt - pk_lds;
    if (tmo_v == call) {
      bits[0] = bits[1] = 0xffffffffu;
      base = 512u + 16384u;
      for (int o = 0; o < 64; ++o) *(float*)(pk + 512 + o * 256 + lane * 4) = __builtin_nanf("");
    }
    *(uint2*)(pk + lane * 8) = make_uint2(bits[0], bits[1]);
    const unsigned bytes = (base + 63u) & ~63u;
    STAMP(9);

    if (tid == 0) {
      const unsigned t = ticket;
      const unsigned* pd = (const unsigned*)(myws + WS_PDONE + g * 128);
      const unsigned* qd = (const unsigned*)(myws + WS_QDONE + g * 128);
      const unsigned long long t0 = __builtin_amdgcn_s_memrealtime();
      while (ld_agent(pd) < call * NPUSH || (int)(ld_agent(qd) + NTOK - t) <= 0) {
        if (ld_agent(tmo) == call || __builtin_amdgcn_s_memrealtime() - t0 > DEADLINE_TICKS) {
          st_agent(tmo, call);
          break;
        }
        __builtin_amdgcn_s_sleep(8);
      }
    }
    asm volatile("s_waitcnt lgkmcnt(0)" ::: "memory");
    __builtin_amdgcn_s_barrier();
    asm volatile("" ::: "memory");
    STAMP(6);
    {
      const __amdgpu_buffer_rsrc_t rp = make_rsrc(p.ws[g] + WS_PK, (unsigned)PK_BYTES);
      const unsigned slot = (unsigned)((rank * 32 + idx_in) * 8 + wave) * PK_SLOT;
#pragma unroll
      for (int o = 0; o < 17; ++o) {
        const unsigned off = o * 1024u + lane * 16u;
        if (o * 1024u < bytes) {
          const v4u v = *(const v4u*)(pk + off);
          if (off < bytes) __builtin_amdgcn_raw_buffer_store_b128(v, rp, slot + off, 0, AUX_SYS);
        }
        if (o == REL_AT) {
          asm volatile("" ::: "memory");
          __builtin_amdgcn_s_barrier();
          if (tid == 0)
            __hip_atomic_fetch_add((unsigned*)(myws + WS_QDONE + g * 128), 1u, __ATOMIC_RELAXED, __HIP_MEMORY_SCOPE_AGENT);
          asm volatile("" ::: "memory");
        }
      }
    }
    asm volatile("s_waitcnt vmcnt(0)" ::: "memory");
    __builtin_amdgcn_s_barrier();
    STAMP(4);
    if (tid == 0) st_sys((unsigned*)(p.ws[g] + WS_OFLAG + ((size_t)rank * 32 + idx_in) * 128), call);

    if (wave == 0) {
      const unsigned* f = lane == 0 ? (const unsigned*)(myws + WS_OFLAG + ((size_t)g * 32 + idx_in) * 128) : nullptr;
      wait_flags<false>(f, call, tmo);
    }
    asm volatile("" ::: "memory");
    __builtin_amdgcn_s_barrier();
    asm volatile("" ::: "memory");
    STAMP(5);
    const bool bad2 = ld_agent(tmo) == call;
    {
      const __amdgpu_buffer_rsrc_t rq = make_rsrc(myws + WS_PK, (unsigned)PK_BYTES);
      const unsigned src = (unsigned)((g * 32 + idx_in) * 8 + wave) * PK_SLOT;
#pragma unroll
      for (int o = 0; o < 17; ++o) bufld16(rq, pk + o * 1024, lane * 16, src + o * 1024, AUX_SYS);
      asm volatile("s_waitcnt vmcnt(0)" ::: "memory");
      STAMP(8);
      const uint2 mm = *(const uint2*)(pk + lane * 8);
      *(float*)(pk + 17152u + lane * 4u) = 0.f;
      const unsigned zero_at = pk_lds + 17152u + lane * 4u;
      unsigned rb = pk_lds + 512u;
      unsigned mb[2] = {mm.x, mm.y};
#define DEC4(v0, v1, v2, v3) \
            "v_add_co_u32_e64 %[mb], s[84:85], %[mb], %[mb]\n\t" \
            "v_add_co_u32_e64 %[mb], s[86:87], %[mb], %[mb]\n\t" \
            "v_add_co_u32_e64 %[mb], s[88:89], %[mb], %[mb]\n\t" \
            "v_add_co_u32_e64 %[mb], s[90:91], %[mb], %[mb]\n\t" \
            "s_bcnt1_i32_b64 %[n], s[84:85]\n\t" \
            "s_lshl2_add_u32 %[b1], %[n], %[rb]\n\t" \
            "s_bcnt1_i32_b64 %[n], s[86:87]\n\t" \
            "s_lshl2_add_u32 %[b2], %[n], %[b1]\n\t" \
            "s_bcnt1_i32_b64 %[n], s[88:89]\n\t" \
            "s_lshl2_add_u32 %[b3], %[n], %[b2]\n\t" \
            "v_mbcnt_lo_u32_b32 " v0 ", s84, 0\n\t" \
            "v_mbcnt_lo_u32_b32 " v1 ", s86, 0\n\t" \
            "v_mbcnt_lo_u32_b32 " v2 ", s88, 0\n\t" \
            "v_mbcnt_lo_u32_b32 " v3 ", s90, 0\n\t" \
            "v_mbcnt_hi_u32_b32 " v0 ", s85, " v0 "\n\t" \
            "v_mbcnt_hi_u32_b32 " v1 ", s87, " v1 "\n\t" \
            "v_mbcnt_hi_u32_b32 " v2 ", s89, " v2 "\n\t" \
            "v_mbcnt_hi_u32_b32 " v3 ", s91, " v3 "\n\t" \
            "v_lshl_add_u32 " v0 ", " v0 ", 2, %[rb]\n\t" \
            "v_lshl_add_u32 " v1 ", " v1 ", 2, %[b1]\n\t" \
            "v_lshl_add_u32 " v2 ", " v2 ", 2, %[b2]\n\t" \
            "v_lshl_add_u32 " v3 ", " v3 ", 2, %[b3]\n\t" \
            "s_bcnt1_i32_b64 %[n], s[90:91]\n\t" \
            "s_lshl2_add_u32 %[rb], %[n], %[b3]\n\t" \
            "v_cndmask_b32_e64 " v0 ", %[z], " v0 ", s[84:85]\n\t" \
            "v_cndmask_b32_e64 " v1 ", %[z], " v1 ", s[86:87]\n\t" \
            "v_cndmask_b32_e64 " v2 ", %[z], " v2 ", s[88:89]\n\t" \
            "v_cndmask_b32_e64 " v3 ", %[z], " v3 ", s[90:91]\n\t" \
            "ds_read_b32 " v0 ", " v0 "\n\t" \
            "ds_read_b32 " v1 ", " v1 "\n\t" \
            "ds_read_b32 " v2 ", " v2 "\n\t" \
            "ds_read_b32 " v3 ", " v3 "\n\t"
#pragma unroll
      for (int i = 0; i < 4; ++i) {
        unsigned e[16], n, b1, b2, b3;
        asm volatile(
            DEC4("%[e0]", "%[e1]", "%[e2]", "%[e3]")
            DEC4("%[e4]", "%[e5]", "%[e6]", "%[e7]")
            DEC4("%[e8]", "%[e9]", "%[e10]", "%[e11]")
            DEC4("%[e12]", "%[e13]", "%[e14]", "%[e15]")
            "s_waitcnt lgkmcnt(0)"
            : [e0] "=&v"(e[0]), [e1] "=&v"(e[1]), [e2] "=&v"(e[2]), [e3] "=&v"(e[3]), [e4] "=&v"(e[4]), [e5] "=&v"(e[5]),
              [e6] "=&v"(e[6]), [e7] "=&v"(e[7]), [e8] "=&v"(e[8]), [e9] "=&v"(e[9]), [e10] "=&v"(e[10]), [e11] "=&v"(e[11]),
              [e12] "=&v"(e[12]), [e13] "=&v"(e[13]), [e14] "=&v"(e[14]), [e15] "=&v"(e[15]),
              [n] "=&s"(n), [b1] "=&s"(b1), [b2] "=&s"(b2), [b3] "=&s"(b3), [rb] "+s"(rb), [mb] "+v"(mb[i >> 1])
            : [z] "v"(zero_at)
            : "scc", "s84", "s85", "s86", "s87", "s88", "s89", "s90", "s91", "memory");
#pragma unroll
        for (int r = 0; r < 4; ++r) {
          v4u v = {e[r * 4], e[r * 4 + 1], e[r * 4 + 2], e[r * 4 + 3]};
          if (bad2) v = v4u{0x7fc00000u, 0x7fc00000u, 0x7fc00000u, 0x7fc00000u};
          __builtin_amdgcn_raw_buffer_store_b128(v, ro, row_off + col_off + g * (NS * 4) + (unsigned)(i * 16 + r) * (GN * 4), 0, DEC_AUX);
        }
      }
    }
  }
  STAMP(7);
  if (tid == 0) {
    const unsigned t = __hip_atomic_fetch_add((unsigned*)(myws + WS_EXIT), 1u, __ATOMIC_RELAXED, __HIP_MEMORY_SCOPE_AGENT);
    if ((t + 1u) % NGRID == 0u) st_agent((unsigned*)(myws + WS_CALL), call);
  }
}

}

extern "C" void kernel_launch(int rank, int nranks, ncclComm_t comm,
                              hipStream_t stream,
                              void* const* d_in, const int* in_sizes, int n_in,
                              void* d_out, int out_size,
                              const int* in_dtypes, int out_dtype,
                              void* const* d_out_all,
                              void* const* d_ws_all, size_t ws_bytes) {
  (void)comm; (void)in_dtypes; (void)out_dtype; (void)d_out;
  if (nranks != WORLD || n_in != 4 || ws_bytes < WS_NEED || (unsigned)rank >= (unsigned)WORLD ||
      in_sizes[0] != GM * KS || in_sizes[1] != KS * GN || out_size != MB * GN) {
    fprintf(stderr, "kernel_launch: unexpected problem shape\n");
    return;
  }
  Params p;
  p.x = (const uint8_t*)d_in[0];
  p.w = (const uint8_t*)d_in[1];
  p.sx = (const float*)d_in[2];
  p.sw = (const float*)d_in[3];
  for (int r = 0; r < WORLD; ++r) {
    p.out[r] = (float*)d_out_all[r];
    p.ws[r] = (uint8_t*)d_ws_all[r];
  }
  p.rank = rank;
  gemm_rs_kernel<<<dim3(NGRID), dim3(NTHREADS), 0, stream>>>(p);
  HIP_CHECK(hipGetLastError());
}
```

```cpp
#include <hip/hip_runtime.h>
#include <rccl/rccl.h>
#include <cstdio>
#include <cstdint>

#define HIP_CHECK(expr) do { hipError_t _e = (expr); if (_e != hipSuccess) { \
    fprintf(stderr, "HIP error %s at %s:%d\n", hipGetErrorString(_e), __FILE__, __LINE__); \
    return; } } while (0)

namespace {

constexpr int WORLD = 8;
constexpr int GM = 4096, GK = 4096, GN = 16384;
constexpr int KS = GK / WORLD;
constexpr int NS = GN / WORLD;
constexpr int MB = GM / WORLD;
constexpr int BM = 128, BN = 256, BK = 128;
constexpr int NKB = GK / BK;
constexpr int A_STAGE = BM * BK;
constexpr int B_STAGE = BN * BK;
constexpr int STAGE = A_STAGE + B_STAGE;
constexpr int NSTAGE = 3;
constexpr int NTHREADS = 512;
constexpr int NGRID = 256;
constexpr int NPUSH = 8;
constexpr unsigned NTOK = 2;
constexpr int REL_AT = 4;

constexpr size_t WS_CALL = 0;
constexpr size_t WS_EXIT = 128;
constexpr size_t WS_TMO = 256;
constexpr size_t WS_QTICK = 1024;
constexpr size_t WS_QDONE = 2048;
constexpr size_t WS_PDONE = 3072;
constexpr size_t WS_XFLAG = 4096;
constexpr size_t WS_WFLAG = 8192;
constexpr size_t WS_PQ = 12288;
constexpr size_t WS_OFLAG = 16384;
constexpr size_t WS_STAGE = 1u << 20;
constexpr size_t XS_BYTES = (size_t)GM * GK;
constexpr size_t WT_BYTES = (size_t)GK * NS;
constexpr size_t PAR_BYTES = XS_BYTES + WT_BYTES;
constexpr unsigned PK_SLOT = 17408;
constexpr size_t WS_PK = WS_STAGE + 2 * PAR_BYTES;
constexpr size_t PK_BYTES = (size_t)WORLD * 32 * 8 * PK_SLOT;
constexpr size_t WS_NEED = WS_PK + PK_BYTES;
constexpr size_t OUT_BYTES = (size_t)MB * GN * 4;

constexpr unsigned long long DEADLINE_TICKS = 50000000ull;

typedef int v4i __attribute__((ext_vector_type(4)));
typedef unsigned v4u __attribute__((ext_vector_type(4)));
typedef int v8i __attribute__((ext_vector_type(8)));
typedef float v4f __attribute__((ext_vector_type(4)));

struct Params {
  const uint8_t* x;
  const uint8_t* w;
  const float* sx;
  const float* sw;
  float* out[WORLD];
  uint8_t* ws[WORLD];
  int rank;
};

#define AUX_SYS 17

__device__ __forceinline__ __amdgpu_buffer_rsrc_t make_rsrc(void* p, unsigned bytes) {
  return __builtin_amdgcn_make_buffer_rsrc(p, 0, bytes, 0x00020000);
}
#define bufld16(rs, l, voff, soff, aux) \
  __builtin_amdgcn_raw_ptr_buffer_load_lds(rs, (__attribute__((address_space(3))) void*)(l), 16, voff, soff, 0, aux)
__device__ __forceinline__ unsigned ld_sys(const unsigned* p) {
  return __hip_atomic_load(p, __ATOMIC_RELAXED, __HIP_MEMORY_SCOPE_SYSTEM);
}
__device__ __forceinline__ unsigned ld_agent(const unsigned* p) {
  return __hip_atomic_load(p, __ATOMIC_RELAXED, __HIP_MEMORY_SCOPE_AGENT);
}
__device__ __forceinline__ void st_sys(unsigned* p, unsigned v) {
  __hip_atomic_store(p, v, __ATOMIC_RELAXED, __HIP_MEMORY_SCOPE_SYSTEM);
}
__device__ __forceinline__ void st_agent(unsigned* p, unsigned v) {
  __hip_atomic_store(p, v, __ATOMIC_RELAXED, __HIP_MEMORY_SCOPE_AGENT);
}

__device__ __forceinline__ void wait_flag(const unsigned* flag, unsigned call, unsigned* tmo) {
  const unsigned long long t0 = __builtin_amdgcn_s_memrealtime();
  for (;;) {
    unsigned v = call;
    if (flag) v = ld_sys(flag);
    if (__all(v >= call)) break;
    if (ld_agent(tmo) == call || __builtin_amdgcn_s_memrealtime() - t0 > DEADLINE_TICKS) {
      st_agent(tmo, call);
      break;
    }
    __builtin_amdgcn_s_sleep(2);
  }
  asm volatile("s_waitcnt vmcnt(0)" ::: "memory");
}

__global__ void __launch_bounds__(NTHREADS) gemm_rs_kernel(Params p) {
  __shared__ __attribute__((aligned(16))) uint8_t smem[NSTAGE * STAGE + 64];

  const int tid = threadIdx.x;
  const int lane = tid & 63;
  const int wave = __builtin_amdgcn_readfirstlane(tid >> 6);
  const unsigned g = blockIdx.x & 7u;
  const int idx_in = blockIdx.x >> 3;
  const int rank = p.rank;
  uint8_t* const myws = p.ws[rank];
  unsigned* const tmo = (unsigned*)(myws + WS_TMO);
  const unsigned call_v = ld_agent((const unsigned*)(myws + WS_CALL));

  const int c16 = lane & 15, gq = lane >> 4;

  if (idx_in < NPUSH) {
    uint8_t* const dws = p.ws[g];
    auto* const lds_next = (__attribute__((address_space(3))) unsigned*)smem;
    const int wn = wave & 3;
    const uint8_t* const wsrc = p.w + (size_t)((wave >> 2) * 128 + (idx_in & 1) * 256 + gq * 16) * GN + (size_t)g * NS + (idx_in >> 1) * BN + wn * 64 + c16 * 4;
    const unsigned wdst = (unsigned)XS_BYTES + (unsigned)(idx_in >> 1) * (NKB * B_STAGE) + (unsigned)(rank * 4 + (idx_in & 1) * 2 + (wave >> 2)) * B_STAGE +
                          (unsigned)wn * 4 * 2048 + lane * 16;
    const unsigned xsrc = (unsigned)(wave * 16 + c16) * KS + gq * 16;
    const unsigned xdst = (unsigned)(rank * 4) * A_STAGE + (unsigned)wave * 2048 + lane * 16;
    unsigned* const pq = (unsigned*)(myws + WS_PQ + g * 128);

    auto load_x = [&](v4u (&b)[8], int tmq) {
      const __amdgpu_buffer_rsrc_t rx = make_rsrc((void*)p.x, (unsigned)tmq < 32u ? (unsigned)(GM * KS) : 0u);
#pragma unroll
      for (int u = 0; u < 8; ++u) b[u] = __builtin_amdgcn_raw_buffer_load_b128(rx, xsrc, (tmq & 31) * (BM * KS) + u * 64, 0);
    };
    auto load_w = [&](unsigned (&b)[32], int half) {
#pragma unroll
      for (int t = 0; t < 32; ++t) b[t] = *(const unsigned*)(wsrc + half * (4 * BN) + (size_t)((t >> 4) * 64 + (t & 15)) * GN);
    };
    v4u xa[8], xb[8];
    unsigned wa[32], wb[32];
    load_x(xa, idx_in * 4);
    asm volatile("" ::: "memory");
    load_w(wa, 0);
    asm volatile("" ::: "memory");
    const unsigned call = __builtin_amdgcn_readfirstlane(call_v) + 1u;
    const __amdgpu_buffer_rsrc_t rs = make_rsrc(dws + WS_STAGE + (size_t)(call & 1u) * PAR_BYTES, (unsigned)PAR_BYTES);
    const unsigned pq_base = (call - 1u) * (24u + NPUSH);

    auto store_x = [&](const v4u (&b)[8], int tmq) {
      const unsigned dst = xdst + (unsigned)tmq * (NKB * A_STAGE);
#pragma unroll
      for (int u = 0; u < 8; ++u)
        __builtin_amdgcn_raw_buffer_store_b128(b[u], rs, dst + (u >> 1) * A_STAGE + (u & 1) * 1024, 0, AUX_SYS);
    };
    auto store_w = [&](const unsigned (&b)[32], int half) {
      const unsigned dst = wdst + (unsigned)half * (4u * NKB * B_STAGE);
#pragma unroll
      for (int qh = 0; qh < 2; ++qh) {
        v4u o[4];
#pragma unroll
        for (int u = 0; u < 4; ++u) {
          const unsigned* v = b + qh * 16 + 4 * u;
          const unsigned t0 = __builtin_amdgcn_perm(v[1], v[0], 0x05010400u);
          const unsigned t1 = __builtin_amdgcn_perm(v[1], v[0], 0x07030602u);
          const unsigned t2 = __builtin_amdgcn_perm(v[3], v[2], 0x05010400u);
          const unsigned t3 = __builtin_amdgcn_perm(v[3], v[2], 0x07030602u);
          o[0][u] = __builtin_amdgcn_perm(t2, t0, 0x05040100u);
          o[1][u] = __builtin_amdgcn_perm(t2, t0, 0x07060302u);
          o[2][u] = __builtin_amdgcn_perm(t3, t1, 0x05040100u);
          o[3][u] = __builtin_amdgcn_perm(t3, t1, 0x07060302u);
        }
#pragma unroll
        for (int j = 0; j < 4; ++j)
          __builtin_amdgcn_raw_buffer_store_b128(o[j], rs, dst + j * 2048 + qh * 1024, 0, AUX_SYS);
      }
    };
    auto xflag = [&](int tmq) { return (unsigned*)(dws + WS_XFLAG + (size_t)tmq * 128) + rank; };
    auto wflag = [&](int half) { return (unsigned*)(dws + WS_WFLAG + (size_t)(half * 8 + idx_in) * 128) + rank; };
#define PUSH_FENCE() asm volatile("" ::: "memory")
#define PUSH_PUBLISH(N, flagptr) do { \
      asm volatile("s_waitcnt vmcnt(" #N ")" ::: "memory"); \
      __builtin_amdgcn_s_barrier(); PUSH_FENCE(); \
      if (tid == 0) st_sys(flagptr, call); \
      PUSH_FENCE(); } while (0)
#define PUSH_DRAW() do { if (tid == 0) { \
      const unsigned d = __hip_atomic_fetch_add(pq, 1u, __ATOMIC_RELAXED, __HIP_MEMORY_SCOPE_AGENT) - pq_base; \
      *lds_next = d < 24u ? (d & 7u) * 4u + 1u + (d >> 3) : 32u; } } while (0)
    auto ticket_done = [&]() -> int {
      asm volatile("s_waitcnt lgkmcnt(0)" ::: "memory");
      __builtin_amdgcn_s_barrier();
      PUSH_FENCE();
      return __builtin_amdgcn_readfirstlane(*lds_next);
    };
    store_x(xa, idx_in * 4); PUSH_FENCE();
    load_w(wb, 1); PUSH_FENCE();
    PUSH_PUBLISH(32, xflag(idx_in * 4));
    PUSH_DRAW();
    store_w(wa, 0); PUSH_FENCE();
    int ta = ticket_done();
    load_x(xa, ta); PUSH_FENCE();
    PUSH_PUBLISH(8, wflag(0));
    if (ta < 32) PUSH_DRAW();
    store_w(wb, 1); PUSH_FENCE();
    int tb = 32;
    if (ta < 32) tb = ticket_done();
    else if (tid == 0) __hip_atomic_fetch_add((unsigned*)(myws + WS_PDONE + g * 128), 1u, __ATOMIC_RELAXED, __HIP_MEMORY_SCOPE_AGENT);
    load_x(xb, tb); PUSH_FENCE();
    PUSH_PUBLISH(8, wflag(1));
    for (;;) {
      if (ta >= 32) break;
      {
        const int cur = ta;
        if (tb < 32) PUSH_DRAW();
        store_x(xa, cur); PUSH_FENCE();
        if (tb < 32) ta = ticket_done();
        else {
          ta = 32;
          if (tid == 0) __hip_atomic_fetch_add((unsigned*)(myws + WS_PDONE + g * 128), 1u, __ATOMIC_RELAXED, __HIP_MEMORY_SCOPE_AGENT);
        }
        load_x(xa, ta); PUSH_FENCE();
        PUSH_PUBLISH(8, xflag(cur));
      }
      if (tb >= 32) break;
      {
        const int cur = tb;
        if (ta < 32) PUSH_DRAW();
        store_x(xb, cur); PUSH_FENCE();
        if (ta < 32) tb = ticket_done();
        else {
          tb = 32;
          if (tid == 0) __hip_atomic_fetch_add((unsigned*)(myws + WS_PDONE + g * 128), 1u, __ATOMIC_RELAXED, __HIP_MEMORY_SCOPE_AGENT);
        }
        load_x(xb, tb); PUSH_FENCE();
        PUSH_PUBLISH(8, xflag(cur));
      }
    }
  }
  const unsigned call = __builtin_amdgcn_readfirstlane(call_v) + 1u;
  const size_t stage_off = WS_STAGE + (size_t)(call & 1u) * PAR_BYTES;

  const int idx = (idx_in - NPUSH) & 31;
  const int mt = idx >> 3, nt = idx & 7;
  const int tm = (int)g * 4 + mt;
  auto* const lds_sel = (__attribute__((address_space(3))) int*)(smem + NSTAGE * STAGE);
  const unsigned* myflag = nullptr;
  if (lane < 8) myflag = (const unsigned*)(myws + WS_XFLAG + (size_t)tm * 128) + lane;
  else if (lane < 24) myflag = (const unsigned*)(myws + WS_WFLAG + (size_t)(nt * 2 + ((lane - 8) >> 3)) * 128) + (lane & 7);
  unsigned rem = 0xffu, ready = 0u;
  auto select = [&]() {
    const unsigned long long t0 = __builtin_amdgcn_s_memrealtime();
    while ((ready & rem) == 0u) {
      unsigned v = call;
      if (myflag) v = ld_sys(myflag);
      const unsigned m = (unsigned)__builtin_amdgcn_ballot_w64(v >= call);
      ready = m & (m >> 8) & (m >> 16) & 0xffu;
      if (ready & rem) break;
      if (ld_agent(tmo) == call || __builtin_amdgcn_s_memrealtime() - t0 > DEADLINE_TICKS) {
        if (lane == 0) st_agent(tmo, call);
        ready = 0xffu;
        break;
      }
      __builtin_amdgcn_s_sleep(4);
    }
    const unsigned cand = ready & rem;
    const int sel = (cand >> rank) & 1u ? rank : __builtin_ctz(cand);
    rem &= ~(1u << sel);
    if (lane == 0) *lds_sel = sel;
  };
  if (wave == 0) {
    select();
    __builtin_amdgcn_fence(__ATOMIC_ACQUIRE, "");
    asm volatile("s_waitcnt vmcnt(0) lgkmcnt(0)" ::: "memory");
  }
  __builtin_amdgcn_s_barrier();
  asm volatile("" ::: "memory");
  int src = __builtin_amdgcn_readfirstlane(*lds_sel);

  const int wm = wave >> 2, wn = wave & 3;
  v4f acc[4][4];
#pragma unroll
  for (int i = 0; i < 4; ++i)
#pragma unroll
    for (int j = 0; j < 4; ++j) acc[i][j] = v4f{0.f, 0.f, 0.f, 0.f};
  const float s = p.sx[0] * p.sw[0];
  unsigned ticket = 0;

  {
    uint8_t* const sbase = myws + stage_off;
    const unsigned va = (unsigned)tm * (NKB * A_STAGE) + wave * 2048 + lane * 16;
    const unsigned vb = (unsigned)XS_BYTES + (unsigned)nt * (NKB * B_STAGE) + wave * 4096 + lane * 16;
    auto issue1 = [&](unsigned nrec, int kb, int boff, int c) {
      const __amdgpu_buffer_rsrc_t rs = make_rsrc(sbase, nrec);
      if (c < 2) bufld16(rs, smem + boff + wave * 2048 + c * 1024, va, kb * A_STAGE + c * 1024, 0);
      else bufld16(rs, smem + boff + A_STAGE + wave * 4096 + (c - 2) * 1024, vb, kb * B_STAGE + (c - 2) * 1024, 0);
    };
#pragma unroll
    for (int c = 0; c < 6; ++c) issue1((unsigned)PAR_BYTES, src * 4, 0, c);
#pragma unroll
    for (int c = 0; c < 6; ++c) issue1((unsigned)PAR_BYTES, src * 4 + 1, STAGE, c);
    int boff = 0;
    const int a_rd = wm * 4 * 2048 + lane * 16;
    const int b_rd = A_STAGE + wn * 4 * 2048 + lane * 16;
    auto step = [&](unsigned nrec, int pkb) {
      int nb = boff + 2 * STAGE;
      if (nb >= NSTAGE * STAGE) nb -= NSTAGE * STAGE;
      const uint8_t* sa = smem + boff + a_rd;
      const uint8_t* sb = smem + boff + b_rd;
      v8i af[4], bf[4];
      auto rd = [&](const uint8_t* q) {
        const v4i lo = *(const v4i*)q, hi = *(const v4i*)(q + 1024);
        return __builtin_shufflevector(lo, hi, 0, 1, 2, 3, 4, 5, 6, 7);
      };
      af[0] = rd(sa);
#pragma unroll
      for (int j = 0; j < 4; ++j) bf[j] = rd(sb + j * 2048);
#pragma unroll
      for (int i = 1; i < 4; ++i) af[i] = rd(sa + i * 2048);
      __builtin_amdgcn_sched_barrier(0);
#pragma unroll
      for (int i = 0; i < 4; ++i) {
#pragma unroll
        for (int j = 0; j < 4; ++j) {
          acc[i][j] = __builtin_amdgcn_mfma_scale_f32_16x16x128_f8f6f4(af[i], bf[j], acc[i][j], 0  , 1  , 0, 0, 0, 0);
          if ((j & 1) && i < 3) {
            __builtin_amdgcn_sched_barrier(0);
            issue1(nrec, pkb, nb, i * 2 + (j >> 1));
            __builtin_amdgcn_sched_barrier(0);
          }
        }
      }
      __builtin_amdgcn_s_waitcnt(0xC07F);
      boff += STAGE;
      if (boff >= NSTAGE * STAGE) boff = 0;
    };
#define GEMM_SYNC() do { asm volatile("s_waitcnt vmcnt(6)" ::: "memory"); __builtin_amdgcn_s_barrier(); asm volatile("" ::: "memory"); } while (0)
    __builtin_amdgcn_s_waitcnt(0xC07F);
    for (int slot = 0; slot < WORLD; ++slot) {
      const bool more = slot + 1 < WORLD;
      const unsigned nrec = more ? (unsigned)PAR_BYTES : 0u;
      GEMM_SYNC();
      step((unsigned)PAR_BYTES, src * 4 + 2);
      GEMM_SYNC();
      step((unsigned)PAR_BYTES, src * 4 + 3);
      if (more && wave == 0) {
        select();
        if (slot == WORLD - 2) {
          if (lane == 0 && g != (unsigned)rank)
            ticket = __hip_atomic_fetch_add((unsigned*)(myws + WS_QTICK + g * 128), 1u, __ATOMIC_RELAXED, __HIP_MEMORY_SCOPE_AGENT);
        }
        asm volatile("s_waitcnt lgkmcnt(0)" ::: "memory");
      }
      GEMM_SYNC();
      const int nsrc = more ? __builtin_amdgcn_readfirstlane(*lds_sel) : 0;
      __builtin_amdgcn_s_waitcnt(0xC07F);
      step(nrec, nsrc * 4);
      GEMM_SYNC();
      step(nrec, nsrc * 4 + 1);
      src = nsrc;
    }
  }
  const unsigned tmo_v = ld_agent(tmo);
  asm volatile("s_waitcnt vmcnt(1)" ::: "memory");

  const __amdgpu_buffer_rsrc_t ro = make_rsrc(p.out[rank], (unsigned)OUT_BYTES);
  const unsigned row_off = (unsigned)(mt * BM + wm * 64 + gq * 4) * GN * 4u;
  const unsigned col_off = (unsigned)(nt * BN + wn * 64 + c16 * 4) * 4u;

  if (g == (unsigned)rank) {
    const bool bad = tmo_v == call;
#pragma unroll
    for (int i = 0; i < 4; ++i)
#pragma unroll
      for (int r = 0; r < 4; ++r) {
        v4f v;
#pragma unroll
        for (int j = 0; j < 4; ++j) {
          const float y = acc[i][j][r] * s;
          v[j] = (y <= 0.f) ? 0.f : y;
          if (bad) v[j] = __builtin_nanf("");
        }
        __builtin_amdgcn_raw_buffer_store_b128(__builtin_bit_cast(v4u, v), ro,
                                               row_off + col_off + (unsigned)(rank * NS * 4) + (unsigned)(i * 16 + r) * (GN * 4), 0, AUX_SYS);
      }
  } else {
    asm volatile("s_waitcnt lgkmcnt(0)" ::: "memory");
    __builtin_amdgcn_s_barrier();
    asm volatile("" ::: "memory");
    uint8_t* const pk = smem + wave * PK_SLOT;
    const unsigned pk_lds = (unsigned)(unsigned long)((__attribute__((address_space(3))) uint8_t*)smem) + wave * PK_SLOT;
    unsigned nxt = pk_lds + 512u;
    unsigned bits[2] = {0u, 0u};
#pragma unroll
    for (int i = 0; i < 4; ++i)
#pragma unroll
      for (int r = 0; r < 4; ++r) {
        float y0, y1, y2, y3;
        unsigned t0, t1, t2, t3, n, b1, b2, b3;
        asm volatile(
            "v_mul_f32 %[y0], %[s], %[a0]\n\t"
            "v_mul_f32 %[y1], %[s], %[a1]\n\t"
            "v_mul_f32 %[y2], %[s], %[a2]\n\t"
            "v_mul_f32 %[y3], %[s], %[a3]\n\t"
            "v_cmp_nge_f32_e64 s[84:85], 0, %[y0]\n\t"
            "v_cmp_nge_f32_e64 s[86:87], 0, %[y1]\n\t"
            "v_cmp_nge_f32_e64 s[88:89], 0, %[y2]\n\t"
            "v_cmp_nge_f32_e64 s[90:91], 0, %[y3]\n\t"
            "s_bcnt1_i32_b64 %[n], s[84:85]\n\t"
            "s_lshl2_add_u32 %[b1], %[n], %[nx]\n\t"
            "s_bcnt1_i32_b64 %[n], s[86:87]\n\t"
            "s_lshl2_add_u32 %[b2], %[n], %[b1]\n\t"
            "s_bcnt1_i32_b64 %[n], s[88:89]\n\t"
            "s_lshl2_add_u32 %[b3], %[n], %[b2]\n\t"
            "v_mbcnt_lo_u32_b32 %[t0], s84, 0\n\t"
            "v_mbcnt_lo_u32_b32 %[t1], s86, 0\n\t"
            "v_mbcnt_lo_u32_b32 %[t2], s88, 0\n\t"
            "v_mbcnt_lo_u32_b32 %[t3], s90, 0\n\t"
            "v_mbcnt_hi_u32_b32 %[t0], s85, %[t0]\n\t"
            "v_mbcnt_hi_u32_b32 %[t1], s87, %[t1]\n\t"
            "v_mbcnt_hi_u32_b32 %[t2], s89, %[t2]\n\t"
            "v_mbcnt_hi_u32_b32 %[t3], s91, %[t3]\n\t"
            "v_lshl_add_u32 %[t0], %[t0], 2, %[nx]\n\t"
            "v_lshl_add_u32 %[t1], %[t1], 2, %[b1]\n\t"
            "v_lshl_add_u32 %[t2], %[t2], 2, %[b2]\n\t"
            "v_lshl_add_u32 %[t3], %[t3], 2, %[b3]\n\t"
            "s_bcnt1_i32_b64 %[n], s[90:91]\n\t"
            "s_lshl2_add_u32 %[nx], %[n], %[b3]\n\t"
            "v_addc_co_u32_e64 %[bt], vcc, %[bt], %[bt], s[84:85]\n\t"
            "v_addc_co_u32_e64 %[bt], vcc, %[bt], %[bt], s[86:87]\n\t"
            "v_addc_co_u32_e64 %[bt], vcc, %[bt], %[bt], s[88:89]\n\t"
            "v_addc_co_u32_e64 %[bt], vcc, %[bt], %[bt], s[90:91]\n\t"
            "s_mov_b64 exec, s[84:85]\n\t"
            "ds_write_b32 %[t0], %[y0]\n\t"
            "s_mov_b64 exec, s[86:87]\n\t"
            "ds_write_b32 %[t1], %[y1]\n\t"
            "s_mov_b64 exec, s[88:89]\n\t"
            "ds_write_b32 %[t2], %[y2]\n\t"
            "s_mov_b64 exec, s[90:91]\n\t"
            "ds_write_b32 %[t3], %[y3]\n\t"
            "s_mov_b64 exec, -1"
            : [y0] "=&v"(y0), [y1] "=&v"(y1), [y2] "=&v"(y2), [y3] "=&v"(y3), [t0] "=&v"(t0), [t1] "=&v"(t1), [t2] "=&v"(t2),
              [t3] "=&v"(t3), [n] "=&s"(n), [b1] "=&s"(b1), [b2] "=&s"(b2), [b3] "=&s"(b3), [nx] "+s"(nxt), [bt] "+v"(bits[i >> 1])
            : [s] "s"(s), [a0] "v"(acc[i][0][r]), [a1] "v"(acc[i][1][r]), [a2] "v"(acc[i][2][r]), [a3] "v"(acc[i][3][r])
            : "vcc", "scc", "s84", "s85", "s86", "s87", "s88", "s89", "s90", "s91", "memory");
      }
    unsigned base = nxt - pk_lds;
    if (tmo_v == call) {
      bits[0] = bits[1] = 0xffffffffu;
      base = 512u + 16384u;
      for (int o = 0; o < 64; ++o) *(float*)(pk + 512 + o * 256 + lane * 4) = __builtin_nanf("");
    }
    *(uint2*)(pk + lane * 8) = make_uint2(bits[0], bits[1]);
    const unsigned bytes = (base + 63u) & ~63u;

    if (tid == 0) {
      const unsigned t = ticket;
      const unsigned* pd = (const unsigned*)(myws + WS_PDONE + g * 128);
      const unsigned* qd = (const unsigned*)(myws + WS_QDONE + g * 128);
      const unsigned long long t0 = __builtin_amdgcn_s_memrealtime();
      while (ld_agent(pd) < call * NPUSH || (int)(ld_agent(qd) + NTOK - t) <= 0) {
        if (ld_agent(tmo) == call || __builtin_amdgcn_s_memrealtime() - t0 > DEADLINE_TICKS) {
          st_agent(tmo, call);
          break;
        }
        __builtin_amdgcn_s_sleep(8);
      }
    }
    asm volatile("s_waitcnt lgkmcnt(0)" ::: "memory");
    __builtin_amdgcn_s_barrier();
    asm volatile("" ::: "memory");
    {
      const __amdgpu_buffer_rsrc_t rp = make_rsrc(p.ws[g] + WS_PK, (unsigned)PK_BYTES);
      const unsigned slot = (unsigned)((rank * 32 + idx_in) * 8 + wave) * PK_SLOT;
#pragma unroll
      for (int o = 0; o < 17; ++o) {
        const unsigned off = o * 1024u + lane * 16u;
        if (o * 1024u < bytes) {
          const v4u v = *(const v4u*)(pk + off);
          if (off < bytes) __builtin_amdgcn_raw_buffer_store_b128(v, rp, slot + off, 0, AUX_SYS);
        }
        if (o == REL_AT) {
          asm volatile("" ::: "memory");
          __builtin_amdgcn_s_barrier();
          if (tid == 0)
            __hip_atomic_fetch_add((unsigned*)(myws + WS_QDONE + g * 128), 1u, __ATOMIC_RELAXED, __HIP_MEMORY_SCOPE_AGENT);
          asm volatile("" ::: "memory");
        }
      }
    }
    asm volatile("s_waitcnt vmcnt(0)" ::: "memory");
    __builtin_amdgcn_s_barrier();
    if (tid == 0) st_sys((unsigned*)(p.ws[g] + WS_OFLAG + ((size_t)rank * 32 + idx_in) * 128), call);

    if (wave == 0) {
      const unsigned* f = lane == 0 ? (const unsigned*)(myws + WS_OFLAG + ((size_t)g * 32 + idx_in) * 128) : nullptr;
      wait_flag(f, call, tmo);
    }
    asm volatile("" ::: "memory");
    __builtin_amdgcn_s_barrier();
    asm volatile("" ::: "memory");
    const bool bad2 = ld_agent(tmo) == call;
    {
      const __amdgpu_buffer_rsrc_t rq = make_rsrc(myws + WS_PK, (unsigned)PK_BYTES);
      const unsigned src = (unsigned)((g * 32 + idx_in) * 8 + wave) * PK_SLOT;
#pragma unroll
      for (int o = 0; o < 17; ++o) bufld16(rq, pk + o * 1024, lane * 16, src + o * 1024, AUX_SYS);
      asm volatile("s_waitcnt vmcnt(0)" ::: "memory");
      const uint2 mm = *(const uint2*)(pk + lane * 8);
      *(float*)(pk + 17152u + lane * 4u) = 0.f;
      const unsigned zero_at = pk_lds + 17152u + lane * 4u;
      unsigned rb = pk_lds + 512u;
      unsigned mb[2] = {mm.x, mm.y};
#define DEC4(v0, v1, v2, v3) \
            "v_add_co_u32_e64 %[mb], s[84:85], %[mb], %[mb]\n\t" \
            "v_add_co_u32_e64 %[mb], s[86:87], %[mb], %[mb]\n\t" \
            "v_add_co_u32_e64 %[mb], s[88:89], %[mb], %[mb]\n\t" \
            "v_add_co_u32_e64 %[mb], s[90:91], %[mb], %[mb]\n\t" \
            "s_bcnt1_i32_b64 %[n], s[84:85]\n\t" \
            "s_lshl2_add_u32 %[b1], %[n], %[rb]\n\t" \
            "s_bcnt1_i32_b64 %[n], s[86:87]\n\t" \
            "s_lshl2_add_u32 %[b2], %[n], %[b1]\n\t" \
            "s_bcnt1_i32_b64 %[n], s[88:89]\n\t" \
            "s_lshl2_add_u32 %[b3], %[n], %[b2]\n\t" \
            "v_mbcnt_lo_u32_b32 " v0 ", s84, 0\n\t" \
            "v_mbcnt_lo_u32_b32 " v1 ", s86, 0\n\t" \
            "v_mbcnt_lo_u32_b32 " v2 ", s88, 0\n\t" \
            "v_mbcnt_lo_u32_b32 " v3 ", s90, 0\n\t" \
            "v_mbcnt_hi_u32_b32 " v0 ", s85, " v0 "\n\t" \
            "v_mbcnt_hi_u32_b32 " v1 ", s87, " v1 "\n\t" \
            "v_mbcnt_hi_u32_b32 " v2 ", s89, " v2 "\n\t" \
            "v_mbcnt_hi_u32_b32 " v3 ", s91, " v3 "\n\t" \
            "v_lshl_add_u32 " v0 ", " v0 ", 2, %[rb]\n\t" \
            "v_lshl_add_u32 " v1 ", " v1 ", 2, %[b1]\n\t" \
            "v_lshl_add_u32 " v2 ", " v2 ", 2, %[b2]\n\t" \
            "v_lshl_add_u32 " v3 ", " v3 ", 2, %[b3]\n\t" \
            "s_bcnt1_i32_b64 %[n], s[90:91]\n\t" \
            "s_lshl2_add_u32 %[rb], %[n], %[b3]\n\t" \
            "v_cndmask_b32_e64 " v0 ", %[z], " v0 ", s[84:85]\n\t" \
            "v_cndmask_b32_e64 " v1 ", %[z], " v1 ", s[86:87]\n\t" \
            "v_cndmask_b32_e64 " v2 ", %[z], " v2 ", s[88:89]\n\t" \
            "v_cndmask_b32_e64 " v3 ", %[z], " v3 ", s[90:91]\n\t" \
            "ds_read_b32 " v0 ", " v0 "\n\t" \
            "ds_read_b32 " v1 ", " v1 "\n\t" \
            "ds_read_b32 " v2 ", " v2 "\n\t" \
            "ds_read_b32 " v3 ", " v3 "\n\t"
#pragma unroll
      for (int i = 0; i < 4; ++i) {
        unsigned e[16], n, b1, b2, b3;
        asm volatile(
            DEC4("%[e0]", "%[e1]", "%[e2]", "%[e3]")
            DEC4("%[e4]", "%[e5]", "%[e6]", "%[e7]")
            DEC4("%[e8]", "%[e9]", "%[e10]", "%[e11]")
            DEC4("%[e12]", "%[e13]", "%[e14]", "%[e15]")
            "s_waitcnt lgkmcnt(0)"
            : [e0] "=&v"(e[0]), [e1] "=&v"(e[1]), [e2] "=&v"(e[2]), [e3] "=&v"(e[3]), [e4] "=&v"(e[4]), [e5] "=&v"(e[5]),
              [e6] "=&v"(e[6]), [e7] "=&v"(e[7]), [e8] "=&v"(e[8]), [e9] "=&v"(e[9]), [e10] "=&v"(e[10]), [e11] "=&v"(e[11]),
              [e12] "=&v"(e[12]), [e13] "=&v"(e[13]), [e14] "=&v"(e[14]), [e15] "=&v"(e[15]),
              [n] "=&s"(n), [b1] "=&s"(b1), [b2] "=&s"(b2), [b3] "=&s"(b3), [rb] "+s"(rb), [mb] "+v"(mb[i >> 1])
            : [z] "v"(zero_at)
            : "scc", "s84", "s85", "s86", "s87", "s88", "s89", "s90", "s91", "memory");
#pragma unroll
        for (int r = 0; r < 4; ++r) {
          v4u v = {e[r * 4], e[r * 4 + 1], e[r * 4 + 2], e[r * 4 + 3]};
          if (bad2) v = v4u{0x7fc00000u, 0x7fc00000u, 0x7fc00000u, 0x7fc00000u};
          __builtin_amdgcn_raw_buffer_store_b128(v, ro, row_off + col_off + g * (NS * 4) + (unsigned)(i * 16 + r) * (GN * 4), 0, AUX_SYS);
        }
      }
    }
  }
  if (tid == 0) {
    const unsigned t = __hip_atomic_fetch_add((unsigned*)(myws + WS_EXIT), 1u, __ATOMIC_RELAXED, __HIP_MEMORY_SCOPE_AGENT);
    if ((t + 1u) % NGRID == 0u) st_agent((unsigned*)(myws + WS_CALL), call);
  }
}

}

extern "C" void kernel_launch(int rank, int nranks, ncclComm_t comm,
                              hipStream_t stream,
                              void* const* d_in, const int* in_sizes, int n_in,
                              void* d_out, int out_size,
                              const int* in_dtypes, int out_dtype,
                              void* const* d_out_all,
                              void* const* d_ws_all, size_t ws_bytes) {
  (void)comm; (void)in_dtypes; (void)out_dtype; (void)d_out;
  if (nranks != WORLD || n_in != 4 || ws_bytes < WS_NEED || (unsigned)rank >= (unsigned)WORLD ||
      in_sizes[0] != GM * KS || in_sizes[1] != KS * GN || out_size != MB * GN) {
    fprintf(stderr, "kernel_launch: unexpected problem shape\n");
    return;
  }
  Params p;
  p.x = (const uint8_t*)d_in[0];
  p.w = (const uint8_t*)d_in[1];
  p.sx = (const float*)d_in[2];
  p.sw = (const float*)d_in[3];
  for (int r = 0; r < WORLD; ++r) {
    p.out[r] = (float*)d_out_all[r];
    p.ws[r] = (uint8_t*)d_ws_all[r];
  }
  p.rank = rank;
  gemm_rs_kernel<<<dim3(NGRID), dim3(NTHREADS), 0, stream>>>(p);
  HIP_CHECK(hipGetLastError());
}
```

```cpp
#include <hip/hip_runtime.h>
#include <rccl/rccl.h>
#include <cstdio>
#include <cstdint>

#define HIP_CHECK(expr) do { hipError_t _e = (expr); if (_e != hipSuccess) { \
    fprintf(stderr, "HIP error %s at %s:%d\n", hipGetErrorString(_e), __FILE__, __LINE__); \
    return; } } while (0)

namespace {

constexpr int WORLD = 8;
constexpr int GM = 4096, GK = 4096, GN = 16384;
constexpr int KS = GK / WORLD;
constexpr int NS = GN / WORLD;
constexpr int MB = GM / WORLD;
constexpr int BM = 128, BN = 256, BK = 128;
constexpr int NKB = GK / BK;
constexpr int A_STAGE = BM * BK;
constexpr int B_STAGE = BN * BK;
constexpr int STAGE = A_STAGE + B_STAGE;
constexpr int NSTAGE = 3;
constexpr int NTHREADS = 512;
constexpr int NGRID = 256;
constexpr int NPUSH = 8;
constexpr unsigned NTOK = 2;
constexpr int REL_AT = 4;

constexpr size_t WS_CALL = 0;
constexpr size_t WS_EXIT = 128;
constexpr size_t WS_TMO = 256;
constexpr size_t WS_QTICK = 1024;
constexpr size_t WS_QDONE = 2048;
constexpr size_t WS_PDONE = 3072;
constexpr size_t WS_XFLAG = 4096;
constexpr size_t WS_WFLAG = 8192;
constexpr size_t WS_PQ = 12288;
constexpr size_t WS_OFLAG = 16384;
constexpr size_t WS_STAGE = 1u << 20;
constexpr size_t XS_BYTES = (size_t)GM * GK;
constexpr size_t WT_BYTES = (size_t)GK * NS;
constexpr size_t PAR_BYTES = XS_BYTES + WT_BYTES;
constexpr unsigned PK_SLOT = 17408;
constexpr size_t WS_PK = WS_STAGE + 2 * PAR_BYTES;
constexpr size_t PK_BYTES = (size_t)WORLD * 32 * 8 * PK_SLOT;
constexpr size_t WS_NEED = WS_PK + PK_BYTES;
constexpr size_t OUT_BYTES = (size_t)MB * GN * 4;

constexpr unsigned long long DEADLINE_TICKS = 50000000ull;

typedef int v4i __attribute__((ext_vector_type(4)));
typedef unsigned v4u __attribute__((ext_vector_type(4)));
typedef int v8i __attribute__((ext_vector_type(8)));
typedef float v4f __attribute__((ext_vector_type(4)));

struct Params {
  const uint8_t* x;
  const uint8_t* w;
  const float* sx;
  const float* sw;
  float* out[WORLD];
  uint8_t* ws[WORLD];
  int rank;
};

#define AUX_SYS 17

__device__ __forceinline__ __amdgpu_buffer_rsrc_t make_rsrc(void* p, unsigned bytes) {
  return __builtin_amdgcn_make_buffer_rsrc(p, 0, bytes, 0x00020000);
}
#define bufld16(rs, l, voff, soff, aux) \
  __builtin_amdgcn_raw_ptr_buffer_load_lds(rs, (__attribute__((address_space(3))) void*)(l), 16, voff, soff, 0, aux)
__device__ __forceinline__ unsigned ld_sys(const unsigned* p) {
  return __hip_atomic_load(p, __ATOMIC_RELAXED, __HIP_MEMORY_SCOPE_SYSTEM);
}
__device__ __forceinline__ unsigned ld_agent(const unsigned* p) {
  return __hip_atomic_load(p, __ATOMIC_RELAXED, __HIP_MEMORY_SCOPE_AGENT);
}
__device__ __forceinline__ void st_sys(unsigned* p, unsigned v) {
  __hip_atomic_store(p, v, __ATOMIC_RELAXED, __HIP_MEMORY_SCOPE_SYSTEM);
}
__device__ __forceinline__ void st_agent(unsigned* p, unsigned v) {
  __hip_atomic_store(p, v, __ATOMIC_RELAXED, __HIP_MEMORY_SCOPE_AGENT);
}

__device__ __forceinline__ void wait_flag(const unsigned* flag, unsigned call, unsigned* tmo) {
  const unsigned long long t0 = __builtin_amdgcn_s_memrealtime();
  for (;;) {
    unsigned v = call;
    if (flag) v = ld_sys(flag);
    if (__all(v >= call)) break;
    if (ld_agent(tmo) == call || __builtin_amdgcn_s_memrealtime() - t0 > DEADLINE_TICKS) {
      st_agent(tmo, call);
      break;
    }
    __builtin_amdgcn_s_sleep(2);
  }
  asm volatile("s_waitcnt vmcnt(0)" ::: "memory");
}

__global__ void __launch_bounds__(NTHREADS) gemm_rs_kernel(Params p) {
  __shared__ __attribute__((aligned(16))) uint8_t smem[NSTAGE * STAGE + 64];

  const int tid = threadIdx.x;
  const int lane = tid & 63;
  const int wave = __builtin_amdgcn_readfirstlane(tid >> 6);
  const unsigned g = blockIdx.x & 7u;
  const int idx_in = blockIdx.x >> 3;
  const int rank = p.rank;
  uint8_t* const myws = p.ws[rank];
  unsigned* const tmo = (unsigned*)(myws + WS_TMO);
  const unsigned call_v = ld_agent((const unsigned*)(myws + WS_CALL));

  const int c16 = lane & 15, gq = lane >> 4;

  if (idx_in < NPUSH) {
    uint8_t* const dws = p.ws[g];
    auto* const lds_next = (__attribute__((address_space(3))) unsigned*)smem;
    const int wn = wave & 3;
    const uint8_t* const wsrc = p.w + (size_t)((wave >> 2) * 128 + (idx_in & 1) * 256 + gq * 16) * GN + (size_t)g * NS + (idx_in >> 1) * BN + wn * 64 + c16 * 4;
    const unsigned wdst = (unsigned)XS_BYTES + (unsigned)(idx_in >> 1) * (NKB * B_STAGE) + (unsigned)(rank * 4 + (idx_in & 1) * 2 + (wave >> 2)) * B_STAGE +
                          (unsigned)wn * 4 * 2048 + lane * 16;
    const unsigned xsrc = (unsigned)(wave * 16 + c16) * KS + gq * 16;
    const unsigned xdst = (unsigned)(rank * 4) * A_STAGE + (unsigned)wave * 8192 + lane * 16;
    unsigned* const pq = (unsigned*)(myws + WS_PQ + g * 128);

    auto load_x = [&](v4u (&b)[8], int tmq) {
      const __amdgpu_buffer_rsrc_t rx = make_rsrc((void*)p.x, (unsigned)tmq < 32u ? (unsigned)(GM * KS) : 0u);
#pragma unroll
      for (int u = 0; u < 8; ++u) b[u] = __builtin_amdgcn_raw_buffer_load_b128(rx, xsrc, (tmq & 31) * (BM * KS) + u * 64, 0);
    };
    auto load_w = [&](unsigned (&b)[32], int half) {
#pragma unroll
      for (int t = 0; t < 32; ++t) b[t] = *(const unsigned*)(wsrc + half * (4 * BN) + (size_t)((t >> 4) * 64 + (t & 15)) * GN);
    };
    v4u xa[8], xb[8];
    unsigned wa[32], wb[32];
    load_x(xa, idx_in * 4);
    asm volatile("" ::: "memory");
    load_w(wa, 0);
    asm volatile("" ::: "memory");
    const unsigned call = __builtin_amdgcn_readfirstlane(call_v) + 1u;
    const __amdgpu_buffer_rsrc_t rs = make_rsrc(dws + WS_STAGE + (size_t)(call & 1u) * PAR_BYTES, (unsigned)PAR_BYTES);
    const unsigned pq_base = (call - 1u) * (24u + NPUSH);

    auto store_x = [&](const v4u (&b)[8], int tmq) {
      const unsigned dst = xdst + (unsigned)tmq * (NKB * A_STAGE);
#pragma unroll
      for (int u = 0; u < 8; ++u)
        __builtin_amdgcn_raw_buffer_store_b128(b[u], rs, dst + u * 1024, 0, AUX_SYS);
    };
    auto store_w = [&](const unsigned (&b)[32], int half) {
      const unsigned dst = wdst + (unsigned)half * (4u * NKB * B_STAGE);
#pragma unroll
      for (int qh = 0; qh < 2; ++qh) {
        v4u o[4];
#pragma unroll
        for (int u = 0; u < 4; ++u) {
          const unsigned* v = b + qh * 16 + 4 * u;
          const unsigned t0 = __builtin_amdgcn_perm(v[1], v[0], 0x05010400u);
          const unsigned t1 = __builtin_amdgcn_perm(v[1], v[0], 0x07030602u);
          const unsigned t2 = __builtin_amdgcn_perm(v[3], v[2], 0x05010400u);
          const unsigned t3 = __builtin_amdgcn_perm(v[3], v[2], 0x07030602u);
          o[0][u] = __builtin_amdgcn_perm(t2, t0, 0x05040100u);
          o[1][u] = __builtin_amdgcn_perm(t2, t0, 0x07060302u);
          o[2][u] = __builtin_amdgcn_perm(t3, t1, 0x05040100u);
          o[3][u] = __builtin_amdgcn_perm(t3, t1, 0x07060302u);
        }
#pragma unroll
        for (int j = 0; j < 4; ++j)
          __builtin_amdgcn_raw_buffer_store_b128(o[j], rs, dst + j * 2048 + qh * 1024, 0, AUX_SYS);
      }
    };
    auto xflag = [&](int tmq) { return (unsigned*)(dws + WS_XFLAG + (size_t)tmq * 128) + rank; };
    auto wflag = [&](int half) { return (unsigned*)(dws + WS_WFLAG + (size_t)(half * 8 + idx_in) * 128) + rank; };
#define PUSH_FENCE() asm volatile("" ::: "memory")
#define PUSH_PUBLISH(N, flagptr) do { \
      asm volatile("s_waitcnt vmcnt(" #N ")" ::: "memory"); \
      __builtin_amdgcn_s_barrier(); PUSH_FENCE(); \
      if (tid == 0) st_sys(flagptr, call); \
      PUSH_FENCE(); } while (0)
#define PUSH_DRAW() do { if (tid == 0) { \
      const unsigned d = __hip_atomic_fetch_add(pq, 1u, __ATOMIC_RELAXED, __HIP_MEMORY_SCOPE_AGENT) - pq_base; \
      *lds_next = d < 24u ? (d & 7u) * 4u + 1u + (d >> 3) : 32u; } } while (0)
    auto ticket_done = [&]() -> int {
      asm volatile("s_waitcnt lgkmcnt(0)" ::: "memory");
      __builtin_amdgcn_s_barrier();
      PUSH_FENCE();
      return __builtin_amdgcn_readfirstlane(*lds_next);
    };
    store_x(xa, idx_in * 4); PUSH_FENCE();
    load_w(wb, 1); PUSH_FENCE();
    PUSH_PUBLISH(32, xflag(idx_in * 4));
    PUSH_DRAW();
    store_w(wa, 0); PUSH_FENCE();
    int ta = ticket_done();
    load_x(xa, ta); PUSH_FENCE();
    PUSH_PUBLISH(8, wflag(0));
    if (ta < 32) PUSH_DRAW();
    store_w(wb, 1); PUSH_FENCE();
    int tb = 32;
    if (ta < 32) tb = ticket_done();
    else if (tid == 0) __hip_atomic_fetch_add((unsigned*)(myws + WS_PDONE + g * 128), 1u, __ATOMIC_RELAXED, __HIP_MEMORY_SCOPE_AGENT);
    load_x(xb, tb); PUSH_FENCE();
    PUSH_PUBLISH(8, wflag(1));
    for (;;) {
      if (ta >= 32) break;
      {
        const int cur = ta;
        if (tb < 32) PUSH_DRAW();
        store_x(xa, cur); PUSH_FENCE();
        if (tb < 32) ta = ticket_done();
        else {
          ta = 32;
          if (tid == 0) __hip_atomic_fetch_add((unsigned*)(myws + WS_PDONE + g * 128), 1u, __ATOMIC_RELAXED, __HIP_MEMORY_SCOPE_AGENT);
        }
        load_x(xa, ta); PUSH_FENCE();
        PUSH_PUBLISH(8, xflag(cur));
      }
      if (tb >= 32) break;
      {
        const int cur = tb;
        if (ta < 32) PUSH_DRAW();
        store_x(xb, cur); PUSH_FENCE();
        if (ta < 32) tb = ticket_done();
        else {
          tb = 32;
          if (tid == 0) __hip_atomic_fetch_add((unsigned*)(myws + WS_PDONE + g * 128), 1u, __ATOMIC_RELAXED, __HIP_MEMORY_SCOPE_AGENT);
        }
        load_x(xb, tb); PUSH_FENCE();
        PUSH_PUBLISH(8, xflag(cur));
      }
    }
  }
  const unsigned call = __builtin_amdgcn_readfirstlane(call_v) + 1u;
  const size_t stage_off = WS_STAGE + (size_t)(call & 1u) * PAR_BYTES;

  const int idx = (idx_in - NPUSH) & 31;
  const int mt = idx >> 3, nt = idx & 7;
  const int tm = (int)g * 4 + mt;
  auto* const lds_sel = (__attribute__((address_space(3))) int*)(smem + NSTAGE * STAGE);
  const unsigned* myflag = nullptr;
  if (lane < 8) myflag = (const unsigned*)(myws + WS_XFLAG + (size_t)tm * 128) + lane;
  else if (lane < 24) myflag = (const unsigned*)(myws + WS_WFLAG + (size_t)(nt * 2 + ((lane - 8) >> 3)) * 128) + (lane & 7);
  unsigned rem = 0xffu, ready = 0u;
  auto select = [&]() {
    const unsigned long long t0 = __builtin_amdgcn_s_memrealtime();
    while ((ready & rem) == 0u) {
      unsigned v = call;
      if (myflag) v = ld_sys(myflag);
      const unsigned m = (unsigned)__builtin_amdgcn_ballot_w64(v >= call);
      ready = m & (m >> 8) & (m >> 16) & 0xffu;
      if (ready & rem) break;
      if (ld_agent(tmo) == call || __builtin_amdgcn_s_memrealtime() - t0 > DEADLINE_TICKS) {
        if (lane == 0) st_agent(tmo, call);
        ready = 0xffu;
        break;
      }
      __builtin_amdgcn_s_sleep(4);
    }
    const unsigned cand = ready & rem;
    const int sel = (cand >> rank) & 1u ? rank : __builtin_ctz(cand);
    rem &= ~(1u << sel);
    if (lane == 0) *lds_sel = sel;
  };
  if (wave == 0) {
    select();
    __builtin_amdgcn_fence(__ATOMIC_ACQUIRE, "");
    asm volatile("s_waitcnt vmcnt(0) lgkmcnt(0)" ::: "memory");
  }
  __builtin_amdgcn_s_barrier();
  asm volatile("" ::: "memory");
  int src = __builtin_amdgcn_readfirstlane(*lds_sel);

  const int wm = wave >> 2, wn = wave & 3;
  v4f acc[4][4];
#pragma unroll
  for (int i = 0; i < 4; ++i)
#pragma unroll
    for (int j = 0; j < 4; ++j) acc[i][j] = v4f{0.f, 0.f, 0.f, 0.f};
  const float s = p.sx[0] * p.sw[0];
  unsigned ticket = 0;

  {
    uint8_t* const sbase = myws + stage_off;
    const unsigned va = (unsigned)tm * (NKB * A_STAGE) + wave * 8192 + lane * 16;
    const unsigned vb = (unsigned)XS_BYTES + (unsigned)nt * (NKB * B_STAGE) + wave * 4096 + lane * 16;
    auto issue1 = [&](unsigned nrec, int kb, int boff, int c) {
      const __amdgpu_buffer_rsrc_t rs = make_rsrc(sbase, nrec);
      if (c < 2) bufld16(rs, smem + boff + wave * 2048 + c * 1024, va, (kb >> 2) * (4 * A_STAGE) + (kb & 3) * 2048 + c * 1024, 0);
      else bufld16(rs, smem + boff + A_STAGE + wave * 4096 + (c - 2) * 1024, vb, kb * B_STAGE + (c - 2) * 1024, 0);
    };
#pragma unroll
    for (int c = 0; c < 6; ++c) issue1((unsigned)PAR_BYTES, src * 4, 0, c);
#pragma unroll
    for (int c = 0; c < 6; ++c) issue1((unsigned)PAR_BYTES, src * 4 + 1, STAGE, c);
    int boff = 0;
    const int a_rd = wm * 4 * 2048 + lane * 16;
    const int b_rd = A_STAGE + wn * 4 * 2048 + lane * 16;
    auto step = [&](unsigned nrec, int pkb) {
      int nb = boff + 2 * STAGE;
      if (nb >= NSTAGE * STAGE) nb -= NSTAGE * STAGE;
      const uint8_t* sa = smem + boff + a_rd;
      const uint8_t* sb = smem + boff + b_rd;
      v8i af[4], bf[4];
      auto rd = [&](const uint8_t* q) {
        const v4i lo = *(const v4i*)q, hi = *(const v4i*)(q + 1024);
        return __builtin_shufflevector(lo, hi, 0, 1, 2, 3, 4, 5, 6, 7);
      };
      af[0] = rd(sa);
#pragma unroll
      for (int j = 0; j < 4; ++j) bf[j] = rd(sb + j * 2048);
#pragma unroll
      for (int i = 1; i < 4; ++i) af[i] = rd(sa + i * 2048);
      __builtin_amdgcn_sched_barrier(0);
#pragma unroll
      for (int i = 0; i < 4; ++i) {
#pragma unroll
        for (int j = 0; j < 4; ++j) {
          acc[i][j] = __builtin_amdgcn_mfma_scale_f32_16x16x128_f8f6f4(af[i], bf[j], acc[i][j], 0  , 1  , 0, 0, 0, 0);
          if ((j & 1) && i < 3) {
            __builtin_amdgcn_sched_barrier(0);
            issue1(nrec, pkb, nb, i * 2 + (j >> 1));
            __builtin_amdgcn_sched_barrier(0);
          }
        }
      }
      __builtin_amdgcn_s_waitcnt(0xC07F);
      boff += STAGE;
      if (boff >= NSTAGE * STAGE) boff = 0;
    };
#define GEMM_SYNC() do { asm volatile("s_waitcnt vmcnt(6)" ::: "memory"); __builtin_amdgcn_s_barrier(); asm volatile("" ::: "memory"); } while (0)
    __builtin_amdgcn_s_waitcnt(0xC07F);
    for (int slot = 0; slot < WORLD; ++slot) {
      const bool more = slot + 1 < WORLD;
      const unsigned nrec = more ? (unsigned)PAR_BYTES : 0u;
      GEMM_SYNC();
      step((unsigned)PAR_BYTES, src * 4 + 2);
      GEMM_SYNC();
      step((unsigned)PAR_BYTES, src * 4 + 3);
      if (more && wave == 0) {
        select();
        if (slot == WORLD - 2) {
          if (lane == 0 && g != (unsigned)rank)
            ticket = __hip_atomic_fetch_add((unsigned*)(myws + WS_QTICK + g * 128), 1u, __ATOMIC_RELAXED, __HIP_MEMORY_SCOPE_AGENT);
        }
        asm volatile("s_waitcnt lgkmcnt(0)" ::: "memory");
      }
      GEMM_SYNC();
      const int nsrc = more ? __builtin_amdgcn_readfirstlane(*lds_sel) : 0;
      __builtin_amdgcn_s_waitcnt(0xC07F);
      step(nrec, nsrc * 4);
      GEMM_SYNC();
      step(nrec, nsrc * 4 + 1);
      src = nsrc;
    }
  }
  const unsigned tmo_v = ld_agent(tmo);
  asm volatile("s_waitcnt vmcnt(1)" ::: "memory");

  const __amdgpu_buffer_rsrc_t ro = make_rsrc(p.out[rank], (unsigned)OUT_BYTES);
  const unsigned row_off = (unsigned)(mt * BM + wm * 64 + gq * 4) * GN * 4u;
  const unsigned col_off = (unsigned)(nt * BN + wn * 64 + c16 * 4) * 4u;

  if (g == (unsigned)rank) {
    const bool bad = tmo_v == call;
#pragma unroll
    for (int i = 0; i < 4; ++i)
#pragma unroll
      for (int r = 0; r < 4; ++r) {
        v4f v;
#pragma unroll
        for (int j = 0; j < 4; ++j) {
          const float y = acc[i][j][r] * s;
          v[j] = (y <= 0.f) ? 0.f : y;
          if (bad) v[j] = __builtin_nanf("");
        }
        __builtin_amdgcn_raw_buffer_store_b128(__builtin_bit_cast(v4u, v), ro,
                                               row_off + col_off + (unsigned)(rank * NS * 4) + (unsigned)(i * 16 + r) * (GN * 4), 0, AUX_SYS);
      }
  } else {
    asm volatile("s_waitcnt lgkmcnt(0)" ::: "memory");
    __builtin_amdgcn_s_barrier();
    asm volatile("" ::: "memory");
    uint8_t* const pk = smem + wave * PK_SLOT;
    const unsigned pk_lds = (unsigned)(unsigned long)((__attribute__((address_space(3))) uint8_t*)smem) + wave * PK_SLOT;
    unsigned nxt = pk_lds + 512u;
    unsigned bits[2] = {0u, 0u};
#pragma unroll
    for (int i = 0; i < 4; ++i)
#pragma unroll
      for (int r = 0; r < 4; ++r) {
        float y0, y1, y2, y3;
        unsigned t0, t1, t2, t3, n, b1, b2, b3;
        asm volatile(
            "v_mul_f32 %[y0], %[s], %[a0]\n\t"
            "v_mul_f32 %[y1], %[s], %[a1]\n\t"
            "v_mul_f32 %[y2], %[s], %[a2]\n\t"
            "v_mul_f32 %[y3], %[s], %[a3]\n\t"
            "v_cmp_nge_f32_e64 s[84:85], 0, %[y0]\n\t"
            "v_cmp_nge_f32_e64 s[86:87], 0, %[y1]\n\t"
            "v_cmp_nge_f32_e64 s[88:89], 0, %[y2]\n\t"
            "v_cmp_nge_f32_e64 s[90:91], 0, %[y3]\n\t"
            "s_bcnt1_i32_b64 %[n], s[84:85]\n\t"
            "s_lshl2_add_u32 %[b1], %[n], %[nx]\n\t"
            "s_bcnt1_i32_b64 %[n], s[86:87]\n\t"
            "s_lshl2_add_u32 %[b2], %[n], %[b1]\n\t"
            "s_bcnt1_i32_b64 %[n], s[88:89]\n\t"
            "s_lshl2_add_u32 %[b3], %[n], %[b2]\n\t"
            "v_mbcnt_lo_u32_b32 %[t0], s84, 0\n\t"
            "v_mbcnt_lo_u32_b32 %[t1], s86, 0\n\t"
            "v_mbcnt_lo_u32_b32 %[t2], s88, 0\n\t"
            "v_mbcnt_lo_u32_b32 %[t3], s90, 0\n\t"
            "v_mbcnt_hi_u32_b32 %[t0], s85, %[t0]\n\t"
            "v_mbcnt_hi_u32_b32 %[t1], s87, %[t1]\n\t"
            "v_mbcnt_hi_u32_b32 %[t2], s89, %[t2]\n\t"
            "v_mbcnt_hi_u32_b32 %[t3], s91, %[t3]\n\t"
            "v_lshl_add_u32 %[t0], %[t0], 2, %[nx]\n\t"
            "v_lshl_add_u32 %[t1], %[t1], 2, %[b1]\n\t"
            "v_lshl_add_u32 %[t2], %[t2], 2, %[b2]\n\t"
            "v_lshl_add_u32 %[t3], %[t3], 2, %[b3]\n\t"
            "s_bcnt1_i32_b64 %[n], s[90:91]\n\t"
            "s_lshl2_add_u32 %[nx], %[n], %[b3]\n\t"
            "v_addc_co_u32_e64 %[bt], vcc, %[bt], %[bt], s[84:85]\n\t"
            "v_addc_co_u32_e64 %[bt], vcc, %[bt], %[bt], s[86:87]\n\t"
            "v_addc_co_u32_e64 %[bt], vcc, %[bt], %[bt], s[88:89]\n\t"
            "v_addc_co_u32_e64 %[bt], vcc, %[bt], %[bt], s[90:91]\n\t"
            "s_mov_b64 exec, s[84:85]\n\t"
            "ds_write_b32 %[t0], %[y0]\n\t"
            "s_mov_b64 exec, s[86:87]\n\t"
            "ds_write_b32 %[t1], %[y1]\n\t"
            "s_mov_b64 exec, s[88:89]\n\t"
            "ds_write_b32 %[t2], %[y2]\n\t"
            "s_mov_b64 exec, s[90:91]\n\t"
            "ds_write_b32 %[t3], %[y3]\n\t"
            "s_mov_b64 exec, -1"
            : [y0] "=&v"(y0), [y1] "=&v"(y1), [y2] "=&v"(y2), [y3] "=&v"(y3), [t0] "=&v"(t0), [t1] "=&v"(t1), [t2] "=&v"(t2),
              [t3] "=&v"(t3), [n] "=&s"(n), [b1] "=&s"(b1), [b2] "=&s"(b2), [b3] "=&s"(b3), [nx] "+s"(nxt), [bt] "+v"(bits[i >> 1])
            : [s] "s"(s), [a0] "v"(acc[i][0][r]), [a1] "v"(acc[i][1][r]), [a2] "v"(acc[i][2][r]), [a3] "v"(acc[i][3][r])
            : "vcc", "scc", "s84", "s85", "s86", "s87", "s88", "s89", "s90", "s91", "memory");
      }
    unsigned base = nxt - pk_lds;
    if (tmo_v == call) {
      bits[0] = bits[1] = 0xffffffffu;
      base = 512u + 16384u;
      for (int o = 0; o < 64; ++o) *(float*)(pk + 512 + o * 256 + lane * 4) = __builtin_nanf("");
    }
    *(uint2*)(pk + lane * 8) = make_uint2(bits[0], bits[1]);
    const unsigned bytes = (base + 63u) & ~63u;

    if (tid == 0) {
      const unsigned t = ticket;
      const unsigned* pd = (const unsigned*)(myws + WS_PDONE + g * 128);
      const unsigned* qd = (const unsigned*)(myws + WS_QDONE + g * 128);
      const unsigned long long t0 = __builtin_amdgcn_s_memrealtime();
      while (ld_agent(pd) < call * NPUSH || (int)(ld_agent(qd) + NTOK - t) <= 0) {
        if (ld_agent(tmo) == call || __builtin_amdgcn_s_memrealtime() - t0 > DEADLINE_TICKS) {
          st_agent(tmo, call);
          break;
        }
        __builtin_amdgcn_s_sleep(8);
      }
    }
    asm volatile("s_waitcnt lgkmcnt(0)" ::: "memory");
    __builtin_amdgcn_s_barrier();
    asm volatile("" ::: "memory");
    {
      const __amdgpu_buffer_rsrc_t rp = make_rsrc(p.ws[g] + WS_PK, (unsigned)PK_BYTES);
      const unsigned slot = (unsigned)((rank * 32 + idx_in) * 8 + wave) * PK_SLOT;
#pragma unroll
      for (int o = 0; o < 17; ++o) {
        const unsigned off = o * 1024u + lane * 16u;
        if (o * 1024u < bytes) {
          const v4u v = *(const v4u*)(pk + off);
          if (off < bytes) __builtin_amdgcn_raw_buffer_store_b128(v, rp, slot + off, 0, AUX_SYS);
        }
        if (o == REL_AT) {
          asm volatile("" ::: "memory");
          __builtin_amdgcn_s_barrier();
          if (tid == 0)
            __hip_atomic_fetch_add((unsigned*)(myws + WS_QDONE + g * 128), 1u, __ATOMIC_RELAXED, __HIP_MEMORY_SCOPE_AGENT);
          asm volatile("" ::: "memory");
        }
      }
    }
    asm volatile("s_waitcnt vmcnt(0)" ::: "memory");
    __builtin_amdgcn_s_barrier();
    if (tid == 0) st_sys((unsigned*)(p.ws[g] + WS_OFLAG + ((size_t)rank * 32 + idx_in) * 128), call);

    if (wave == 0) {
      const unsigned* f = lane == 0 ? (const unsigned*)(myws + WS_OFLAG + ((size_t)g * 32 + idx_in) * 128) : nullptr;
      wait_flag(f, call, tmo);
    }
    asm volatile("" ::: "memory");
    __builtin_amdgcn_s_barrier();
    asm volatile("" ::: "memory");
    const bool bad2 = ld_agent(tmo) == call;
    {
      const __amdgpu_buffer_rsrc_t rq = make_rsrc(myws + WS_PK, (unsigned)PK_BYTES);
      const unsigned src = (unsigned)((g * 32 + idx_in) * 8 + wave) * PK_SLOT;
#pragma unroll
      for (int o = 0; o < 17; ++o) bufld16(rq, pk + o * 1024, lane * 16, src + o * 1024, AUX_SYS);
      asm volatile("s_waitcnt vmcnt(0)" ::: "memory");
      const uint2 mm = *(const uint2*)(pk + lane * 8);
      *(float*)(pk + 17152u + lane * 4u) = 0.f;
      const unsigned zero_at = pk_lds + 17152u + lane * 4u;
      unsigned rb = pk_lds + 512u;
      unsigned mb[2] = {mm.x, mm.y};
#define DEC4(v0, v1, v2, v3) \
            "v_add_co_u32_e64 %[mb], s[84:85], %[mb], %[mb]\n\t" \
            "v_add_co_u32_e64 %[mb], s[86:87], %[mb], %[mb]\n\t" \
            "v_add_co_u32_e64 %[mb], s[88:89], %[mb], %[mb]\n\t" \
            "v_add_co_u32_e64 %[mb], s[90:91], %[mb], %[mb]\n\t" \
            "s_bcnt1_i32_b64 %[n], s[84:85]\n\t" \
            "s_lshl2_add_u32 %[b1], %[n], %[rb]\n\t" \
            "s_bcnt1_i32_b64 %[n], s[86:87]\n\t" \
            "s_lshl2_add_u32 %[b2], %[n], %[b1]\n\t" \
            "s_bcnt1_i32_b64 %[n], s[88:89]\n\t" \
            "s_lshl2_add_u32 %[b3], %[n], %[b2]\n\t" \
            "v_mbcnt_lo_u32_b32 " v0 ", s84, 0\n\t" \
            "v_mbcnt_lo_u32_b32 " v1 ", s86, 0\n\t" \
            "v_mbcnt_lo_u32_b32 " v2 ", s88, 0\n\t" \
            "v_mbcnt_lo_u32_b32 " v3 ", s90, 0\n\t" \
            "v_mbcnt_hi_u32_b32 " v0 ", s85, " v0 "\n\t" \
            "v_mbcnt_hi_u32_b32 " v1 ", s87, " v1 "\n\t" \
            "v_mbcnt_hi_u32_b32 " v2 ", s89, " v2 "\n\t" \
            "v_mbcnt_hi_u32_b32 " v3 ", s91, " v3 "\n\t" \
            "v_lshl_add_u32 " v0 ", " v0 ", 2, %[rb]\n\t" \
            "v_lshl_add_u32 " v1 ", " v1 ", 2, %[b1]\n\t" \
            "v_lshl_add_u32 " v2 ", " v2 ", 2, %[b2]\n\t" \
            "v_lshl_add_u32 " v3 ", " v3 ", 2, %[b3]\n\t" \
            "s_bcnt1_i32_b64 %[n], s[90:91]\n\t" \
            "s_lshl2_add_u32 %[rb], %[n], %[b3]\n\t" \
            "v_cndmask_b32_e64 " v0 ", %[z], " v0 ", s[84:85]\n\t" \
            "v_cndmask_b32_e64 " v1 ", %[z], " v1 ", s[86:87]\n\t" \
            "v_cndmask_b32_e64 " v2 ", %[z], " v2 ", s[88:89]\n\t" \
            "v_cndmask_b32_e64 " v3 ", %[z], " v3 ", s[90:91]\n\t" \
            "ds_read_b32 " v0 ", " v0 "\n\t" \
            "ds_read_b32 " v1 ", " v1 "\n\t" \
            "ds_read_b32 " v2 ", " v2 "\n\t" \
            "ds_read_b32 " v3 ", " v3 "\n\t"
#pragma unroll
      for (int i = 0; i < 4; ++i) {
        unsigned e[16], n, b1, b2, b3;
        asm volatile(
            DEC4("%[e0]", "%[e1]", "%[e2]", "%[e3]")
            DEC4("%[e4]", "%[e5]", "%[e6]", "%[e7]")
            DEC4("%[e8]", "%[e9]", "%[e10]", "%[e11]")
            DEC4("%[e12]", "%[e13]", "%[e14]", "%[e15]")
            "s_waitcnt lgkmcnt(0)"
            : [e0] "=&v"(e[0]), [e1] "=&v"(e[1]), [e2] "=&v"(e[2]), [e3] "=&v"(e[3]), [e4] "=&v"(e[4]), [e5] "=&v"(e[5]),
              [e6] "=&v"(e[6]), [e7] "=&v"(e[7]), [e8] "=&v"(e[8]), [e9] "=&v"(e[9]), [e10] "=&v"(e[10]), [e11] "=&v"(e[11]),
              [e12] "=&v"(e[12]), [e13] "=&v"(e[13]), [e14] "=&v"(e[14]), [e15] "=&v"(e[15]),
              [n] "=&s"(n), [b1] "=&s"(b1), [b2] "=&s"(b2), [b3] "=&s"(b3), [rb] "+s"(rb), [mb] "+v"(mb[i >> 1])
            : [z] "v"(zero_at)
            : "scc", "s84", "s85", "s86", "s87", "s88", "s89", "s90", "s91", "memory");
#pragma unroll
        for (int r = 0; r < 4; ++r) {
          v4u v = {e[r * 4], e[r * 4 + 1], e[r * 4 + 2], e[r * 4 + 3]};
          if (bad2) v = v4u{0x7fc00000u, 0x7fc00000u, 0x7fc00000u, 0x7fc00000u};
          __builtin_amdgcn_raw_buffer_store_b128(v, ro, row_off + col_off + g * (NS * 4) + (unsigned)(i * 16 + r) * (GN * 4), 0, AUX_SYS);
        }
      }
    }
  }
  if (tid == 0) {
    const unsigned t = __hip_atomic_fetch_add((unsigned*)(myws + WS_EXIT), 1u, __ATOMIC_RELAXED, __HIP_MEMORY_SCOPE_AGENT);
    if ((t + 1u) % NGRID == 0u) st_agent((unsigned*)(myws + WS_CALL), call);
  }
}

}

extern "C" void kernel_launch(int rank, int nranks, ncclComm_t comm,
                              hipStream_t stream,
                              void* const* d_in, const int* in_sizes, int n_in,
                              void* d_out, int out_size,
                              const int* in_dtypes, int out_dtype,
                              void* const* d_out_all,
                              void* const* d_ws_all, size_t ws_bytes) {
  (void)comm; (void)in_dtypes; (void)out_dtype; (void)d_out;
  if (nranks != WORLD || n_in != 4 || ws_bytes < WS_NEED || (unsigned)rank >= (unsigned)WORLD ||
      in_sizes[0] != GM * KS || in_sizes[1] != KS * GN || out_size != MB * GN) {
    fprintf(stderr, "kernel_launch: unexpected problem shape\n");
    return;
  }
  Params p;
  p.x = (const uint8_t*)d_in[0];
  p.w = (const uint8_t*)d_in[1];
  p.sx = (const float*)d_in[2];
  p.sw = (const float*)d_in[3];
  for (int r = 0; r < WORLD; ++r) {
    p.out[r] = (float*)d_out_all[r];
    p.ws[r] = (uint8_t*)d_ws_all[r];
  }
  p.rank = rank;
  gemm_rs_kernel<<<dim3(NGRID), dim3(NTHREADS), 0, stream>>>(p);
  HIP_CHECK(hipGetLastError());
}
```

```cpp
#include <hip/hip_runtime.h>
#include <rccl/rccl.h>
#include <cstdio>
#include <cstdint>

#define HIP_CHECK(expr) do { hipError_t _e = (expr); if (_e != hipSuccess) { \
    fprintf(stderr, "HIP error %s at %s:%d\n", hipGetErrorString(_e), __FILE__, __LINE__); \
    return; } } while (0)

namespace {

constexpr int WORLD = 8;
constexpr int GM = 4096, GK = 4096, GN = 16384;
constexpr int KS = GK / WORLD;
constexpr int NS = GN / WORLD;
constexpr int MB = GM / WORLD;
constexpr int BM = 128, BN = 256, BK = 128;
constexpr int NKB = GK / BK;
constexpr int A_STAGE = BM * BK;
constexpr int B_STAGE = BN * BK;
constexpr int STAGE = A_STAGE + B_STAGE;
constexpr int NSTAGE = 3;
constexpr int NTHREADS = 512;
constexpr int NGRID = 256;
constexpr int NPUSH = 8;
constexpr unsigned NTOK = 2;
constexpr int REL_AT = 4;

constexpr size_t WS_CALL = 0;
constexpr size_t WS_EXIT = 128;
constexpr size_t WS_TMO = 256;
constexpr size_t WS_QTICK = 1024;
constexpr size_t WS_QDONE = 2048;
constexpr size_t WS_XFLAG = 4096;
constexpr size_t WS_WFLAG = 8192;
constexpr size_t WS_PQ = 12288;
constexpr size_t WS_OFLAG = 16384;
constexpr size_t WS_STAGE = 1u << 20;
constexpr size_t XS_BYTES = (size_t)GM * GK;
constexpr size_t WT_BYTES = (size_t)GK * NS;
constexpr size_t PAR_BYTES = XS_BYTES + WT_BYTES;
constexpr unsigned PK_SLOT = 17408;
constexpr size_t WS_PK = WS_STAGE + 2 * PAR_BYTES;
constexpr size_t PK_BYTES = (size_t)WORLD * 32 * 8 * PK_SLOT;
constexpr size_t WS_NEED = WS_PK + PK_BYTES;
constexpr size_t OUT_BYTES = (size_t)MB * GN * 4;

constexpr unsigned long long DEADLINE_TICKS = 50000000ull;

typedef int v4i __attribute__((ext_vector_type(4)));
typedef unsigned v4u __attribute__((ext_vector_type(4)));
typedef int v8i __attribute__((ext_vector_type(8)));
typedef float v4f __attribute__((ext_vector_type(4)));

struct Params {
  const uint8_t* x;
  const uint8_t* w;
  const float* sx;
  const float* sw;
  float* out[WORLD];
  uint8_t* ws[WORLD];
  int rank;
};

#define AUX_SYS 17

__device__ __forceinline__ __amdgpu_buffer_rsrc_t make_rsrc(void* p, unsigned bytes) {
  return __builtin_amdgcn_make_buffer_rsrc(p, 0, bytes, 0x00020000);
}
#define bufld16(rs, l, voff, soff, aux) \
  __builtin_amdgcn_raw_ptr_buffer_load_lds(rs, (__attribute__((address_space(3))) void*)(l), 16, voff, soff, 0, aux)
__device__ __forceinline__ unsigned ld_sys(const unsigned* p) {
  return __hip_atomic_load(p, __ATOMIC_RELAXED, __HIP_MEMORY_SCOPE_SYSTEM);
}
__device__ __forceinline__ unsigned ld_agent(const unsigned* p) {
  return __hip_atomic_load(p, __ATOMIC_RELAXED, __HIP_MEMORY_SCOPE_AGENT);
}
__device__ __forceinline__ void st_sys(unsigned* p, unsigned v) {
  __hip_atomic_store(p, v, __ATOMIC_RELAXED, __HIP_MEMORY_SCOPE_SYSTEM);
}
__device__ __forceinline__ void st_agent(unsigned* p, unsigned v) {
  __hip_atomic_store(p, v, __ATOMIC_RELAXED, __HIP_MEMORY_SCOPE_AGENT);
}

__device__ __forceinline__ void wait_flag(const unsigned* flag, unsigned call, unsigned* tmo) {
  const unsigned long long t0 = __builtin_amdgcn_s_memrealtime();
  for (unsigned it = 1;; ++it) {
    unsigned v = call;
    if (flag) v = ld_sys(flag);
    if (__all(v >= call)) break;
    if ((it & 7u) == 0u && (ld_agent(tmo) == call || __builtin_amdgcn_s_memrealtime() - t0 > DEADLINE_TICKS)) {
      st_agent(tmo, call);
      break;
    }
    __builtin_amdgcn_s_sleep(2);
  }
  asm volatile("s_waitcnt vmcnt(0)" ::: "memory");
}

__global__ void __launch_bounds__(NTHREADS) gemm_rs_kernel(Params p) {
  __shared__ __attribute__((aligned(16))) uint8_t smem[NSTAGE * STAGE + 64];

  const int tid = threadIdx.x;
  const int lane = tid & 63;
  const int wave = __builtin_amdgcn_readfirstlane(tid >> 6);
  const unsigned g = blockIdx.x & 7u;
  const int idx_in = blockIdx.x >> 3;
  const int rank = p.rank;
  uint8_t* const myws = p.ws[rank];
  unsigned* const tmo = (unsigned*)(myws + WS_TMO);
  const unsigned call_v = ld_agent((const unsigned*)(myws + WS_CALL));

  const int c16 = lane & 15, gq = lane >> 4;
  const bool remote = g != (unsigned)rank;

  unsigned* const qdone = (unsigned*)(myws + WS_QDONE + g * 128);
  auto link_wait = [&](unsigned ticket, unsigned call) {
    const unsigned long long t0 = __builtin_amdgcn_s_memrealtime();
    for (unsigned it = 1; (int)(ld_agent(qdone) + NTOK - ticket) <= 0; ++it) {
      if ((it & 15u) == 0u && (ld_agent(tmo) == call || __builtin_amdgcn_s_memrealtime() - t0 > DEADLINE_TICKS)) {
        st_agent(tmo, call);
        break;
      }
      __builtin_amdgcn_s_sleep(2);
    }
  };
  auto link_release = [&]() {
    asm volatile("" ::: "memory");
    __builtin_amdgcn_s_barrier();
    if (tid == 0 && remote) __hip_atomic_fetch_add(qdone, 1u, __ATOMIC_RELAXED, __HIP_MEMORY_SCOPE_AGENT);
    asm volatile("" ::: "memory");
  };

  if (idx_in < NPUSH) {
    uint8_t* const dws = p.ws[g];
    auto* const lds_next = (__attribute__((address_space(3))) unsigned*)smem;
    const int wn = wave & 3;
    const uint8_t* const wsrc = p.w + (size_t)((wave >> 2) * 128 + (idx_in & 1) * 256 + gq * 16) * GN + (size_t)g * NS + (idx_in >> 1) * BN + wn * 64 + c16 * 4;
    const unsigned wdst = (unsigned)XS_BYTES + (unsigned)(idx_in >> 1) * (NKB * B_STAGE) + (unsigned)(rank * 4 + (idx_in & 1) * 2 + (wave >> 2)) * B_STAGE +
                          (unsigned)wn * 4 * 2048 + lane * 16;
    const unsigned xsrc = (unsigned)(wave * 16 + c16) * KS + gq * 16;
    const unsigned xdst = (unsigned)(rank * 4) * A_STAGE + (unsigned)wave * 8192 + lane * 16;
    unsigned* const pq = (unsigned*)(myws + WS_PQ + g * 128);

    auto load_x = [&](v4u (&b)[8], int tmq) {
      const __amdgpu_buffer_rsrc_t rx = make_rsrc((void*)p.x, (unsigned)tmq < 32u ? (unsigned)(GM * KS) : 0u);
#pragma unroll
      for (int u = 0; u < 8; ++u) b[u] = __builtin_amdgcn_raw_buffer_load_b128(rx, xsrc, (tmq & 31) * (BM * KS) + u * 64, 0);
    };
    auto load_w = [&](unsigned (&b)[32], int half) {
#pragma unroll
      for (int t = 0; t < 32; ++t) b[t] = *(const unsigned*)(wsrc + half * (4 * BN) + (size_t)((t >> 4) * 64 + (t & 15)) * GN);
    };
    v4u xa[8], xb[8];
    unsigned wa[32], wb[32];
    load_x(xa, idx_in * 4);
    asm volatile("" ::: "memory");
    const unsigned call = __builtin_amdgcn_readfirstlane(call_v) + 1u;
    const __amdgpu_buffer_rsrc_t rs = make_rsrc(dws + WS_STAGE + (size_t)(call & 1u) * PAR_BYTES, (unsigned)PAR_BYTES);
    const unsigned pq_base = (call - 1u) * (24u + NPUSH);

    auto store_x = [&](const v4u (&b)[8], int tmq) {
      const unsigned dst = xdst + (unsigned)tmq * (NKB * A_STAGE);
#pragma unroll
      for (int u = 0; u < 8; ++u) {
        __builtin_amdgcn_raw_buffer_store_b128(b[u], rs, dst + u * 1024, 0, AUX_SYS);
        if (u == REL_AT) link_release();
      }
    };
    auto store_w = [&](const unsigned (&b)[32], int half) {
      const unsigned dst = wdst + (unsigned)half * (4u * NKB * B_STAGE);
#pragma unroll
      for (int qh = 0; qh < 2; ++qh) {
        v4u o[4];
#pragma unroll
        for (int u = 0; u < 4; ++u) {
          const unsigned* v = b + qh * 16 + 4 * u;
          const unsigned t0 = __builtin_amdgcn_perm(v[1], v[0], 0x05010400u);
          const unsigned t1 = __builtin_amdgcn_perm(v[1], v[0], 0x07030602u);
          const unsigned t2 = __builtin_amdgcn_perm(v[3], v[2], 0x05010400u);
          const unsigned t3 = __builtin_amdgcn_perm(v[3], v[2], 0x07030602u);
          o[0][u] = __builtin_amdgcn_perm(t2, t0, 0x05040100u);
          o[1][u] = __builtin_amdgcn_perm(t2, t0, 0x07060302u);
          o[2][u] = __builtin_amdgcn_perm(t3, t1, 0x05040100u);
          o[3][u] = __builtin_amdgcn_perm(t3, t1, 0x07060302u);
        }
#pragma unroll
        for (int j = 0; j < 4; ++j) {
          __builtin_amdgcn_raw_buffer_store_b128(o[j], rs, dst + j * 2048 + qh * 1024, 0, AUX_SYS);
          if (qh * 4 + j == REL_AT) link_release();
        }
      }
    };
    auto xflag = [&](int tmq) { return (unsigned*)(dws + WS_XFLAG + (size_t)tmq * 128) + rank; };
    auto wflag = [&](int half) { return (unsigned*)(dws + WS_WFLAG + (size_t)(half * 8 + idx_in) * 128) + rank; };
#define PUSH_FENCE() asm volatile("" ::: "memory")
#define PUSH_PUBLISH(N, flagptr) do { \
      asm volatile("s_waitcnt vmcnt(" #N ")" ::: "memory"); \
      __builtin_amdgcn_s_barrier(); PUSH_FENCE(); \
      if (tid == 0) st_sys(flagptr, call); \
      PUSH_FENCE(); } while (0)
#define PUSH_DRAW() do { if (tid == 0) { \
      const unsigned d = __hip_atomic_fetch_add(pq, 1u, __ATOMIC_RELAXED, __HIP_MEMORY_SCOPE_AGENT) - pq_base; \
      *lds_next = d < 24u ? (d & 7u) * 4u + 1u + (d >> 3) : 32u; } } while (0)
    const unsigned tk_base = (call - 1u) * 80u;
    auto xticket = [&](int tmq) { return tk_base + ((tmq & 3) == 0 ? (unsigned)(tmq >> 2) : 16u + (unsigned)(tmq & 3) * 8u + (unsigned)(tmq >> 2)); };
    auto go = [&](unsigned ticket) {
      if (tid == 0 && remote) link_wait(ticket, call);
      asm volatile("s_waitcnt lgkmcnt(0)" ::: "memory");
      __builtin_amdgcn_s_barrier();
      PUSH_FENCE();
    };
    if (idx_in >= (int)NTOK) go(xticket(idx_in * 4));
    store_x(xa, idx_in * 4); PUSH_FENCE();
    load_w(wa, 0); PUSH_FENCE();
    PUSH_PUBLISH(32, xflag(idx_in * 4));
    PUSH_DRAW();
    go(tk_base + 8u + idx_in);
    int ta = __builtin_amdgcn_readfirstlane(*lds_next);
    store_w(wa, 0); PUSH_FENCE();
    load_w(wb, 1); PUSH_FENCE();
    load_x(xa, ta); PUSH_FENCE();
    PUSH_PUBLISH(40, wflag(0));
    if (ta < 32) PUSH_DRAW();
    go(tk_base + 16u + idx_in);
    int tb = ta < 32 ? __builtin_amdgcn_readfirstlane(*lds_next) : 32;
    store_w(wb, 1); PUSH_FENCE();
    load_x(xb, tb); PUSH_FENCE();
    PUSH_PUBLISH(8, wflag(1));
    for (;;) {
      if (ta >= 32) break;
      {
        const int cur = ta;
        if (tb < 32) PUSH_DRAW();
        go(xticket(cur));
        ta = tb < 32 ? __builtin_amdgcn_readfirstlane(*lds_next) : 32;
        store_x(xa, cur); PUSH_FENCE();
        load_x(xa, ta); PUSH_FENCE();
        PUSH_PUBLISH(8, xflag(cur));
      }
      if (tb >= 32) break;
      {
        const int cur = tb;
        if (ta < 32) PUSH_DRAW();
        go(xticket(cur));
        tb = ta < 32 ? __builtin_amdgcn_readfirstlane(*lds_next) : 32;
        store_x(xb, cur); PUSH_FENCE();
        load_x(xb, tb); PUSH_FENCE();
        PUSH_PUBLISH(8, xflag(cur));
      }
    }
  }
  const unsigned call = __builtin_amdgcn_readfirstlane(call_v) + 1u;
  const size_t stage_off = WS_STAGE + (size_t)(call & 1u) * PAR_BYTES;

  const int idx = (idx_in - NPUSH) & 31;
  const int mt = idx >> 3, nt = idx & 7;
  const int tm = (int)g * 4 + mt;
  auto* const lds_sel = (__attribute__((address_space(3))) int*)(smem + NSTAGE * STAGE);
  const unsigned* myflag = nullptr;
  if (lane < 8) myflag = (const unsigned*)(myws + WS_XFLAG + (size_t)tm * 128) + lane;
  else if (lane < 24) myflag = (const unsigned*)(myws + WS_WFLAG + (size_t)(nt * 2 + ((lane - 8) >> 3)) * 128) + (lane & 7);
  unsigned rem = 0xffu, ready = 0u;
  auto select = [&]() {
    const unsigned long long t0 = __builtin_amdgcn_s_memrealtime();
    while ((ready & rem) == 0u) {
      unsigned v = call;
      if (myflag) v = ld_sys(myflag);
      const unsigned m = (unsigned)__builtin_amdgcn_ballot_w64(v >= call);
      ready = m & (m >> 8) & (m >> 16) & 0xffu;
      if (ready & rem) break;
      if (ld_agent(tmo) == call || __builtin_amdgcn_s_memrealtime() - t0 > DEADLINE_TICKS) {
        if (lane == 0) st_agent(tmo, call);
        ready = 0xffu;
        break;
      }
      __builtin_amdgcn_s_sleep(4);
    }
    const unsigned cand = ready & rem;
    const int sel = (cand >> rank) & 1u ? rank : __builtin_ctz(cand);
    rem &= ~(1u << sel);
    if (lane == 0) *lds_sel = sel;
  };
  if (wave == 0) {
    select();
    __builtin_amdgcn_fence(__ATOMIC_ACQUIRE, "");
    asm volatile("s_waitcnt vmcnt(0) lgkmcnt(0)" ::: "memory");
  }
  __builtin_amdgcn_s_barrier();
  asm volatile("" ::: "memory");
  int src = __builtin_amdgcn_readfirstlane(*lds_sel);

  const int wm = wave >> 2, wn = wave & 3;
  v4f acc[4][4];
#pragma unroll
  for (int i = 0; i < 4; ++i)
#pragma unroll
    for (int j = 0; j < 4; ++j) acc[i][j] = v4f{0.f, 0.f, 0.f, 0.f};
  const float s = p.sx[0] * p.sw[0];
  unsigned ticket = 0;

  {
    uint8_t* const sbase = myws + stage_off;
    const unsigned va = (unsigned)tm * (NKB * A_STAGE) + wave * 8192 + lane * 16;
    const unsigned vb = (unsigned)XS_BYTES + (unsigned)nt * (NKB * B_STAGE) + wave * 4096 + lane * 16;
    auto issue1 = [&](unsigned nrec, int kb, int boff, int c) {
      const __amdgpu_buffer_rsrc_t rs = make_rsrc(sbase, nrec);
      if (c < 2) bufld16(rs, smem + boff + wave * 2048 + c * 1024, va, (kb >> 2) * (4 * A_STAGE) + (kb & 3) * 2048 + c * 1024, 0);
      else bufld16(rs, smem + boff + A_STAGE + wave * 4096 + (c - 2) * 1024, vb, kb * B_STAGE + (c - 2) * 1024, 0);
    };
#pragma unroll
    for (int c = 0; c < 6; ++c) issue1((unsigned)PAR_BYTES, src * 4, 0, c);
#pragma unroll
    for (int c = 0; c < 6; ++c) issue1((unsigned)PAR_BYTES, src * 4 + 1, STAGE, c);
    int boff = 0;
    const int a_rd = wm * 4 * 2048 + lane * 16;
    const int b_rd = A_STAGE + wn * 4 * 2048 + lane * 16;
    auto step = [&](unsigned nrec, int pkb) {
      int nb = boff + 2 * STAGE;
      if (nb >= NSTAGE * STAGE) nb -= NSTAGE * STAGE;
      const uint8_t* sa = smem + boff + a_rd;
      const uint8_t* sb = smem + boff + b_rd;
      v8i af[4], bf[4];
      auto rd = [&](const uint8_t* q) {
        const v4i lo = *(const v4i*)q, hi = *(const v4i*)(q + 1024);
        return __builtin_shufflevector(lo, hi, 0, 1, 2, 3, 4, 5, 6, 7);
      };
      af[0] = rd(sa);
#pragma unroll
      for (int j = 0; j < 4; ++j) bf[j] = rd(sb + j * 2048);
#pragma unroll
      for (int i = 1; i < 4; ++i) af[i] = rd(sa + i * 2048);
      __builtin_amdgcn_sched_barrier(0);
#pragma unroll
      for (int i = 0; i < 4; ++i) {
#pragma unroll
        for (int j = 0; j < 4; ++j) {
          acc[i][j] = __builtin_amdgcn_mfma_scale_f32_16x16x128_f8f6f4(af[i], bf[j], acc[i][j], 0  , 1  , 0, 0, 0, 0);
          if ((j & 1) && i < 3) {
            __builtin_amdgcn_sched_barrier(0);
            issue1(nrec, pkb, nb, i * 2 + (j >> 1));
            __builtin_amdgcn_sched_barrier(0);
          }
        }
      }
      __builtin_amdgcn_s_waitcnt(0xC07F);
      boff += STAGE;
      if (boff >= NSTAGE * STAGE) boff = 0;
    };
#define GEMM_SYNC() do { asm volatile("s_waitcnt vmcnt(6)" ::: "memory"); __builtin_amdgcn_s_barrier(); asm volatile("" ::: "memory"); } while (0)
    __builtin_amdgcn_s_waitcnt(0xC07F);
    for (int slot = 0; slot < WORLD; ++slot) {
      const bool more = slot + 1 < WORLD;
      const unsigned nrec = more ? (unsigned)PAR_BYTES : 0u;
      GEMM_SYNC();
      step((unsigned)PAR_BYTES, src * 4 + 2);
      GEMM_SYNC();
      step((unsigned)PAR_BYTES, src * 4 + 3);
      if (more && wave == 0) {
        select();
        if (slot == WORLD - 2) {
          if (lane == 0 && g != (unsigned)rank)
            ticket = __hip_atomic_fetch_add((unsigned*)(myws + WS_QTICK + g * 128), 1u, __ATOMIC_RELAXED, __HIP_MEMORY_SCOPE_AGENT);
        }
        asm volatile("s_waitcnt lgkmcnt(0)" ::: "memory");
      }
      GEMM_SYNC();
      const int nsrc = more ? __builtin_amdgcn_readfirstlane(*lds_sel) : 0;
      __builtin_amdgcn_s_waitcnt(0xC07F);
      step(nrec, nsrc * 4);
      GEMM_SYNC();
      step(nrec, nsrc * 4 + 1);
      src = nsrc;
    }
  }
  const unsigned tmo_v = ld_agent(tmo);
  asm volatile("s_waitcnt vmcnt(1)" ::: "memory");

  const __amdgpu_buffer_rsrc_t ro = make_rsrc(p.out[rank], (unsigned)OUT_BYTES);
  const unsigned row_off = (unsigned)(mt * BM + wm * 64 + gq * 4) * GN * 4u;
  const unsigned col_off = (unsigned)(nt * BN + wn * 64 + c16 * 4) * 4u;

  if (g == (unsigned)rank) {
    const bool bad = tmo_v == call;
#pragma unroll
    for (int i = 0; i < 4; ++i)
#pragma unroll
      for (int r = 0; r < 4; ++r) {
        v4f v;
#pragma unroll
        for (int j = 0; j < 4; ++j) {
          const float y = acc[i][j][r] * s;
          v[j] = (y <= 0.f) ? 0.f : y;
          if (bad) v[j] = __builtin_nanf("");
        }
        __builtin_amdgcn_raw_buffer_store_b128(__builtin_bit_cast(v4u, v), ro,
                                               row_off + col_off + (unsigned)(rank * NS * 4) + (unsigned)(i * 16 + r) * (GN * 4), 0, AUX_SYS);
      }
  } else {
    asm volatile("s_waitcnt lgkmcnt(0)" ::: "memory");
    __builtin_amdgcn_s_barrier();
    asm volatile("" ::: "memory");
    uint8_t* const pk = smem + wave * PK_SLOT;
    const unsigned pk_lds = (unsigned)(unsigned long)((__attribute__((address_space(3))) uint8_t*)smem) + wave * PK_SLOT;
    unsigned nxt = pk_lds + 512u;
    unsigned bits[2] = {0u, 0u};
#pragma unroll
    for (int i = 0; i < 4; ++i)
#pragma unroll
      for (int r = 0; r < 4; ++r) {
        float y0, y1, y2, y3;
        unsigned t0, t1, t2, t3, n, b1, b2, b3;
        asm volatile(
            "v_mul_f32 %[y0], %[s], %[a0]\n\t"
            "v_mul_f32 %[y1], %[s], %[a1]\n\t"
            "v_mul_f32 %[y2], %[s], %[a2]\n\t"
            "v_mul_f32 %[y3], %[s], %[a3]\n\t"
            "v_cmp_nge_f32_e64 s[84:85], 0, %[y0]\n\t"
            "v_cmp_nge_f32_e64 s[86:87], 0, %[y1]\n\t"
            "v_cmp_nge_f32_e64 s[88:89], 0, %[y2]\n\t"
            "v_cmp_nge_f32_e64 s[90:91], 0, %[y3]\n\t"
            "s_bcnt1_i32_b64 %[n], s[84:85]\n\t"
            "s_lshl2_add_u32 %[b1], %[n], %[nx]\n\t"
            "s_bcnt1_i32_b64 %[n], s[86:87]\n\t"
            "s_lshl2_add_u32 %[b2], %[n], %[b1]\n\t"
            "s_bcnt1_i32_b64 %[n], s[88:89]\n\t"
            "s_lshl2_add_u32 %[b3], %[n], %[b2]\n\t"
            "v_mbcnt_lo_u32_b32 %[t0], s84, 0\n\t"
            "v_mbcnt_lo_u32_b32 %[t1], s86, 0\n\t"
            "v_mbcnt_lo_u32_b32 %[t2], s88, 0\n\t"
            "v_mbcnt_lo_u32_b32 %[t3], s90, 0\n\t"
            "v_mbcnt_hi_u32_b32 %[t0], s85, %[t0]\n\t"
            "v_mbcnt_hi_u32_b32 %[t1], s87, %[t1]\n\t"
            "v_mbcnt_hi_u32_b32 %[t2], s89, %[t2]\n\t"
            "v_mbcnt_hi_u32_b32 %[t3], s91, %[t3]\n\t"
            "v_lshl_add_u32 %[t0], %[t0], 2, %[nx]\n\t"
            "v_lshl_add_u32 %[t1], %[t1], 2, %[b1]\n\t"
            "v_lshl_add_u32 %[t2], %[t2], 2, %[b2]\n\t"
            "v_lshl_add_u32 %[t3], %[t3], 2, %[b3]\n\t"
            "s_bcnt1_i32_b64 %[n], s[90:91]\n\t"
            "s_lshl2_add_u32 %[nx], %[n], %[b3]\n\t"
            "v_addc_co_u32_e64 %[bt], vcc, %[bt], %[bt], s[84:85]\n\t"
            "v_addc_co_u32_e64 %[bt], vcc, %[bt], %[bt], s[86:87]\n\t"
            "v_addc_co_u32_e64 %[bt], vcc, %[bt], %[bt], s[88:89]\n\t"
            "v_addc_co_u32_e64 %[bt], vcc, %[bt], %[bt], s[90:91]\n\t"
            "s_mov_b64 exec, s[84:85]\n\t"
            "ds_write_b32 %[t0], %[y0]\n\t"
            "s_mov_b64 exec, s[86:87]\n\t"
            "ds_write_b32 %[t1], %[y1]\n\t"
            "s_mov_b64 exec, s[88:89]\n\t"
            "ds_write_b32 %[t2], %[y2]\n\t"
            "s_mov_b64 exec, s[90:91]\n\t"
            "ds_write_b32 %[t3], %[y3]\n\t"
            "s_mov_b64 exec, -1"
            : [y0] "=&v"(y0), [y1] "=&v"(y1), [y2] "=&v"(y2), [y3] "=&v"(y3), [t0] "=&v"(t0), [t1] "=&v"(t1), [t2] "=&v"(t2),
              [t3] "=&v"(t3), [n] "=&s"(n), [b1] "=&s"(b1), [b2] "=&s"(b2), [b3] "=&s"(b3), [nx] "+s"(nxt), [bt] "+v"(bits[i >> 1])
            : [s] "s"(s), [a0] "v"(acc[i][0][r]), [a1] "v"(acc[i][1][r]), [a2] "v"(acc[i][2][r]), [a3] "v"(acc[i][3][r])
            : "vcc", "scc", "s84", "s85", "s86", "s87", "s88", "s89", "s90", "s91", "memory");
      }
    unsigned base = nxt - pk_lds;
    if (tmo_v == call) {
      bits[0] = bits[1] = 0xffffffffu;
      base = 512u + 16384u;
      for (int o = 0; o < 64; ++o) *(float*)(pk + 512 + o * 256 + lane * 4) = __builtin_nanf("");
    }
    *(uint2*)(pk + lane * 8) = make_uint2(bits[0], bits[1]);
    const unsigned bytes = (base + 63u) & ~63u;

    if (tid == 0) link_wait((call - 1u) * 80u + 48u + (ticket - (call - 1u) * 32u), call);
    asm volatile("s_waitcnt lgkmcnt(0)" ::: "memory");
    __builtin_amdgcn_s_barrier();
    asm volatile("" ::: "memory");
    {
      const __amdgpu_buffer_rsrc_t rp = make_rsrc(p.ws[g] + WS_PK, (unsigned)PK_BYTES);
      const unsigned slot = (unsigned)((rank * 32 + idx_in) * 8 + wave) * PK_SLOT;
#pragma unroll
      for (int o = 0; o < 17; ++o) {
        const unsigned off = o * 1024u + lane * 16u;
        if (o * 1024u < bytes) {
          const v4u v = *(const v4u*)(pk + off);
          if (off < bytes) __builtin_amdgcn_raw_buffer_store_b128(v, rp, slot + off, 0, AUX_SYS);
        }
        if (o == REL_AT) link_release();
      }
    }
    asm volatile("s_waitcnt vmcnt(0)" ::: "memory");
    __builtin_amdgcn_s_barrier();
    if (tid == 0) st_sys((unsigned*)(p.ws[g] + WS_OFLAG + ((size_t)rank * 32 + idx_in) * 128), call);

    if (wave == 0) {
      const unsigned* f = lane == 0 ? (const unsigned*)(myws + WS_OFLAG + ((size_t)g * 32 + idx_in) * 128) : nullptr;
      wait_flag(f, call, tmo);
    }
    asm volatile("" ::: "memory");
    __builtin_amdgcn_s_barrier();
    asm volatile("" ::: "memory");
    const bool bad2 = ld_agent(tmo) == call;
    {
      const __amdgpu_buffer_rsrc_t rq = make_rsrc(myws + WS_PK, (unsigned)PK_BYTES);
      const unsigned src = (unsigned)((g * 32 + idx_in) * 8 + wave) * PK_SLOT;
#pragma unroll
      for (int o = 0; o < 17; ++o) bufld16(rq, pk + o * 1024, lane * 16, src + o * 1024, AUX_SYS);
      asm volatile("s_waitcnt vmcnt(0)" ::: "memory");
      const uint2 mm = *(const uint2*)(pk + lane * 8);
      *(float*)(pk + 17152u + lane * 4u) = 0.f;
      const unsigned zero_at = pk_lds + 17152u + lane * 4u;
      unsigned rb = pk_lds + 512u;
      unsigned mb[2] = {mm.x, mm.y};
#define DEC4(v0, v1, v2, v3) \
            "v_add_co_u32_e64 %[mb], s[84:85], %[mb], %[mb]\n\t" \
            "v_add_co_u32_e64 %[mb], s[86:87], %[mb], %[mb]\n\t" \
            "v_add_co_u32_e64 %[mb], s[88:89], %[mb], %[mb]\n\t" \
            "v_add_co_u32_e64 %[mb], s[90:91], %[mb], %[mb]\n\t" \
            "s_bcnt1_i32_b64 %[n], s[84:85]\n\t" \
            "s_lshl2_add_u32 %[b1], %[n], %[rb]\n\t" \
            "s_bcnt1_i32_b64 %[n], s[86:87]\n\t" \
            "s_lshl2_add_u32 %[b2], %[n], %[b1]\n\t" \
            "s_bcnt1_i32_b64 %[n], s[88:89]\n\t" \
            "s_lshl2_add_u32 %[b3], %[n], %[b2]\n\t" \
            "v_mbcnt_lo_u32_b32 " v0 ", s84, 0\n\t" \
            "v_mbcnt_lo_u32_b32 " v1 ", s86, 0\n\t" \
            "v_mbcnt_lo_u32_b32 " v2 ", s88, 0\n\t" \
            "v_mbcnt_lo_u32_b32 " v3 ", s90, 0\n\t" \
            "v_mbcnt_hi_u32_b32 " v0 ", s85, " v0 "\n\t" \
            "v_mbcnt_hi_u32_b32 " v1 ", s87, " v1 "\n\t" \
            "v_mbcnt_hi_u32_b32 " v2 ", s89, " v2 "\n\t" \
            "v_mbcnt_hi_u32_b32 " v3 ", s91, " v3 "\n\t" \
            "v_lshl_add_u32 " v0 ", " v0 ", 2, %[rb]\n\t" \
            "v_lshl_add_u32 " v1 ", " v1 ", 2, %[b1]\n\t" \
            "v_lshl_add_u32 " v2 ", " v2 ", 2, %[b2]\n\t" \
            "v_lshl_add_u32 " v3 ", " v3 ", 2, %[b3]\n\t" \
            "s_bcnt1_i32_b64 %[n], s[90:91]\n\t" \
            "s_lshl2_add_u32 %[rb], %[n], %[b3]\n\t" \
            "v_cndmask_b32_e64 " v0 ", %[z], " v0 ", s[84:85]\n\t" \
            "v_cndmask_b32_e64 " v1 ", %[z], " v1 ", s[86:87]\n\t" \
            "v_cndmask_b32_e64 " v2 ", %[z], " v2 ", s[88:89]\n\t" \
            "v_cndmask_b32_e64 " v3 ", %[z], " v3 ", s[90:91]\n\t" \
            "ds_read_b32 " v0 ", " v0 "\n\t" \
            "ds_read_b32 " v1 ", " v1 "\n\t" \
            "ds_read_b32 " v2 ", " v2 "\n\t" \
            "ds_read_b32 " v3 ", " v3 "\n\t"
#pragma unroll
      for (int i = 0; i < 4; ++i) {
        unsigned e[16], n, b1, b2, b3;
        asm volatile(
            DEC4("%[e0]", "%[e1]", "%[e2]", "%[e3]")
            DEC4("%[e4]", "%[e5]", "%[e6]", "%[e7]")
            DEC4("%[e8]", "%[e9]", "%[e10]", "%[e11]")
            DEC4("%[e12]", "%[e13]", "%[e14]", "%[e15]")
            "s_waitcnt lgkmcnt(0)"
            : [e0] "=&v"(e[0]), [e1] "=&v"(e[1]), [e2] "=&v"(e[2]), [e3] "=&v"(e[3]), [e4] "=&v"(e[4]), [e5] "=&v"(e[5]),
              [e6] "=&v"(e[6]), [e7] "=&v"(e[7]), [e8] "=&v"(e[8]), [e9] "=&v"(e[9]), [e10] "=&v"(e[10]), [e11] "=&v"(e[11]),
              [e12] "=&v"(e[12]), [e13] "=&v"(e[13]), [e14] "=&v"(e[14]), [e15] "=&v"(e[15]),
              [n] "=&s"(n), [b1] "=&s"(b1), [b2] "=&s"(b2), [b3] "=&s"(b3), [rb] "+s"(rb), [mb] "+v"(mb[i >> 1])
            : [z] "v"(zero_at)
            : "scc", "s84", "s85", "s86", "s87", "s88", "s89", "s90", "s91", "memory");
#pragma unroll
        for (int r = 0; r < 4; ++r) {
          v4u v = {e[r * 4], e[r * 4 + 1], e[r * 4 + 2], e[r * 4 + 3]};
          if (bad2) v = v4u{0x7fc00000u, 0x7fc00000u, 0x7fc00000u, 0x7fc00000u};
          __builtin_amdgcn_raw_buffer_store_b128(v, ro, row_off + col_off + g * (NS * 4) + (unsigned)(i * 16 + r) * (GN * 4), 0, AUX_SYS);
        }
      }
    }
  }
  if (tid == 0) {
    const unsigned t = __hip_atomic_fetch_add((unsigned*)(myws + WS_EXIT), 1u, __ATOMIC_RELAXED, __HIP_MEMORY_SCOPE_AGENT);
    if ((t + 1u) % NGRID == 0u) st_agent((unsigned*)(myws + WS_CALL), call);
  }
}

}

extern "C" void kernel_launch(int rank, int nranks, ncclComm_t comm,
                              hipStream_t stream,
                              void* const* d_in, const int* in_sizes, int n_in,
                              void* d_out, int out_size,
                              const int* in_dtypes, int out_dtype,
                              void* const* d_out_all,
                              void* const* d_ws_all, size_t ws_bytes) {
  (void)comm; (void)in_dtypes; (void)out_dtype; (void)d_out;
  if (nranks != WORLD || n_in != 4 || ws_bytes < WS_NEED || (unsigned)rank >= (unsigned)WORLD ||
      in_sizes[0] != GM * KS || in_sizes[1] != KS * GN || out_size != MB * GN) {
    fprintf(stderr, "kernel_launch: unexpected problem shape\n");
    return;
  }
  Params p;
  p.x = (const uint8_t*)d_in[0];
  p.w = (const uint8_t*)d_in[1];
  p.sx = (const float*)d_in[2];
  p.sw = (const float*)d_in[3];
  for (int r = 0; r < WORLD; ++r) {
    p.out[r] = (float*)d_out_all[r];
    p.ws[r] = (uint8_t*)d_ws_all[r];
  }
  p.rank = rank;
  gemm_rs_kernel<<<dim3(NGRID), dim3(NTHREADS), 0, stream>>>(p);
  HIP_CHECK(hipGetLastError());
}
```

```cpp
#include <hip/hip_runtime.h>
#include <rccl/rccl.h>
#include <cstdio>
#include <cstdint>

#define HIP_CHECK(expr) do { hipError_t _e = (expr); if (_e != hipSuccess) { \
    fprintf(stderr, "HIP error %s at %s:%d\n", hipGetErrorString(_e), __FILE__, __LINE__); \
    return; } } while (0)

namespace {

constexpr int WORLD = 8;
constexpr int GM = 4096, GK = 4096, GN = 16384;
constexpr int KS = GK / WORLD;
constexpr int NS = GN / WORLD;
constexpr int MB = GM / WORLD;
constexpr int BM = 128, BN = 256, BK = 128;
constexpr int NKB = GK / BK;
constexpr int A_STAGE = BM * BK;
constexpr int B_STAGE = BN * BK;
constexpr int STAGE = A_STAGE + B_STAGE;
constexpr int NSTAGE = 3;
constexpr int NTHREADS = 512;
constexpr int NGRID = 256;
constexpr int NPUSH = 8;
constexpr unsigned NTOK = 2;
constexpr int REL_AT = 4;

constexpr size_t WS_CALL = 0;
constexpr size_t WS_EXIT = 128;
constexpr size_t WS_TMO = 256;
constexpr size_t WS_QTICK = 1024;
constexpr size_t WS_QDONE = 2048;
constexpr size_t WS_XFLAG = 4096;
constexpr size_t WS_WFLAG = 8192;
constexpr size_t WS_PQ = 12288;
constexpr size_t WS_OFLAG = 16384;
constexpr size_t WS_STAGE = 1u << 20;
constexpr size_t XS_BYTES = (size_t)GM * GK;
constexpr size_t WT_BYTES = (size_t)GK * NS;
constexpr size_t PAR_BYTES = XS_BYTES + WT_BYTES;
constexpr unsigned PK_SLOT = 17408;
constexpr size_t WS_PK = WS_STAGE + 2 * PAR_BYTES;
constexpr size_t PK_BYTES = (size_t)WORLD * 32 * 8 * PK_SLOT;
constexpr size_t WS_NEED = WS_PK + PK_BYTES;
constexpr size_t OUT_BYTES = (size_t)MB * GN * 4;

constexpr unsigned long long DEADLINE_TICKS = 50000000ull;

typedef int v4i __attribute__((ext_vector_type(4)));
typedef unsigned v4u __attribute__((ext_vector_type(4)));
typedef int v8i __attribute__((ext_vector_type(8)));
typedef float v4f __attribute__((ext_vector_type(4)));

struct Params {
  const uint8_t* x;
  const uint8_t* w;
  const float* sx;
  const float* sw;
  float* out[WORLD];
  uint8_t* ws[WORLD];
  int rank;
  unsigned label_map;
};

#define AUX_SYS 17

__device__ __forceinline__ __amdgpu_buffer_rsrc_t make_rsrc(void* p, unsigned bytes) {
  return __builtin_amdgcn_make_buffer_rsrc(p, 0, bytes, 0x00020000);
}
#define bufld16(rs, l, voff, soff, aux) \
  __builtin_amdgcn_raw_ptr_buffer_load_lds(rs, (__attribute__((address_space(3))) void*)(l), 16, voff, soff, 0, aux)
__device__ __forceinline__ unsigned ld_sys(const unsigned* p) {
  return __hip_atomic_load(p, __ATOMIC_RELAXED, __HIP_MEMORY_SCOPE_SYSTEM);
}
__device__ __forceinline__ unsigned ld_agent(const unsigned* p) {
  return __hip_atomic_load(p, __ATOMIC_RELAXED, __HIP_MEMORY_SCOPE_AGENT);
}
__device__ __forceinline__ void st_sys(unsigned* p, unsigned v) {
  __hip_atomic_store(p, v, __ATOMIC_RELAXED, __HIP_MEMORY_SCOPE_SYSTEM);
}
__device__ __forceinline__ void st_agent(unsigned* p, unsigned v) {
  __hip_atomic_store(p, v, __ATOMIC_RELAXED, __HIP_MEMORY_SCOPE_AGENT);
}

__device__ __forceinline__ void wait_flag(const unsigned* flag, unsigned call, unsigned* tmo) {
  const unsigned long long t0 = __builtin_amdgcn_s_memrealtime();
  for (unsigned it = 1;; ++it) {
    unsigned v = call;
    if (flag) v = ld_sys(flag);
    if (__all(v >= call)) break;
    if ((it & 7u) == 0u && (ld_agent(tmo) == call || __builtin_amdgcn_s_memrealtime() - t0 > DEADLINE_TICKS)) {
      st_agent(tmo, call);
      break;
    }
    __builtin_amdgcn_s_sleep(2);
  }
  asm volatile("s_waitcnt vmcnt(0)" ::: "memory");
}

__global__ void __launch_bounds__(NTHREADS) gemm_rs_kernel(Params p) {
  __shared__ __attribute__((aligned(16))) uint8_t smem[NSTAGE * STAGE + 64];

  const int tid = threadIdx.x;
  const int lane = tid & 63;
  const int wave = __builtin_amdgcn_readfirstlane(tid >> 6);
  const unsigned g = (p.label_map >> ((blockIdx.x & 7u) * 4u)) & 7u;
  const int idx_in = blockIdx.x >> 3;
  const int rank = p.rank;
  uint8_t* const myws = p.ws[rank];
  unsigned* const tmo = (unsigned*)(myws + WS_TMO);
  const unsigned call_v = ld_agent((const unsigned*)(myws + WS_CALL));

  const int c16 = lane & 15, gq = lane >> 4;
  const bool remote = g != (unsigned)rank;

  unsigned* const qdone = (unsigned*)(myws + WS_QDONE + g * 128);
  auto link_wait = [&](unsigned ticket, unsigned call) {
    const unsigned long long t0 = __builtin_amdgcn_s_memrealtime();
    for (unsigned it = 1; (int)(ld_agent(qdone) + NTOK - ticket) <= 0; ++it) {
      if ((it & 15u) == 0u && (ld_agent(tmo) == call || __builtin_amdgcn_s_memrealtime() - t0 > DEADLINE_TICKS)) {
        st_agent(tmo, call);
        break;
      }
      __builtin_amdgcn_s_sleep(2);
    }
  };
  auto link_release = [&]() {
    asm volatile("" ::: "memory");
    __builtin_amdgcn_s_barrier();
    if (tid == 0 && remote) __hip_atomic_fetch_add(qdone, 1u, __ATOMIC_RELAXED, __HIP_MEMORY_SCOPE_AGENT);
    asm volatile("" ::: "memory");
  };

  if (idx_in < NPUSH) {
    uint8_t* const dws = p.ws[g];
    auto* const lds_next = (__attribute__((address_space(3))) unsigned*)smem;
    const int wn = wave & 3;
    const uint8_t* const wsrc = p.w + (size_t)((wave >> 2) * 128 + (idx_in & 1) * 256 + gq * 16) * GN + (size_t)g * NS + (idx_in >> 1) * BN + wn * 64 + c16 * 4;
    const unsigned wdst = (unsigned)XS_BYTES + (unsigned)(idx_in >> 1) * (NKB * B_STAGE) + (unsigned)(rank * 4 + (idx_in & 1) * 2 + (wave >> 2)) * B_STAGE +
                          (unsigned)wn * 4 * 2048 + lane * 16;
    const unsigned xsrc = (unsigned)(wave * 16 + c16) * KS + gq * 16;
    const unsigned xdst = (unsigned)(rank * 4) * A_STAGE + (unsigned)wave * 8192 + lane * 16;
    unsigned* const pq = (unsigned*)(myws + WS_PQ + g * 128);

    auto load_x = [&](v4u (&b)[8], int tmq) {
      const __amdgpu_buffer_rsrc_t rx = make_rsrc((void*)p.x, (unsigned)tmq < 32u ? (unsigned)(GM * KS) : 0u);
#pragma unroll
      for (int u = 0; u < 8; ++u) b[u] = __builtin_amdgcn_raw_buffer_load_b128(rx, xsrc, (tmq & 31) * (BM * KS) + u * 64, 0);
    };
    auto load_w = [&](unsigned (&b)[32], int half) {
#pragma unroll
      for (int t = 0; t < 32; ++t) b[t] = *(const unsigned*)(wsrc + half * (4 * BN) + (size_t)((t >> 4) * 64 + (t & 15)) * GN);
    };
    v4u xa[8], xb[8];
    unsigned wa[32], wb[32];
    load_x(xa, idx_in * 4);
    asm volatile("" ::: "memory");
    const unsigned call = __builtin_amdgcn_readfirstlane(call_v) + 1u;
    const __amdgpu_buffer_rsrc_t rs = make_rsrc(dws + WS_STAGE + (size_t)(call & 1u) * PAR_BYTES, (unsigned)PAR_BYTES);
    const unsigned pq_base = (call - 1u) * (24u + NPUSH);

    auto store_x = [&](const v4u (&b)[8], int tmq) {
      const unsigned dst = xdst + (unsigned)tmq * (NKB * A_STAGE);
#pragma unroll
      for (int u = 0; u < 8; ++u) {
        __builtin_amdgcn_raw_buffer_store_b128(b[u], rs, dst + u * 1024, 0, AUX_SYS);
        if (u == REL_AT) link_release();
      }
    };
    auto store_w = [&](const unsigned (&b)[32], int half) {
      const unsigned dst = wdst + (unsigned)half * (4u * NKB * B_STAGE);
#pragma unroll
      for (int qh = 0; qh < 2; ++qh) {
        v4u o[4];
#pragma unroll
        for (int u = 0; u < 4; ++u) {
          const unsigned* v = b + qh * 16 + 4 * u;
          const unsigned t0 = __builtin_amdgcn_perm(v[1], v[0], 0x05010400u);
          const unsigned t1 = __builtin_amdgcn_perm(v[1], v[0], 0x07030602u);
          const unsigned t2 = __builtin_amdgcn_perm(v[3], v[2], 0x05010400u);
          const unsigned t3 = __builtin_amdgcn_perm(v[3], v[2], 0x07030602u);
          o[0][u] = __builtin_amdgcn_perm(t2, t0, 0x05040100u);
          o[1][u] = __builtin_amdgcn_perm(t2, t0, 0x07060302u);
          o[2][u] = __builtin_amdgcn_perm(t3, t1, 0x05040100u);
          o[3][u] = __builtin_amdgcn_perm(t3, t1, 0x07060302u);
        }
#pragma unroll
        for (int j = 0; j < 4; ++j) {
          __builtin_amdgcn_raw_buffer_store_b128(o[j], rs, dst + j * 2048 + qh * 1024, 0, AUX_SYS);
          if (qh * 4 + j == REL_AT) link_release();
        }
      }
    };
    auto xflag = [&](int tmq) { return (unsigned*)(dws + WS_XFLAG + (size_t)tmq * 128) + rank; };
    auto wflag = [&](int half) { return (unsigned*)(dws + WS_WFLAG + (size_t)(half * 8 + idx_in) * 128) + rank; };
#define PUSH_FENCE() asm volatile("" ::: "memory")
#define PUSH_PUBLISH(N, flagptr) do { \
      asm volatile("s_waitcnt vmcnt(" #N ")" ::: "memory"); \
      __builtin_amdgcn_s_barrier(); PUSH_FENCE(); \
      if (tid == 0) st_sys(flagptr, call); \
      PUSH_FENCE(); } while (0)
#define PUSH_DRAW() do { if (tid == 0) { \
      const unsigned d = __hip_atomic_fetch_add(pq, 1u, __ATOMIC_RELAXED, __HIP_MEMORY_SCOPE_AGENT) - pq_base; \
      *lds_next = d < 24u ? (d & 7u) * 4u + 1u + (d >> 3) : 32u; } } while (0)
    const unsigned tk_base = (call - 1u) * 80u;
    auto xticket = [&](int tmq) { return tk_base + ((tmq & 3) == 0 ? (unsigned)(tmq >> 2) : 16u + (unsigned)(tmq & 3) * 8u + (unsigned)(tmq >> 2)); };
    auto go = [&](unsigned ticket) {
      if (tid == 0 && remote) link_wait(ticket, call);
      asm volatile("s_waitcnt lgkmcnt(0)" ::: "memory");
      __builtin_amdgcn_s_barrier();
      PUSH_FENCE();
    };
    if (idx_in >= (int)NTOK) go(xticket(idx_in * 4));
    store_x(xa, idx_in * 4); PUSH_FENCE();
    load_w(wa, 0); PUSH_FENCE();
    PUSH_PUBLISH(32, xflag(idx_in * 4));
    PUSH_DRAW();
    go(tk_base + 8u + idx_in);
    int ta = __builtin_amdgcn_readfirstlane(*lds_next);
    store_w(wa, 0); PUSH_FENCE();
    load_w(wb, 1); PUSH_FENCE();
    load_x(xa, ta); PUSH_FENCE();
    PUSH_PUBLISH(40, wflag(0));
    if (ta < 32) PUSH_DRAW();
    go(tk_base + 16u + idx_in);
    int tb = ta < 32 ? __builtin_amdgcn_readfirstlane(*lds_next) : 32;
    store_w(wb, 1); PUSH_FENCE();
    load_x(xb, tb); PUSH_FENCE();
    PUSH_PUBLISH(8, wflag(1));
    for (;;) {
      if (ta >= 32) break;
      {
        const int cur = ta;
        if (tb < 32) PUSH_DRAW();
        go(xticket(cur));
        ta = tb < 32 ? __builtin_amdgcn_readfirstlane(*lds_next) : 32;
        store_x(xa, cur); PUSH_FENCE();
        load_x(xa, ta); PUSH_FENCE();
        PUSH_PUBLISH(8, xflag(cur));
      }
      if (tb >= 32) break;
      {
        const int cur = tb;
        if (ta < 32) PUSH_DRAW();
        go(xticket(cur));
        tb = ta < 32 ? __builtin_amdgcn_readfirstlane(*lds_next) : 32;
        store_x(xb, cur); PUSH_FENCE();
        load_x(xb, tb); PUSH_FENCE();
        PUSH_PUBLISH(8, xflag(cur));
      }
    }
  }
  const unsigned call = __builtin_amdgcn_readfirstlane(call_v) + 1u;
  const size_t stage_off = WS_STAGE + (size_t)(call & 1u) * PAR_BYTES;

  const int idx = (idx_in - NPUSH) & 31;
  const int mt = idx >> 3, nt = idx & 7;
  const int tm = (int)g * 4 + mt;
  auto* const lds_sel = (__attribute__((address_space(3))) int*)(smem + NSTAGE * STAGE);
  const unsigned* myflag = nullptr;
  if (lane < 8) myflag = (const unsigned*)(myws + WS_XFLAG + (size_t)tm * 128) + lane;
  else if (lane < 24) myflag = (const unsigned*)(myws + WS_WFLAG + (size_t)(nt * 2 + ((lane - 8) >> 3)) * 128) + (lane & 7);
  unsigned rem = 0xffu, ready = 0u;
  auto select = [&]() {
    const unsigned long long t0 = __builtin_amdgcn_s_memrealtime();
    while ((ready & rem) == 0u) {
      unsigned v = call;
      if (myflag) v = ld_sys(myflag);
      const unsigned m = (unsigned)__builtin_amdgcn_ballot_w64(v >= call);
      ready = m & (m >> 8) & (m >> 16) & 0xffu;
      if (ready & rem) break;
      if (ld_agent(tmo) == call || __builtin_amdgcn_s_memrealtime() - t0 > DEADLINE_TICKS) {
        if (lane == 0) st_agent(tmo, call);
        ready = 0xffu;
        break;
      }
      __builtin_amdgcn_s_sleep(4);
    }
    const unsigned cand = ready & rem;
    const int sel = (cand >> rank) & 1u ? rank : __builtin_ctz(cand);
    rem &= ~(1u << sel);
    if (lane == 0) *lds_sel = sel;
  };
  if (wave == 0) {
    select();
    __builtin_amdgcn_fence(__ATOMIC_ACQUIRE, "");
    asm volatile("s_waitcnt vmcnt(0) lgkmcnt(0)" ::: "memory");
  }
  __builtin_amdgcn_s_barrier();
  asm volatile("" ::: "memory");
  int src = __builtin_amdgcn_readfirstlane(*lds_sel);

  const int wm = wave >> 2, wn = wave & 3;
  v4f acc[4][4];
#pragma unroll
  for (int i = 0; i < 4; ++i)
#pragma unroll
    for (int j = 0; j < 4; ++j) acc[i][j] = v4f{0.f, 0.f, 0.f, 0.f};
  const float s = p.sx[0] * p.sw[0];
  unsigned ticket = 0;

  {
    uint8_t* const sbase = myws + stage_off;
    const unsigned va = (unsigned)tm * (NKB * A_STAGE) + wave * 8192 + lane * 16;
    const unsigned vb = (unsigned)XS_BYTES + (unsigned)nt * (NKB * B_STAGE) + wave * 4096 + lane * 16;
    auto issue1 = [&](unsigned nrec, int kb, int boff, int c) {
      const __amdgpu_buffer_rsrc_t rs = make_rsrc(sbase, nrec);
      if (c < 2) bufld16(rs, smem + boff + wave * 2048 + c * 1024, va, (kb >> 2) * (4 * A_STAGE) + (kb & 3) * 2048 + c * 1024, 0);
      else bufld16(rs, smem + boff + A_STAGE + wave * 4096 + (c - 2) * 1024, vb, kb * B_STAGE + (c - 2) * 1024, 0);
    };
#pragma unroll
    for (int c = 0; c < 6; ++c) issue1((unsigned)PAR_BYTES, src * 4, 0, c);
#pragma unroll
    for (int c = 0; c < 6; ++c) issue1((unsigned)PAR_BYTES, src * 4 + 1, STAGE, c);
    int boff = 0;
    const int a_rd = wm * 4 * 2048 + lane * 16;
    const int b_rd = A_STAGE + wn * 4 * 2048 + lane * 16;
    auto step = [&](unsigned nrec, int pkb) {
      int nb = boff + 2 * STAGE;
      if (nb >= NSTAGE * STAGE) nb -= NSTAGE * STAGE;
      const uint8_t* sa = smem + boff + a_rd;
      const uint8_t* sb = smem + boff + b_rd;
      v8i af[4], bf[4];
      auto rd = [&](const uint8_t* q) {
        const v4i lo = *(const v4i*)q, hi = *(const v4i*)(q + 1024);
        return __builtin_shufflevector(lo, hi, 0, 1, 2, 3, 4, 5, 6, 7);
      };
      af[0] = rd(sa);
#pragma unroll
      for (int j = 0; j < 4; ++j) bf[j] = rd(sb + j * 2048);
#pragma unroll
      for (int i = 1; i < 4; ++i) af[i] = rd(sa + i * 2048);
      __builtin_amdgcn_sched_barrier(0);
#pragma unroll
      for (int i = 0; i < 4; ++i) {
#pragma unroll
        for (int j = 0; j < 4; ++j) {
          acc[i][j] = __builtin_amdgcn_mfma_scale_f32_16x16x128_f8f6f4(af[i], bf[j], acc[i][j], 0  , 1  , 0, 0, 0, 0);
          if ((j & 1) && i < 3) {
            __builtin_amdgcn_sched_barrier(0);
            issue1(nrec, pkb, nb, i * 2 + (j >> 1));
            __builtin_amdgcn_sched_barrier(0);
          }
        }
      }
      __builtin_amdgcn_s_waitcnt(0xC07F);
      boff += STAGE;
      if (boff >= NSTAGE * STAGE) boff = 0;
    };
#define GEMM_SYNC() do { asm volatile("s_waitcnt vmcnt(6)" ::: "memory"); __builtin_amdgcn_s_barrier(); asm volatile("" ::: "memory"); } while (0)
    __builtin_amdgcn_s_waitcnt(0xC07F);
    for (int slot = 0; slot < WORLD; ++slot) {
      const bool more = slot + 1 < WORLD;
      const unsigned nrec = more ? (unsigned)PAR_BYTES : 0u;
      GEMM_SYNC();
      step((unsigned)PAR_BYTES, src * 4 + 2);
      GEMM_SYNC();
      step((unsigned)PAR_BYTES, src * 4 + 3);
      if (more && wave == 0) {
        select();
        if (slot == WORLD - 2) {
          if (lane == 0 && g != (unsigned)rank)
            ticket = __hip_atomic_fetch_add((unsigned*)(myws + WS_QTICK + g * 128), 1u, __ATOMIC_RELAXED, __HIP_MEMORY_SCOPE_AGENT);
        }
        asm volatile("s_waitcnt lgkmcnt(0)" ::: "memory");
      }
      GEMM_SYNC();
      const int nsrc = more ? __builtin_amdgcn_readfirstlane(*lds_sel) : 0;
      __builtin_amdgcn_s_waitcnt(0xC07F);
      step(nrec, nsrc * 4);
      GEMM_SYNC();
      step(nrec, nsrc * 4 + 1);
      src = nsrc;
    }
  }
  const unsigned tmo_v = ld_agent(tmo);
  asm volatile("s_waitcnt vmcnt(1)" ::: "memory");

  const __amdgpu_buffer_rsrc_t ro = make_rsrc(p.out[rank], (unsigned)OUT_BYTES);
  const unsigned row_off = (unsigned)(mt * BM + wm * 64 + gq * 4) * GN * 4u;
  const unsigned col_off = (unsigned)(nt * BN + wn * 64 + c16 * 4) * 4u;

  if (g == (unsigned)rank) {
    const bool bad = tmo_v == call;
#pragma unroll
    for (int i = 0; i < 4; ++i)
#pragma unroll
      for (int r = 0; r < 4; ++r) {
        v4f v;
#pragma unroll
        for (int j = 0; j < 4; ++j) {
          const float y = acc[i][j][r] * s;
          v[j] = (y <= 0.f) ? 0.f : y;
          if (bad) v[j] = __builtin_nanf("");
        }
        __builtin_amdgcn_raw_buffer_store_b128(__builtin_bit_cast(v4u, v), ro,
                                               row_off + col_off + (unsigned)(rank * NS * 4) + (unsigned)(i * 16 + r) * (GN * 4), 0, AUX_SYS);
      }
  } else {
    asm volatile("s_waitcnt lgkmcnt(0)" ::: "memory");
    __builtin_amdgcn_s_barrier();
    asm volatile("" ::: "memory");
    uint8_t* const pk = smem + wave * PK_SLOT;
    const unsigned pk_lds = (unsigned)(unsigned long)((__attribute__((address_space(3))) uint8_t*)smem) + wave * PK_SLOT;
    unsigned nxt = pk_lds + 512u;
    unsigned bits[2] = {0u, 0u};
#pragma unroll
    for (int i = 0; i < 4; ++i)
#pragma unroll
      for (int r = 0; r < 4; ++r) {
        float y0, y1, y2, y3;
        unsigned t0, t1, t2, t3, n, b1, b2, b3;
        asm volatile(
            "v_mul_f32 %[y0], %[s], %[a0]\n\t"
            "v_mul_f32 %[y1], %[s], %[a1]\n\t"
            "v_mul_f32 %[y2], %[s], %[a2]\n\t"
            "v_mul_f32 %[y3], %[s], %[a3]\n\t"
            "v_cmp_nge_f32_e64 s[84:85], 0, %[y0]\n\t"
            "v_cmp_nge_f32_e64 s[86:87], 0, %[y1]\n\t"
            "v_cmp_nge_f32_e64 s[88:89], 0, %[y2]\n\t"
            "v_cmp_nge_f32_e64 s[90:91], 0, %[y3]\n\t"
            "s_bcnt1_i32_b64 %[n], s[84:85]\n\t"
            "s_lshl2_add_u32 %[b1], %[n], %[nx]\n\t"
            "s_bcnt1_i32_b64 %[n], s[86:87]\n\t"
            "s_lshl2_add_u32 %[b2], %[n], %[b1]\n\t"
            "s_bcnt1_i32_b64 %[n], s[88:89]\n\t"
            "s_lshl2_add_u32 %[b3], %[n], %[b2]\n\t"
            "v_mbcnt_lo_u32_b32 %[t0], s84, 0\n\t"
            "v_mbcnt_lo_u32_b32 %[t1], s86, 0\n\t"
            "v_mbcnt_lo_u32_b32 %[t2], s88, 0\n\t"
            "v_mbcnt_lo_u32_b32 %[t3], s90, 0\n\t"
            "v_mbcnt_hi_u32_b32 %[t0], s85, %[t0]\n\t"
            "v_mbcnt_hi_u32_b32 %[t1], s87, %[t1]\n\t"
            "v_mbcnt_hi_u32_b32 %[t2], s89, %[t2]\n\t"
            "v_mbcnt_hi_u32_b32 %[t3], s91, %[t3]\n\t"
            "v_lshl_add_u32 %[t0], %[t0], 2, %[nx]\n\t"
            "v_lshl_add_u32 %[t1], %[t1], 2, %[b1]\n\t"
            "v_lshl_add_u32 %[t2], %[t2], 2, %[b2]\n\t"
            "v_lshl_add_u32 %[t3], %[t3], 2, %[b3]\n\t"
            "s_bcnt1_i32_b64 %[n], s[90:91]\n\t"
            "s_lshl2_add_u32 %[nx], %[n], %[b3]\n\t"
            "v_addc_co_u32_e64 %[bt], vcc, %[bt], %[bt], s[84:85]\n\t"
            "v_addc_co_u32_e64 %[bt], vcc, %[bt], %[bt], s[86:87]\n\t"
            "v_addc_co_u32_e64 %[bt], vcc, %[bt], %[bt], s[88:89]\n\t"
            "v_addc_co_u32_e64 %[bt], vcc, %[bt], %[bt], s[90:91]\n\t"
            "s_mov_b64 exec, s[84:85]\n\t"
            "ds_write_b32 %[t0], %[y0]\n\t"
            "s_mov_b64 exec, s[86:87]\n\t"
            "ds_write_b32 %[t1], %[y1]\n\t"
            "s_mov_b64 exec, s[88:89]\n\t"
            "ds_write_b32 %[t2], %[y2]\n\t"
            "s_mov_b64 exec, s[90:91]\n\t"
            "ds_write_b32 %[t3], %[y3]\n\t"
            "s_mov_b64 exec, -1"
            : [y0] "=&v"(y0), [y1] "=&v"(y1), [y2] "=&v"(y2), [y3] "=&v"(y3), [t0] "=&v"(t0), [t1] "=&v"(t1), [t2] "=&v"(t2),
              [t3] "=&v"(t3), [n] "=&s"(n), [b1] "=&s"(b1), [b2] "=&s"(b2), [b3] "=&s"(b3), [nx] "+s"(nxt), [bt] "+v"(bits[i >> 1])
            : [s] "s"(s), [a0] "v"(acc[i][0][r]), [a1] "v"(acc[i][1][r]), [a2] "v"(acc[i][2][r]), [a3] "v"(acc[i][3][r])
            : "vcc", "scc", "s84", "s85", "s86", "s87", "s88", "s89", "s90", "s91", "memory");
      }
    unsigned base = nxt - pk_lds;
    if (tmo_v == call) {
      bits[0] = bits[1] = 0xffffffffu;
      base = 512u + 16384u;
      for (int o = 0; o < 64; ++o) *(float*)(pk + 512 + o * 256 + lane * 4) = __builtin_nanf("");
    }
    *(uint2*)(pk + lane * 8) = make_uint2(bits[0], bits[1]);
    const unsigned bytes = (base + 63u) & ~63u;

    if (tid == 0) link_wait((call - 1u) * 80u + 48u + (ticket - (call - 1u) * 32u), call);
    asm volatile("s_waitcnt lgkmcnt(0)" ::: "memory");
    __builtin_amdgcn_s_barrier();
    asm volatile("" ::: "memory");
    {
      const __amdgpu_buffer_rsrc_t rp = make_rsrc(p.ws[g] + WS_PK, (unsigned)PK_BYTES);
      const unsigned slot = (unsigned)((rank * 32 + idx_in) * 8 + wave) * PK_SLOT;
#pragma unroll
      for (int o = 0; o < 17; ++o) {
        const unsigned off = o * 1024u + lane * 16u;
        if (o * 1024u < bytes) {
          const v4u v = *(const v4u*)(pk + off);
          if (off < bytes) __builtin_amdgcn_raw_buffer_store_b128(v, rp, slot + off, 0, AUX_SYS);
        }
        if (o == REL_AT) link_release();
      }
    }
    asm volatile("s_waitcnt vmcnt(0)" ::: "memory");
    __builtin_amdgcn_s_barrier();
    if (tid == 0) st_sys((unsigned*)(p.ws[g] + WS_OFLAG + ((size_t)rank * 32 + idx_in) * 128), call);

    if (wave == 0) {
      const unsigned* f = lane == 0 ? (const unsigned*)(myws + WS_OFLAG + ((size_t)g * 32 + idx_in) * 128) : nullptr;
      wait_flag(f, call, tmo);
    }
    asm volatile("" ::: "memory");
    __builtin_amdgcn_s_barrier();
    asm volatile("" ::: "memory");
    const bool bad2 = ld_agent(tmo) == call;
    {
      const __amdgpu_buffer_rsrc_t rq = make_rsrc(myws + WS_PK, (unsigned)PK_BYTES);
      const unsigned src = (unsigned)((g * 32 + idx_in) * 8 + wave) * PK_SLOT;
#pragma unroll
      for (int o = 0; o < 17; ++o) bufld16(rq, pk + o * 1024, lane * 16, src + o * 1024, AUX_SYS);
      asm volatile("s_waitcnt vmcnt(0)" ::: "memory");
      const uint2 mm = *(const uint2*)(pk + lane * 8);
      *(float*)(pk + 17152u + lane * 4u) = 0.f;
      const unsigned zero_at = pk_lds + 17152u + lane * 4u;
      unsigned rb = pk_lds + 512u;
      unsigned mb[2] = {mm.x, mm.y};
#define DEC4(v0, v1, v2, v3) \
            "v_add_co_u32_e64 %[mb], s[84:85], %[mb], %[mb]\n\t" \
            "v_add_co_u32_e64 %[mb], s[86:87], %[mb], %[mb]\n\t" \
            "v_add_co_u32_e64 %[mb], s[88:89], %[mb], %[mb]\n\t" \
            "v_add_co_u32_e64 %[mb], s[90:91], %[mb], %[mb]\n\t" \
            "s_bcnt1_i32_b64 %[n], s[84:85]\n\t" \
            "s_lshl2_add_u32 %[b1], %[n], %[rb]\n\t" \
            "s_bcnt1_i32_b64 %[n], s[86:87]\n\t" \
            "s_lshl2_add_u32 %[b2], %[n], %[b1]\n\t" \
            "s_bcnt1_i32_b64 %[n], s[88:89]\n\t" \
            "s_lshl2_add_u32 %[b3], %[n], %[b2]\n\t" \
            "v_mbcnt_lo_u32_b32 " v0 ", s84, 0\n\t" \
            "v_mbcnt_lo_u32_b32 " v1 ", s86, 0\n\t" \
            "v_mbcnt_lo_u32_b32 " v2 ", s88, 0\n\t" \
            "v_mbcnt_lo_u32_b32 " v3 ", s90, 0\n\t" \
            "v_mbcnt_hi_u32_b32 " v0 ", s85, " v0 "\n\t" \
            "v_mbcnt_hi_u32_b32 " v1 ", s87, " v1 "\n\t" \
            "v_mbcnt_hi_u32_b32 " v2 ", s89, " v2 "\n\t" \
            "v_mbcnt_hi_u32_b32 " v3 ", s91, " v3 "\n\t" \
            "v_lshl_add_u32 " v0 ", " v0 ", 2, %[rb]\n\t" \
            "v_lshl_add_u32 " v1 ", " v1 ", 2, %[b1]\n\t" \
            "v_lshl_add_u32 " v2 ", " v2 ", 2, %[b2]\n\t" \
            "v_lshl_add_u32 " v3 ", " v3 ", 2, %[b3]\n\t" \
            "s_bcnt1_i32_b64 %[n], s[90:91]\n\t" \
            "s_lshl2_add_u32 %[rb], %[n], %[b3]\n\t" \
            "v_cndmask_b32_e64 " v0 ", %[z], " v0 ", s[84:85]\n\t" \
            "v_cndmask_b32_e64 " v1 ", %[z], " v1 ", s[86:87]\n\t" \
            "v_cndmask_b32_e64 " v2 ", %[z], " v2 ", s[88:89]\n\t" \
            "v_cndmask_b32_e64 " v3 ", %[z], " v3 ", s[90:91]\n\t" \
            "ds_read_b32 " v0 ", " v0 "\n\t" \
            "ds_read_b32 " v1 ", " v1 "\n\t" \
            "ds_read_b32 " v2 ", " v2 "\n\t" \
            "ds_read_b32 " v3 ", " v3 "\n\t"
#pragma unroll
      for (int i = 0; i < 4; ++i) {
        unsigned e[16], n, b1, b2, b3;
        asm volatile(
            DEC4("%[e0]", "%[e1]", "%[e2]", "%[e3]")
            DEC4("%[e4]", "%[e5]", "%[e6]", "%[e7]")
            DEC4("%[e8]", "%[e9]", "%[e10]", "%[e11]")
            DEC4("%[e12]", "%[e13]", "%[e14]", "%[e15]")
            "s_waitcnt lgkmcnt(0)"
            : [e0] "=&v"(e[0]), [e1] "=&v"(e[1]), [e2] "=&v"(e[2]), [e3] "=&v"(e[3]), [e4] "=&v"(e[4]), [e5] "=&v"(e[5]),
              [e6] "=&v"(e[6]), [e7] "=&v"(e[7]), [e8] "=&v"(e[8]), [e9] "=&v"(e[9]), [e10] "=&v"(e[10]), [e11] "=&v"(e[11]),
              [e12] "=&v"(e[12]), [e13] "=&v"(e[13]), [e14] "=&v"(e[14]), [e15] "=&v"(e[15]),
              [n] "=&s"(n), [b1] "=&s"(b1), [b2] "=&s"(b2), [b3] "=&s"(b3), [rb] "+s"(rb), [mb] "+v"(mb[i >> 1])
            : [z] "v"(zero_at)
            : "scc", "s84", "s85", "s86", "s87", "s88", "s89", "s90", "s91", "memory");
#pragma unroll
        for (int r = 0; r < 4; ++r) {
          v4u v = {e[r * 4], e[r * 4 + 1], e[r * 4 + 2], e[r * 4 + 3]};
          if (bad2) v = v4u{0x7fc00000u, 0x7fc00000u, 0x7fc00000u, 0x7fc00000u};
          __builtin_amdgcn_raw_buffer_store_b128(v, ro, row_off + col_off + g * (NS * 4) + (unsigned)(i * 16 + r) * (GN * 4), 0, AUX_SYS);
        }
      }
    }
  }
  if (tid == 0) {
    const unsigned t = __hip_atomic_fetch_add((unsigned*)(myws + WS_EXIT), 1u, __ATOMIC_RELAXED, __HIP_MEMORY_SCOPE_AGENT);
    if ((t + 1u) % NGRID == 0u) st_agent((unsigned*)(myws + WS_CALL), call);
  }
}

}

extern "C" void kernel_launch(int rank, int nranks, ncclComm_t comm,
                              hipStream_t stream,
                              void* const* d_in, const int* in_sizes, int n_in,
                              void* d_out, int out_size,
                              const int* in_dtypes, int out_dtype,
                              void* const* d_out_all,
                              void* const* d_ws_all, size_t ws_bytes) {
  (void)comm; (void)in_dtypes; (void)out_dtype; (void)d_out;
  if (nranks != WORLD || n_in != 4 || ws_bytes < WS_NEED || (unsigned)rank >= (unsigned)WORLD ||
      in_sizes[0] != GM * KS || in_sizes[1] != KS * GN || out_size != MB * GN) {
    fprintf(stderr, "kernel_launch: unexpected problem shape\n");
    return;
  }
  Params p;
  p.x = (const uint8_t*)d_in[0];
  p.w = (const uint8_t*)d_in[1];
  p.sx = (const float*)d_in[2];
  p.sw = (const float*)d_in[3];
  for (int r = 0; r < WORLD; ++r) {
    p.out[r] = (float*)d_out_all[r];
    p.ws[r] = (uint8_t*)d_ws_all[r];
  }
  p.rank = rank;
  {
    const unsigned group1 = 0xB2u;
    unsigned lo = 0, hi = 4;
    p.label_map = 0;
    for (int r = 0; r < WORLD; ++r) {
      const bool same = ((group1 >> r) & 1u) == ((group1 >> rank) & 1u);
      p.label_map |= (unsigned)r << (4u * (same ? lo++ : hi++));
    }
  }
  gemm_rs_kernel<<<dim3(NGRID), dim3(NTHREADS), 0, stream>>>(p);
  HIP_CHECK(hipGetLastError());
}
```

```cpp
#include <hip/hip_runtime.h>
#include <rccl/rccl.h>
#include <cstdio>
#include <cstdint>

#define HIP_CHECK(expr) do { hipError_t _e = (expr); if (_e != hipSuccess) { \
    fprintf(stderr, "HIP error %s at %s:%d\n", hipGetErrorString(_e), __FILE__, __LINE__); \
    return; } } while (0)

namespace {

constexpr int WORLD = 8;
constexpr int GM = 4096, GK = 4096, GN = 16384;
constexpr int KS = GK / WORLD;
constexpr int NS = GN / WORLD;
constexpr int MB = GM / WORLD;
constexpr int BM = 128, BN = 256, BK = 128;
constexpr int NKB = GK / BK;
constexpr int A_STAGE = BM * BK;
constexpr int B_STAGE = BN * BK;
constexpr int STAGE = A_STAGE + B_STAGE;
constexpr int NSTAGE = 3;
constexpr int NTHREADS = 512;
constexpr int NGRID = 256;
constexpr int NPUSH = 8;
constexpr unsigned NTOK = 2;
constexpr int REL_AT = 4;

constexpr size_t WS_CALL = 0;
constexpr size_t WS_EXIT = 128;
constexpr size_t WS_TMO = 256;
constexpr size_t WS_QTICK = 1024;
constexpr size_t WS_QDONE = 2048;
constexpr size_t WS_CLAIM = 3072;
constexpr size_t WS_XFLAG = 4096;
constexpr size_t WS_WFLAG = 8192;
constexpr size_t WS_PQ = 12288;
constexpr size_t WS_OFLAG = 16384;
constexpr size_t WS_STAGE = 1u << 20;
constexpr size_t XS_BYTES = (size_t)GM * GK;
constexpr size_t WT_BYTES = (size_t)GK * NS;
constexpr size_t PAR_BYTES = XS_BYTES + WT_BYTES;
constexpr unsigned PK_SLOT = 17408;
constexpr size_t WS_PK = WS_STAGE + 2 * PAR_BYTES;
constexpr size_t PK_BYTES = (size_t)WORLD * 32 * 8 * PK_SLOT;
constexpr size_t WS_NEED = WS_PK + PK_BYTES;
constexpr size_t OUT_BYTES = (size_t)MB * GN * 4;

constexpr unsigned long long DEADLINE_TICKS = 50000000ull;

typedef int v4i __attribute__((ext_vector_type(4)));
typedef unsigned v4u __attribute__((ext_vector_type(4)));
typedef int v8i __attribute__((ext_vector_type(8)));
typedef float v4f __attribute__((ext_vector_type(4)));

struct Params {
  const uint8_t* x;
  const uint8_t* w;
  const float* sx;
  const float* sw;
  float* out[WORLD];
  uint8_t* ws[WORLD];
  int rank;
  unsigned label_map;
};

#define AUX_SYS 17

__device__ __forceinline__ __amdgpu_buffer_rsrc_t make_rsrc(void* p, unsigned bytes) {
  return __builtin_amdgcn_make_buffer_rsrc(p, 0, bytes, 0x00020000);
}
#define bufld16(rs, l, voff, soff, aux) \
  __builtin_amdgcn_raw_ptr_buffer_load_lds(rs, (__attribute__((address_space(3))) void*)(l), 16, voff, soff, 0, aux)
__device__ __forceinline__ unsigned ld_sys(const unsigned* p) {
  return __hip_atomic_load(p, __ATOMIC_RELAXED, __HIP_MEMORY_SCOPE_SYSTEM);
}
__device__ __forceinline__ unsigned ld_agent(const unsigned* p) {
  return __hip_atomic_load(p, __ATOMIC_RELAXED, __HIP_MEMORY_SCOPE_AGENT);
}
__device__ __forceinline__ void st_sys(unsigned* p, unsigned v) {
  __hip_atomic_store(p, v, __ATOMIC_RELAXED, __HIP_MEMORY_SCOPE_SYSTEM);
}
__device__ __forceinline__ void st_agent(unsigned* p, unsigned v) {
  __hip_atomic_store(p, v, __ATOMIC_RELAXED, __HIP_MEMORY_SCOPE_AGENT);
}

__device__ __forceinline__ void wait_flag(const unsigned* flag, unsigned call, unsigned* tmo) {
  const unsigned long long t0 = __builtin_amdgcn_s_memrealtime();
  for (unsigned it = 1;; ++it) {
    unsigned v = call;
    if (flag) v = ld_sys(flag);
    if (__all(v >= call)) break;
    if ((it & 7u) == 0u && (ld_agent(tmo) == call || __builtin_amdgcn_s_memrealtime() - t0 > DEADLINE_TICKS)) {
      st_agent(tmo, call);
      break;
    }
    __builtin_amdgcn_s_sleep(2);
  }
  asm volatile("s_waitcnt vmcnt(0)" ::: "memory");
}

__global__ void __launch_bounds__(NTHREADS) gemm_rs_kernel(Params p) {
  __shared__ __attribute__((aligned(16))) uint8_t smem[NSTAGE * STAGE + 64];

  const int tid = threadIdx.x;
  const int lane = tid & 63;
  const int wave = __builtin_amdgcn_readfirstlane(tid >> 6);
  const int rank = p.rank;
  uint8_t* const myws = p.ws[rank];
  unsigned* const tmo = (unsigned*)(myws + WS_TMO);
  const unsigned call_v = ld_agent((const unsigned*)(myws + WS_CALL));

  auto* const lds_slot = (__attribute__((address_space(3))) unsigned*)(smem + NSTAGE * STAGE + 16);
  if (tid == 0) {
    const unsigned xcc = __builtin_amdgcn_s_getreg(20   | (0 << 6) | ((4 - 1) << 11)) & 7u;
    unsigned slot = 0;
    for (unsigned k = 0; k < 8u; ++k) {
      const unsigned lab = (p.label_map >> (((xcc + k) & 7u) * 4u)) & 7u;
      const unsigned i = __hip_atomic_fetch_add((unsigned*)(myws + WS_CLAIM + lab * 128), 1u, __ATOMIC_RELAXED, __HIP_MEMORY_SCOPE_AGENT);
      slot = lab | (i << 3);
      if (i < 32u) break;
    }
    *lds_slot = slot;
  }
  asm volatile("s_waitcnt lgkmcnt(0)" ::: "memory");
  __builtin_amdgcn_s_barrier();
  asm volatile("" ::: "memory");
  const unsigned slot_v = __builtin_amdgcn_readfirstlane(*lds_slot);
  const unsigned g = slot_v & 7u;
  const int idx_in = (int)(slot_v >> 3) & 31;

  const int c16 = lane & 15, gq = lane >> 4;
  const bool remote = g != (unsigned)rank;

  unsigned* const qdone = (unsigned*)(myws + WS_QDONE + g * 128);
  auto link_wait = [&](unsigned ticket, unsigned call) {
    const unsigned long long t0 = __builtin_amdgcn_s_memrealtime();
    for (unsigned it = 1; (int)(ld_agent(qdone) + NTOK - ticket) <= 0; ++it) {
      if ((it & 15u) == 0u && (ld_agent(tmo) == call || __builtin_amdgcn_s_memrealtime() - t0 > DEADLINE_TICKS)) {
        st_agent(tmo, call);
        break;
      }
      __builtin_amdgcn_s_sleep(2);
    }
  };
  auto link_release = [&]() {
    asm volatile("" ::: "memory");
    __builtin_amdgcn_s_barrier();
    if (tid == 0 && remote) __hip_atomic_fetch_add(qdone, 1u, __ATOMIC_RELAXED, __HIP_MEMORY_SCOPE_AGENT);
    asm volatile("" ::: "memory");
  };

  if (idx_in < NPUSH) {
    uint8_t* const dws = p.ws[g];
    auto* const lds_next = (__attribute__((address_space(3))) unsigned*)smem;
    const int wn = wave & 3;
    const uint8_t* const wsrc = p.w + (size_t)((wave >> 2) * 128 + (idx_in & 1) * 256 + gq * 16) * GN + (size_t)g * NS + (idx_in >> 1) * BN + wn * 64 + c16 * 4;
    const unsigned wdst = (unsigned)XS_BYTES + (unsigned)(idx_in >> 1) * (NKB * B_STAGE) + (unsigned)(rank * 4 + (idx_in & 1) * 2 + (wave >> 2)) * B_STAGE +
                          (unsigned)wn * 4 * 2048 + lane * 16;
    const unsigned xsrc = (unsigned)(wave * 16 + c16) * KS + gq * 16;
    const unsigned xdst = (unsigned)(rank * 4) * A_STAGE + (unsigned)wave * 8192 + lane * 16;
    unsigned* const pq = (unsigned*)(myws + WS_PQ + g * 128);

    auto load_x = [&](v4u (&b)[8], int tmq) {
      const __amdgpu_buffer_rsrc_t rx = make_rsrc((void*)p.x, (unsigned)tmq < 32u ? (unsigned)(GM * KS) : 0u);
#pragma unroll
      for (int u = 0; u < 8; ++u) b[u] = __builtin_amdgcn_raw_buffer_load_b128(rx, xsrc, (tmq & 31) * (BM * KS) + u * 64, 0);
    };
    auto load_w = [&](unsigned (&b)[32], int half) {
#pragma unroll
      for (int t = 0; t < 32; ++t) b[t] = *(const unsigned*)(wsrc + half * (4 * BN) + (size_t)((t >> 4) * 64 + (t & 15)) * GN);
    };
    v4u xa[8], xb[8];
    unsigned wa[32], wb[32];
    load_x(xa, idx_in * 4);
    asm volatile("" ::: "memory");
    const unsigned call = __builtin_amdgcn_readfirstlane(call_v) + 1u;
    const __amdgpu_buffer_rsrc_t rs = make_rsrc(dws + WS_STAGE + (size_t)(call & 1u) * PAR_BYTES, (unsigned)PAR_BYTES);
    const unsigned pq_base = (call - 1u) * (24u + NPUSH);

    auto store_x = [&](const v4u (&b)[8], int tmq) {
      const unsigned dst = xdst + (unsigned)tmq * (NKB * A_STAGE);
#pragma unroll
      for (int u = 0; u < 8; ++u) {
        __builtin_amdgcn_raw_buffer_store_b128(b[u], rs, dst + u * 1024, 0, AUX_SYS);
        if (u == REL_AT) link_release();
      }
    };
    auto store_w = [&](const unsigned (&b)[32], int half) {
      const unsigned dst = wdst + (unsigned)half * (4u * NKB * B_STAGE);
#pragma unroll
      for (int qh = 0; qh < 2; ++qh) {
        v4u o[4];
#pragma unroll
        for (int u = 0; u < 4; ++u) {
          const unsigned* v = b + qh * 16 + 4 * u;
          const unsigned t0 = __builtin_amdgcn_perm(v[1], v[0], 0x05010400u);
          const unsigned t1 = __builtin_amdgcn_perm(v[1], v[0], 0x07030602u);
          const unsigned t2 = __builtin_amdgcn_perm(v[3], v[2], 0x05010400u);
          const unsigned t3 = __builtin_amdgcn_perm(v[3], v[2], 0x07030602u);
          o[0][u] = __builtin_amdgcn_perm(t2, t0, 0x05040100u);
          o[1][u] = __builtin_amdgcn_perm(t2, t0, 0x07060302u);
          o[2][u] = __builtin_amdgcn_perm(t3, t1, 0x05040100u);
          o[3][u] = __builtin_amdgcn_perm(t3, t1, 0x07060302u);
        }
#pragma unroll
        for (int j = 0; j < 4; ++j) {
          __builtin_amdgcn_raw_buffer_store_b128(o[j], rs, dst + j * 2048 + qh * 1024, 0, AUX_SYS);
          if (qh * 4 + j == REL_AT) link_release();
        }
      }
    };
    auto xflag = [&](int tmq) { return (unsigned*)(dws + WS_XFLAG + (size_t)tmq * 128) + rank; };
    auto wflag = [&](int half) { return (unsigned*)(dws + WS_WFLAG + (size_t)(half * 8 + idx_in) * 128) + rank; };
#define PUSH_FENCE() asm volatile("" ::: "memory")
#define PUSH_PUBLISH(N, flagptr) do { \
      asm volatile("s_waitcnt vmcnt(" #N ")" ::: "memory"); \
      __builtin_amdgcn_s_barrier(); PUSH_FENCE(); \
      if (tid == 0) st_sys(flagptr, call); \
      PUSH_FENCE(); } while (0)
#define PUSH_DRAW() do { if (tid == 0) { \
      const unsigned d = __hip_atomic_fetch_add(pq, 1u, __ATOMIC_RELAXED, __HIP_MEMORY_SCOPE_AGENT) - pq_base; \
      *lds_next = d < 24u ? (d & 7u) * 4u + 1u + (d >> 3) : 32u; } } while (0)
    const unsigned tk_base = (call - 1u) * 80u;
    auto xticket = [&](int tmq) { return tk_base + ((tmq & 3) == 0 ? (unsigned)(tmq >> 2) : 16u + (unsigned)(tmq & 3) * 8u + (unsigned)(tmq >> 2)); };
    auto go = [&](unsigned ticket) {
      if (tid == 0 && remote) link_wait(ticket, call);
      asm volatile("s_waitcnt lgkmcnt(0)" ::: "memory");
      __builtin_amdgcn_s_barrier();
      PUSH_FENCE();
    };
    if (idx_in >= (int)NTOK) go(xticket(idx_in * 4));
    store_x(xa, idx_in * 4); PUSH_FENCE();
    load_w(wa, 0); PUSH_FENCE();
    PUSH_PUBLISH(32, xflag(idx_in * 4));
    PUSH_DRAW();
    go(tk_base + 8u + idx_in);
    int ta = __builtin_amdgcn_readfirstlane(*lds_next);
    store_w(wa, 0); PUSH_FENCE();
    load_w(wb, 1); PUSH_FENCE();
    load_x(xa, ta); PUSH_FENCE();
    PUSH_PUBLISH(40, wflag(0));
    if (ta < 32) PUSH_DRAW();
    go(tk_base + 16u + idx_in);
    int tb = ta < 32 ? __builtin_amdgcn_readfirstlane(*lds_next) : 32;
    store_w(wb, 1); PUSH_FENCE();
    load_x(xb, tb); PUSH_FENCE();
    PUSH_PUBLISH(8, wflag(1));
    for (;;) {
      if (ta >= 32) break;
      {
        const int cur = ta;
        if (tb < 32) PUSH_DRAW();
        go(xticket(cur));
        ta = tb < 32 ? __builtin_amdgcn_readfirstlane(*lds_next) : 32;
        store_x(xa, cur); PUSH_FENCE();
        load_x(xa, ta); PUSH_FENCE();
        PUSH_PUBLISH(8, xflag(cur));
      }
      if (tb >= 32) break;
      {
        const int cur = tb;
        if (ta < 32) PUSH_DRAW();
        go(xticket(cur));
        tb = ta < 32 ? __builtin_amdgcn_readfirstlane(*lds_next) : 32;
        store_x(xb, cur); PUSH_FENCE();
        load_x(xb, tb); PUSH_FENCE();
        PUSH_PUBLISH(8, xflag(cur));
      }
    }
  }
  const unsigned call = __builtin_amdgcn_readfirstlane(call_v) + 1u;
  const size_t stage_off = WS_STAGE + (size_t)(call & 1u) * PAR_BYTES;

  const int idx = (idx_in - NPUSH) & 31;
  const int mt = idx >> 3, nt = idx & 7;
  const int tm = (int)g * 4 + mt;
  auto* const lds_sel = (__attribute__((address_space(3))) int*)(smem + NSTAGE * STAGE);
  const unsigned* myflag = nullptr;
  if (lane < 8) myflag = (const unsigned*)(myws + WS_XFLAG + (size_t)tm * 128) + lane;
  else if (lane < 24) myflag = (const unsigned*)(myws + WS_WFLAG + (size_t)(nt * 2 + ((lane - 8) >> 3)) * 128) + (lane & 7);
  unsigned rem = 0xffu, ready = 0u;
  auto select = [&]() {
    const unsigned long long t0 = __builtin_amdgcn_s_memrealtime();
    while ((ready & rem) == 0u) {
      unsigned v = call;
      if (myflag) v = ld_sys(myflag);
      const unsigned m = (unsigned)__builtin_amdgcn_ballot_w64(v >= call);
      ready = m & (m >> 8) & (m >> 16) & 0xffu;
      if (ready & rem) break;
      if (ld_agent(tmo) == call || __builtin_amdgcn_s_memrealtime() - t0 > DEADLINE_TICKS) {
        if (lane == 0) st_agent(tmo, call);
        ready = 0xffu;
        break;
      }
      __builtin_amdgcn_s_sleep(4);
    }
    const unsigned cand = ready & rem;
    const int sel = (cand >> rank) & 1u ? rank : __builtin_ctz(cand);
    rem &= ~(1u << sel);
    if (lane == 0) *lds_sel = sel;
  };
  if (wave == 0) {
    select();
    __builtin_amdgcn_fence(__ATOMIC_ACQUIRE, "");
    asm volatile("s_waitcnt vmcnt(0) lgkmcnt(0)" ::: "memory");
  }
  __builtin_amdgcn_s_barrier();
  asm volatile("" ::: "memory");
  int src = __builtin_amdgcn_readfirstlane(*lds_sel);

  const int wm = wave >> 2, wn = wave & 3;
  v4f acc[4][4];
#pragma unroll
  for (int i = 0; i < 4; ++i)
#pragma unroll
    for (int j = 0; j < 4; ++j) acc[i][j] = v4f{0.f, 0.f, 0.f, 0.f};
  const float s = p.sx[0] * p.sw[0];
  unsigned ticket = 0;

  {
    uint8_t* const sbase = myws + stage_off;
    const unsigned va = (unsigned)tm * (NKB * A_STAGE) + wave * 8192 + lane * 16;
    const unsigned vb = (unsigned)XS_BYTES + (unsigned)nt * (NKB * B_STAGE) + wave * 4096 + lane * 16;
    auto issue1 = [&](unsigned nrec, int kb, int boff, int c) {
      const __amdgpu_buffer_rsrc_t rs = make_rsrc(sbase, nrec);
      if (c < 2) bufld16(rs, smem + boff + wave * 2048 + c * 1024, va, (kb >> 2) * (4 * A_STAGE) + (kb & 3) * 2048 + c * 1024, 0);
      else bufld16(rs, smem + boff + A_STAGE + wave * 4096 + (c - 2) * 1024, vb, kb * B_STAGE + (c - 2) * 1024, 0);
    };
#pragma unroll
    for (int c = 0; c < 6; ++c) issue1((unsigned)PAR_BYTES, src * 4, 0, c);
#pragma unroll
    for (int c = 0; c < 6; ++c) issue1((unsigned)PAR_BYTES, src * 4 + 1, STAGE, c);
    int boff = 0;
    const int a_rd = wm * 4 * 2048 + lane * 16;
    const int b_rd = A_STAGE + wn * 4 * 2048 + lane * 16;
    auto step = [&](unsigned nrec, int pkb) {
      int nb = boff + 2 * STAGE;
      if (nb >= NSTAGE * STAGE) nb -= NSTAGE * STAGE;
      const uint8_t* sa = smem + boff + a_rd;
      const uint8_t* sb = smem + boff + b_rd;
      v8i af[4], bf[4];
      auto rd = [&](const uint8_t* q) {
        const v4i lo = *(const v4i*)q, hi = *(const v4i*)(q + 1024);
        return __builtin_shufflevector(lo, hi, 0, 1, 2, 3, 4, 5, 6, 7);
      };
      af[0] = rd(sa);
#pragma unroll
      for (int j = 0; j < 4; ++j) bf[j] = rd(sb + j * 2048);
#pragma unroll
      for (int i = 1; i < 4; ++i) af[i] = rd(sa + i * 2048);
      __builtin_amdgcn_sched_barrier(0);
#pragma unroll
      for (int i = 0; i < 4; ++i) {
#pragma unroll
        for (int j = 0; j < 4; ++j) {
          acc[i][j] = __builtin_amdgcn_mfma_scale_f32_16x16x128_f8f6f4(af[i], bf[j], acc[i][j], 0  , 1  , 0, 0, 0, 0);
          if ((j & 1) && i < 3) {
            __builtin_amdgcn_sched_barrier(0);
            issue1(nrec, pkb, nb, i * 2 + (j >> 1));
            __builtin_amdgcn_sched_barrier(0);
          }
        }
      }
      __builtin_amdgcn_s_waitcnt(0xC07F);
      boff += STAGE;
      if (boff >= NSTAGE * STAGE) boff = 0;
    };
#define GEMM_SYNC() do { asm volatile("s_waitcnt vmcnt(6)" ::: "memory"); __builtin_amdgcn_s_barrier(); asm volatile("" ::: "memory"); } while (0)
    __builtin_amdgcn_s_waitcnt(0xC07F);
    for (int slot = 0; slot < WORLD; ++slot) {
      const bool more = slot + 1 < WORLD;
      const unsigned nrec = more ? (unsigned)PAR_BYTES : 0u;
      GEMM_SYNC();
      step((unsigned)PAR_BYTES, src * 4 + 2);
      GEMM_SYNC();
      step((unsigned)PAR_BYTES, src * 4 + 3);
      if (more && wave == 0) {
        select();
        if (slot == WORLD - 2) {
          if (lane == 0 && g != (unsigned)rank)
            ticket = __hip_atomic_fetch_add((unsigned*)(myws + WS_QTICK + g * 128), 1u, __ATOMIC_RELAXED, __HIP_MEMORY_SCOPE_AGENT);
        }
        asm volatile("s_waitcnt lgkmcnt(0)" ::: "memory");
      }
      GEMM_SYNC();
      const int nsrc = more ? __builtin_amdgcn_readfirstlane(*lds_sel) : 0;
      __builtin_amdgcn_s_waitcnt(0xC07F);
      step(nrec, nsrc * 4);
      GEMM_SYNC();
      step(nrec, nsrc * 4 + 1);
      src = nsrc;
    }
  }
  const unsigned tmo_v = ld_agent(tmo);
  asm volatile("s_waitcnt vmcnt(1)" ::: "memory");

  const __amdgpu_buffer_rsrc_t ro = make_rsrc(p.out[rank], (unsigned)OUT_BYTES);
  const unsigned row_off = (unsigned)(mt * BM + wm * 64 + gq * 4) * GN * 4u;
  const unsigned col_off = (unsigned)(nt * BN + wn * 64 + c16 * 4) * 4u;

  if (g == (unsigned)rank) {
    const bool bad = tmo_v == call;
#pragma unroll
    for (int i = 0; i < 4; ++i)
#pragma unroll
      for (int r = 0; r < 4; ++r) {
        v4f v;
#pragma unroll
        for (int j = 0; j < 4; ++j) {
          const float y = acc[i][j][r] * s;
          v[j] = (y <= 0.f) ? 0.f : y;
          if (bad) v[j] = __builtin_nanf("");
        }
        __builtin_amdgcn_raw_buffer_store_b128(__builtin_bit_cast(v4u, v), ro,
                                               row_off + col_off + (unsigned)(rank * NS * 4) + (unsigned)(i * 16 + r) * (GN * 4), 0, AUX_SYS);
      }
  } else {
    asm volatile("s_waitcnt lgkmcnt(0)" ::: "memory");
    __builtin_amdgcn_s_barrier();
    asm volatile("" ::: "memory");
    uint8_t* const pk = smem + wave * PK_SLOT;
    const unsigned pk_lds = (unsigned)(unsigned long)((__attribute__((address_space(3))) uint8_t*)smem) + wave * PK_SLOT;
    unsigned nxt = pk_lds + 512u;
    unsigned bits[2] = {0u, 0u};
#pragma unroll
    for (int i = 0; i < 4; ++i)
#pragma unroll
      for (int r = 0; r < 4; ++r) {
        float y0, y1, y2, y3;
        unsigned t0, t1, t2, t3, n, b1, b2, b3;
        asm volatile(
            "v_mul_f32 %[y0], %[s], %[a0]\n\t"
            "v_mul_f32 %[y1], %[s], %[a1]\n\t"
            "v_mul_f32 %[y2], %[s], %[a2]\n\t"
            "v_mul_f32 %[y3], %[s], %[a3]\n\t"
            "v_cmp_nge_f32_e64 s[84:85], 0, %[y0]\n\t"
            "v_cmp_nge_f32_e64 s[86:87], 0, %[y1]\n\t"
            "v_cmp_nge_f32_e64 s[88:89], 0, %[y2]\n\t"
            "v_cmp_nge_f32_e64 s[90:91], 0, %[y3]\n\t"
            "s_bcnt1_i32_b64 %[n], s[84:85]\n\t"
            "s_lshl2_add_u32 %[b1], %[n], %[nx]\n\t"
            "s_bcnt1_i32_b64 %[n], s[86:87]\n\t"
            "s_lshl2_add_u32 %[b2], %[n], %[b1]\n\t"
            "s_bcnt1_i32_b64 %[n], s[88:89]\n\t"
            "s_lshl2_add_u32 %[b3], %[n], %[b2]\n\t"
            "v_mbcnt_lo_u32_b32 %[t0], s84, 0\n\t"
            "v_mbcnt_lo_u32_b32 %[t1], s86, 0\n\t"
            "v_mbcnt_lo_u32_b32 %[t2], s88, 0\n\t"
            "v_mbcnt_lo_u32_b32 %[t3], s90, 0\n\t"
            "v_mbcnt_hi_u32_b32 %[t0], s85, %[t0]\n\t"
            "v_mbcnt_hi_u32_b32 %[t1], s87, %[t1]\n\t"
            "v_mbcnt_hi_u32_b32 %[t2], s89, %[t2]\n\t"
            "v_mbcnt_hi_u32_b32 %[t3], s91, %[t3]\n\t"
            "v_lshl_add_u32 %[t0], %[t0], 2, %[nx]\n\t"
            "v_lshl_add_u32 %[t1], %[t1], 2, %[b1]\n\t"
            "v_lshl_add_u32 %[t2], %[t2], 2, %[b2]\n\t"
            "v_lshl_add_u32 %[t3], %[t3], 2, %[b3]\n\t"
            "s_bcnt1_i32_b64 %[n], s[90:91]\n\t"
            "s_lshl2_add_u32 %[nx], %[n], %[b3]\n\t"
            "v_addc_co_u32_e64 %[bt], vcc, %[bt], %[bt], s[84:85]\n\t"
            "v_addc_co_u32_e64 %[bt], vcc, %[bt], %[bt], s[86:87]\n\t"
            "v_addc_co_u32_e64 %[bt], vcc, %[bt], %[bt], s[88:89]\n\t"
            "v_addc_co_u32_e64 %[bt], vcc, %[bt], %[bt], s[90:91]\n\t"
            "s_mov_b64 exec, s[84:85]\n\t"
            "ds_write_b32 %[t0], %[y0]\n\t"
            "s_mov_b64 exec, s[86:87]\n\t"
            "ds_write_b32 %[t1], %[y1]\n\t"
            "s_mov_b64 exec, s[88:89]\n\t"
            "ds_write_b32 %[t2], %[y2]\n\t"
            "s_mov_b64 exec, s[90:91]\n\t"
            "ds_write_b32 %[t3], %[y3]\n\t"
            "s_mov_b64 exec, -1"
            : [y0] "=&v"(y0), [y1] "=&v"(y1), [y2] "=&v"(y2), [y3] "=&v"(y3), [t0] "=&v"(t0), [t1] "=&v"(t1), [t2] "=&v"(t2),
              [t3] "=&v"(t3), [n] "=&s"(n), [b1] "=&s"(b1), [b2] "=&s"(b2), [b3] "=&s"(b3), [nx] "+s"(nxt), [bt] "+v"(bits[i >> 1])
            : [s] "s"(s), [a0] "v"(acc[i][0][r]), [a1] "v"(acc[i][1][r]), [a2] "v"(acc[i][2][r]), [a3] "v"(acc[i][3][r])
            : "vcc", "scc", "s84", "s85", "s86", "s87", "s88", "s89", "s90", "s91", "memory");
      }
    unsigned base = nxt - pk_lds;
    if (tmo_v == call) {
      bits[0] = bits[1] = 0xffffffffu;
      base = 512u + 16384u;
      for (int o = 0; o < 64; ++o) *(float*)(pk + 512 + o * 256 + lane * 4) = __builtin_nanf("");
    }
    *(uint2*)(pk + lane * 8) = make_uint2(bits[0], bits[1]);
    const unsigned bytes = (base + 63u) & ~63u;

    if (tid == 0) link_wait((call - 1u) * 80u + 48u + (ticket - (call - 1u) * 32u), call);
    asm volatile("s_waitcnt lgkmcnt(0)" ::: "memory");
    __builtin_amdgcn_s_barrier();
    asm volatile("" ::: "memory");
    {
      const __amdgpu_buffer_rsrc_t rp = make_rsrc(p.ws[g] + WS_PK, (unsigned)PK_BYTES);
      const unsigned slot = (unsigned)((rank * 32 + idx_in) * 8 + wave) * PK_SLOT;
#pragma unroll
      for (int o = 0; o < 17; ++o) {
        const unsigned off = o * 1024u + lane * 16u;
        if (o * 1024u < bytes) {
          const v4u v = *(const v4u*)(pk + off);
          if (off < bytes) __builtin_amdgcn_raw_buffer_store_b128(v, rp, slot + off, 0, AUX_SYS);
        }
        if (o == REL_AT) link_release();
      }
    }
    asm volatile("s_waitcnt vmcnt(0)" ::: "memory");
    __builtin_amdgcn_s_barrier();
    if (tid == 0) st_sys((unsigned*)(p.ws[g] + WS_OFLAG + ((size_t)rank * 32 + idx_in) * 128), call);

    if (wave == 0) {
      const unsigned* f = lane == 0 ? (const unsigned*)(myws + WS_OFLAG + ((size_t)g * 32 + idx_in) * 128) : nullptr;
      wait_flag(f, call, tmo);
    }
    asm volatile("" ::: "memory");
    __builtin_amdgcn_s_barrier();
    asm volatile("" ::: "memory");
    const bool bad2 = ld_agent(tmo) == call;
    {
      const __amdgpu_buffer_rsrc_t rq = make_rsrc(myws + WS_PK, (unsigned)PK_BYTES);
      const unsigned src = (unsigned)((g * 32 + idx_in) * 8 + wave) * PK_SLOT;
#pragma unroll
      for (int o = 0; o < 17; ++o) bufld16(rq, pk + o * 1024, lane * 16, src + o * 1024, AUX_SYS);
      asm volatile("s_waitcnt vmcnt(0)" ::: "memory");
      const uint2 mm = *(const uint2*)(pk + lane * 8);
      *(float*)(pk + 17152u + lane * 4u) = 0.f;
      const unsigned zero_at = pk_lds + 17152u + lane * 4u;
      unsigned rb = pk_lds + 512u;
      unsigned mb[2] = {mm.x, mm.y};
#define DEC4(v0, v1, v2, v3) \
            "v_add_co_u32_e64 %[mb], s[84:85], %[mb], %[mb]\n\t" \
            "v_add_co_u32_e64 %[mb], s[86:87], %[mb], %[mb]\n\t" \
            "v_add_co_u32_e64 %[mb], s[88:89], %[mb], %[mb]\n\t" \
            "v_add_co_u32_e64 %[mb], s[90:91], %[mb], %[mb]\n\t" \
            "s_bcnt1_i32_b64 %[n], s[84:85]\n\t" \
            "s_lshl2_add_u32 %[b1], %[n], %[rb]\n\t" \
            "s_bcnt1_i32_b64 %[n], s[86:87]\n\t" \
            "s_lshl2_add_u32 %[b2], %[n], %[b1]\n\t" \
            "s_bcnt1_i32_b64 %[n], s[88:89]\n\t" \
            "s_lshl2_add_u32 %[b3], %[n], %[b2]\n\t" \
            "v_mbcnt_lo_u32_b32 " v0 ", s84, 0\n\t" \
            "v_mbcnt_lo_u32_b32 " v1 ", s86, 0\n\t" \
            "v_mbcnt_lo_u32_b32 " v2 ", s88, 0\n\t" \
            "v_mbcnt_lo_u32_b32 " v3 ", s90, 0\n\t" \
            "v_mbcnt_hi_u32_b32 " v0 ", s85, " v0 "\n\t" \
            "v_mbcnt_hi_u32_b32 " v1 ", s87, " v1 "\n\t" \
            "v_mbcnt_hi_u32_b32 " v2 ", s89, " v2 "\n\t" \
            "v_mbcnt_hi_u32_b32 " v3 ", s91, " v3 "\n\t" \
            "v_lshl_add_u32 " v0 ", " v0 ", 2, %[rb]\n\t" \
            "v_lshl_add_u32 " v1 ", " v1 ", 2, %[b1]\n\t" \
            "v_lshl_add_u32 " v2 ", " v2 ", 2, %[b2]\n\t" \
            "v_lshl_add_u32 " v3 ", " v3 ", 2, %[b3]\n\t" \
            "s_bcnt1_i32_b64 %[n], s[90:91]\n\t" \
            "s_lshl2_add_u32 %[rb], %[n], %[b3]\n\t" \
            "v_cndmask_b32_e64 " v0 ", %[z], " v0 ", s[84:85]\n\t" \
            "v_cndmask_b32_e64 " v1 ", %[z], " v1 ", s[86:87]\n\t" \
            "v_cndmask_b32_e64 " v2 ", %[z], " v2 ", s[88:89]\n\t" \
            "v_cndmask_b32_e64 " v3 ", %[z], " v3 ", s[90:91]\n\t" \
            "ds_read_b32 " v0 ", " v0 "\n\t" \
            "ds_read_b32 " v1 ", " v1 "\n\t" \
            "ds_read_b32 " v2 ", " v2 "\n\t" \
            "ds_read_b32 " v3 ", " v3 "\n\t"
#pragma unroll
      for (int i = 0; i < 4; ++i) {
        unsigned e[16], n, b1, b2, b3;
        asm volatile(
            DEC4("%[e0]", "%[e1]", "%[e2]", "%[e3]")
            DEC4("%[e4]", "%[e5]", "%[e6]", "%[e7]")
            DEC4("%[e8]", "%[e9]", "%[e10]", "%[e11]")
            DEC4("%[e12]", "%[e13]", "%[e14]", "%[e15]")
            "s_waitcnt lgkmcnt(0)"
            : [e0] "=&v"(e[0]), [e1] "=&v"(e[1]), [e2] "=&v"(e[2]), [e3] "=&v"(e[3]), [e4] "=&v"(e[4]), [e5] "=&v"(e[5]),
              [e6] "=&v"(e[6]), [e7] "=&v"(e[7]), [e8] "=&v"(e[8]), [e9] "=&v"(e[9]), [e10] "=&v"(e[10]), [e11] "=&v"(e[11]),
              [e12] "=&v"(e[12]), [e13] "=&v"(e[13]), [e14] "=&v"(e[14]), [e15] "=&v"(e[15]),
              [n] "=&s"(n), [b1] "=&s"(b1), [b2] "=&s"(b2), [b3] "=&s"(b3), [rb] "+s"(rb), [mb] "+v"(mb[i >> 1])
            : [z] "v"(zero_at)
            : "scc", "s84", "s85", "s86", "s87", "s88", "s89", "s90", "s91", "memory");
#pragma unroll
        for (int r = 0; r < 4; ++r) {
          v4u v = {e[r * 4], e[r * 4 + 1], e[r * 4 + 2], e[r * 4 + 3]};
          if (bad2) v = v4u{0x7fc00000u, 0x7fc00000u, 0x7fc00000u, 0x7fc00000u};
          __builtin_amdgcn_raw_buffer_store_b128(v, ro, row_off + col_off + g * (NS * 4) + (unsigned)(i * 16 + r) * (GN * 4), 0, AUX_SYS);
        }
      }
    }
  }
  if (tid == 0) {
    const unsigned t = __hip_atomic_fetch_add((unsigned*)(myws + WS_EXIT), 1u, __ATOMIC_RELAXED, __HIP_MEMORY_SCOPE_AGENT);
    if ((t + 1u) % NGRID == 0u) {
      for (int l = 0; l < WORLD; ++l) st_agent((unsigned*)(myws + WS_CLAIM + l * 128), 0u);
      st_agent((unsigned*)(myws + WS_CALL), call);
    }
  }
}

}

extern "C" void kernel_launch(int rank, int nranks, ncclComm_t comm,
                              hipStream_t stream,
                              void* const* d_in, const int* in_sizes, int n_in,
                              void* d_out, int out_size,
                              const int* in_dtypes, int out_dtype,
                              void* const* d_out_all,
                              void* const* d_ws_all, size_t ws_bytes) {
  (void)comm; (void)in_dtypes; (void)out_dtype; (void)d_out;
  if (nranks != WORLD || n_in != 4 || ws_bytes < WS_NEED || (unsigned)rank >= (unsigned)WORLD ||
      in_sizes[0] != GM * KS || in_sizes[1] != KS * GN || out_size != MB * GN) {
    fprintf(stderr, "kernel_launch: unexpected problem shape\n");
    return;
  }
  Params p;
  p.x = (const uint8_t*)d_in[0];
  p.w = (const uint8_t*)d_in[1];
  p.sx = (const float*)d_in[2];
  p.sw = (const float*)d_in[3];
  for (int r = 0; r < WORLD; ++r) {
    p.out[r] = (float*)d_out_all[r];
    p.ws[r] = (uint8_t*)d_ws_all[r];
  }
  p.rank = rank;
  {
    const unsigned group1 = 0xB2u;
    unsigned lo = 0, hi = 4;
    p.label_map = 0;
    for (int r = 0; r < WORLD; ++r) {
      const bool same = ((group1 >> r) & 1u) == ((group1 >> rank) & 1u);
      p.label_map |= (unsigned)r << (4u * (same ? lo++ : hi++));
    }
  }
  gemm_rs_kernel<<<dim3(NGRID), dim3(NTHREADS), 0, stream>>>(p);
  HIP_CHECK(hipGetLastError());
}
```

```cpp
#include <hip/hip_runtime.h>
#include <rccl/rccl.h>
#include <cstdio>
#include <cstdint>

#define HIP_CHECK(expr) do { hipError_t _e = (expr); if (_e != hipSuccess) { \
    fprintf(stderr, "HIP error %s at %s:%d\n", hipGetErrorString(_e), __FILE__, __LINE__); \
    return; } } while (0)

namespace {

constexpr int WORLD = 8;
constexpr int GM = 4096, GK = 4096, GN = 16384;
constexpr int KS = GK / WORLD;
constexpr int NS = GN / WORLD;
constexpr int MB = GM / WORLD;
constexpr int BM = 128, BN = 256, BK = 128;
constexpr int NKB = GK / BK;
constexpr int A_STAGE = BM * BK;
constexpr int B_STAGE = BN * BK;
constexpr int STAGE = A_STAGE + B_STAGE;
constexpr int NSTAGE = 3;
constexpr int NTHREADS = 512;
constexpr int NGRID = 256;
constexpr int NPUSH = 8;
constexpr unsigned NTOK = 2;
constexpr int REL_AT = 4;

constexpr size_t WS_CALL = 0;
constexpr size_t WS_EXIT = 128;
constexpr size_t WS_TMO = 256;
constexpr size_t WS_QTICK = 1024;
constexpr size_t WS_QDONE = 2048;
constexpr size_t WS_CLAIM = 3072;
constexpr size_t WS_XFLAG = 4096;
constexpr size_t WS_WFLAG = 8192;
constexpr size_t WS_PQ = 12288;
constexpr size_t WS_OFLAG = 16384;
constexpr size_t WS_STAGE = 1u << 20;
constexpr size_t XS_BYTES = (size_t)GM * GK;
constexpr size_t WT_BYTES = (size_t)GK * NS;
constexpr size_t PAR_BYTES = XS_BYTES + WT_BYTES;
constexpr unsigned PK_SLOT = 17408;
constexpr size_t WS_PK = WS_STAGE + 2 * PAR_BYTES;
constexpr size_t PK_BYTES = (size_t)WORLD * 32 * 8 * PK_SLOT;
constexpr size_t WS_NEED = WS_PK + PK_BYTES;
constexpr size_t OUT_BYTES = (size_t)MB * GN * 4;

constexpr unsigned long long DEADLINE_TICKS = 50000000ull;

typedef int v4i __attribute__((ext_vector_type(4)));
typedef unsigned v4u __attribute__((ext_vector_type(4)));
typedef int v8i __attribute__((ext_vector_type(8)));
typedef float v4f __attribute__((ext_vector_type(4)));

struct Params {
  const uint8_t* x;
  const uint8_t* w;
  const float* sx;
  const float* sw;
  float* out[WORLD];
  uint8_t* ws[WORLD];
  int rank;
  unsigned label_map;
};

#define AUX_SYS 17

__device__ __forceinline__ __amdgpu_buffer_rsrc_t make_rsrc(void* p, unsigned bytes) {
  return __builtin_amdgcn_make_buffer_rsrc(p, 0, bytes, 0x00020000);
}
#define bufld16(rs, l, voff, soff, aux) \
  __builtin_amdgcn_raw_ptr_buffer_load_lds(rs, (__attribute__((address_space(3))) void*)(l), 16, voff, soff, 0, aux)
__device__ __forceinline__ unsigned ld_sys(const unsigned* p) {
  return __hip_atomic_load(p, __ATOMIC_RELAXED, __HIP_MEMORY_SCOPE_SYSTEM);
}
__device__ __forceinline__ unsigned ld_agent(const unsigned* p) {
  return __hip_atomic_load(p, __ATOMIC_RELAXED, __HIP_MEMORY_SCOPE_AGENT);
}
__device__ __forceinline__ void st_sys(unsigned* p, unsigned v) {
  __hip_atomic_store(p, v, __ATOMIC_RELAXED, __HIP_MEMORY_SCOPE_SYSTEM);
}
__device__ __forceinline__ void st_agent(unsigned* p, unsigned v) {
  __hip_atomic_store(p, v, __ATOMIC_RELAXED, __HIP_MEMORY_SCOPE_AGENT);
}

__device__ __forceinline__ void wait_flag(const unsigned* flag, unsigned call, unsigned* tmo) {
  const unsigned long long t0 = __builtin_amdgcn_s_memrealtime();
  for (unsigned it = 1;; ++it) {
    unsigned v = call;
    if (flag) v = ld_sys(flag);
    if (__all(v >= call)) break;
    if ((it & 7u) == 0u && (ld_agent(tmo) == call || __builtin_amdgcn_s_memrealtime() - t0 > DEADLINE_TICKS)) {
      st_agent(tmo, call);
      break;
    }
    __builtin_amdgcn_s_sleep(2);
  }
  asm volatile("s_waitcnt vmcnt(0)" ::: "memory");
}

__global__ void __launch_bounds__(NTHREADS) gemm_rs_kernel(Params p) {
  __shared__ __attribute__((aligned(16))) uint8_t smem[NSTAGE * STAGE + 64];

  const int tid = threadIdx.x;
  const int lane = tid & 63;
  const int wave = __builtin_amdgcn_readfirstlane(tid >> 6);
  const int rank = p.rank;
  uint8_t* const myws = p.ws[rank];
  unsigned* const tmo = (unsigned*)(myws + WS_TMO);
  const unsigned call_v = ld_agent((const unsigned*)(myws + WS_CALL));

  const int c16 = lane & 15, gq = lane >> 4;

  const int idx_guess = blockIdx.x >> 3;
  const unsigned xsrc = (unsigned)(wave * 16 + c16) * KS + gq * 16;
  auto load_x = [&](v4u (&b)[8], int tmq) {
    const __amdgpu_buffer_rsrc_t rx = make_rsrc((void*)p.x, (unsigned)tmq < 32u ? (unsigned)(GM * KS) : 0u);
#pragma unroll
    for (int u = 0; u < 8; ++u) b[u] = __builtin_amdgcn_raw_buffer_load_b128(rx, xsrc, (tmq & 31) * (BM * KS) + u * 64, 0);
  };
  v4u xa[8];
  load_x(xa, idx_guess < NPUSH ? idx_guess * 4 : 32);
  asm volatile("" ::: "memory");

  auto* const lds_slot = (__attribute__((address_space(3))) unsigned*)(smem + NSTAGE * STAGE + 16);
  if (tid == 0) {
    const unsigned xcc = __builtin_amdgcn_s_getreg(20   | (0 << 6) | ((4 - 1) << 11)) & 7u;
    unsigned slot = 0;
    for (unsigned k = 0; k < 256u; ++k) {
      const unsigned lab = (p.label_map >> (((xcc + (k >> 5)) & 7u) * 4u)) & 7u;
      const unsigned i = ((unsigned)idx_guess + k) & 31u;
      const unsigned old = __hip_atomic_fetch_or((unsigned*)(myws + WS_CLAIM + lab * 128), 1u << i, __ATOMIC_RELAXED, __HIP_MEMORY_SCOPE_AGENT);
      slot = lab | (i << 3);
      if (!((old >> i) & 1u)) break;
    }
    *lds_slot = slot;
  }
  asm volatile("s_waitcnt lgkmcnt(0)" ::: "memory");
  __builtin_amdgcn_s_barrier();
  asm volatile("" ::: "memory");
  const unsigned slot_v = __builtin_amdgcn_readfirstlane(*lds_slot);
  const unsigned g = slot_v & 7u;
  const int idx_in = (int)(slot_v >> 3) & 31;

  const bool remote = g != (unsigned)rank;

  unsigned* const qdone = (unsigned*)(myws + WS_QDONE + g * 128);
  auto link_wait = [&](unsigned ticket, unsigned call) {
    const unsigned long long t0 = __builtin_amdgcn_s_memrealtime();
    for (unsigned it = 1; (int)(ld_agent(qdone) + NTOK - ticket) <= 0; ++it) {
      if ((it & 15u) == 0u && (ld_agent(tmo) == call || __builtin_amdgcn_s_memrealtime() - t0 > DEADLINE_TICKS)) {
        st_agent(tmo, call);
        break;
      }
      __builtin_amdgcn_s_sleep(2);
    }
  };
  auto link_release = [&]() {
    asm volatile("" ::: "memory");
    __builtin_amdgcn_s_barrier();
    if (tid == 0 && remote) __hip_atomic_fetch_add(qdone, 1u, __ATOMIC_RELAXED, __HIP_MEMORY_SCOPE_AGENT);
    asm volatile("" ::: "memory");
  };

  if (idx_in < NPUSH) {
    uint8_t* const dws = p.ws[g];
    auto* const lds_next = (__attribute__((address_space(3))) unsigned*)smem;
    const int wn = wave & 3;
    const uint8_t* const wsrc = p.w + (size_t)((wave >> 2) * 128 + (idx_in & 1) * 256 + gq * 16) * GN + (size_t)g * NS + (idx_in >> 1) * BN + wn * 64 + c16 * 4;
    const unsigned wdst = (unsigned)XS_BYTES + (unsigned)(idx_in >> 1) * (NKB * B_STAGE) + (unsigned)(rank * 4 + (idx_in & 1) * 2 + (wave >> 2)) * B_STAGE +
                          (unsigned)wn * 4 * 2048 + lane * 16;
    const unsigned xdst = (unsigned)(rank * 4) * A_STAGE + (unsigned)wave * 8192 + lane * 16;
    unsigned* const pq = (unsigned*)(myws + WS_PQ + g * 128);

    auto load_w = [&](unsigned (&b)[32], int half) {
#pragma unroll
      for (int t = 0; t < 32; ++t) b[t] = *(const unsigned*)(wsrc + half * (4 * BN) + (size_t)((t >> 4) * 64 + (t & 15)) * GN);
    };
    v4u xb[8];
    unsigned wa[32], wb[32];
    if (idx_in != idx_guess) load_x(xa, idx_in * 4);
    asm volatile("" ::: "memory");
    const unsigned call = __builtin_amdgcn_readfirstlane(call_v) + 1u;
    const __amdgpu_buffer_rsrc_t rs = make_rsrc(dws + WS_STAGE + (size_t)(call & 1u) * PAR_BYTES, (unsigned)PAR_BYTES);
    const unsigned pq_base = (call - 1u) * (24u + NPUSH);

    auto store_x = [&](const v4u (&b)[8], int tmq) {
      const unsigned dst = xdst + (unsigned)tmq * (NKB * A_STAGE);
#pragma unroll
      for (int u = 0; u < 8; ++u) {
        __builtin_amdgcn_raw_buffer_store_b128(b[u], rs, dst + u * 1024, 0, AUX_SYS);
        if (u == REL_AT) link_release();
      }
    };
    auto store_w = [&](const unsigned (&b)[32], int half) {
      const unsigned dst = wdst + (unsigned)half * (4u * NKB * B_STAGE);
#pragma unroll
      for (int qh = 0; qh < 2; ++qh) {
        v4u o[4];
#pragma unroll
        for (int u = 0; u < 4; ++u) {
          const unsigned* v = b + qh * 16 + 4 * u;
          const unsigned t0 = __builtin_amdgcn_perm(v[1], v[0], 0x05010400u);
          const unsigned t1 = __builtin_amdgcn_perm(v[1], v[0], 0x07030602u);
          const unsigned t2 = __builtin_amdgcn_perm(v[3], v[2], 0x05010400u);
          const unsigned t3 = __builtin_amdgcn_perm(v[3], v[2], 0x07030602u);
          o[0][u] = __builtin_amdgcn_perm(t2, t0, 0x05040100u);
          o[1][u] = __builtin_amdgcn_perm(t2, t0, 0x07060302u);
          o[2][u] = __builtin_amdgcn_perm(t3, t1, 0x05040100u);
          o[3][u] = __builtin_amdgcn_perm(t3, t1, 0x07060302u);
        }
#pragma unroll
        for (int j = 0; j < 4; ++j) {
          __builtin_amdgcn_raw_buffer_store_b128(o[j], rs, dst + j * 2048 + qh * 1024, 0, AUX_SYS);
          if (qh * 4 + j == REL_AT) link_release();
        }
      }
    };
    auto xflag = [&](int tmq) { return (unsigned*)(dws + WS_XFLAG + (size_t)tmq * 128) + rank; };
    auto wflag = [&](int half) { return (unsigned*)(dws + WS_WFLAG + (size_t)(half * 8 + idx_in) * 128) + rank; };
#define PUSH_FENCE() asm volatile("" ::: "memory")
#define PUSH_PUBLISH(N, flagptr) do { \
      asm volatile("s_waitcnt vmcnt(" #N ")" ::: "memory"); \
      __builtin_amdgcn_s_barrier(); PUSH_FENCE(); \
      if (tid == 0) st_sys(flagptr, call); \
      PUSH_FENCE(); } while (0)
#define PUSH_DRAW() do { if (tid == 0) { \
      const unsigned d = __hip_atomic_fetch_add(pq, 1u, __ATOMIC_RELAXED, __HIP_MEMORY_SCOPE_AGENT) - pq_base; \
      *lds_next = d < 24u ? (d & 7u) * 4u + 1u + (d >> 3) : 32u; } } while (0)
    const unsigned tk_base = (call - 1u) * 80u;
    auto xticket = [&](int tmq) { return tk_base + ((tmq & 3) == 0 ? (unsigned)(tmq >> 2) : 16u + (unsigned)(tmq & 3) * 8u + (unsigned)(tmq >> 2)); };
    auto go = [&](unsigned ticket) {
      if (tid == 0 && remote) link_wait(ticket, call);
      asm volatile("s_waitcnt lgkmcnt(0)" ::: "memory");
      __builtin_amdgcn_s_barrier();
      PUSH_FENCE();
    };
    if (idx_in >= (int)NTOK) go(xticket(idx_in * 4));
    store_x(xa, idx_in * 4); PUSH_FENCE();
    load_w(wa, 0); PUSH_FENCE();
    PUSH_PUBLISH(32, xflag(idx_in * 4));
    PUSH_DRAW();
    go(tk_base + 8u + idx_in);
    int ta = __builtin_amdgcn_readfirstlane(*lds_next);
    store_w(wa, 0); PUSH_FENCE();
    load_w(wb, 1); PUSH_FENCE();
    load_x(xa, ta); PUSH_FENCE();
    PUSH_PUBLISH(40, wflag(0));
    if (ta < 32) PUSH_DRAW();
    go(tk_base + 16u + idx_in);
    int tb = ta < 32 ? __builtin_amdgcn_readfirstlane(*lds_next) : 32;
    store_w(wb, 1); PUSH_FENCE();
    load_x(xb, tb); PUSH_FENCE();
    PUSH_PUBLISH(8, wflag(1));
    for (;;) {
      if (ta >= 32) break;
      {
        const int cur = ta;
        if (tb < 32) PUSH_DRAW();
        go(xticket(cur));
        ta = tb < 32 ? __builtin_amdgcn_readfirstlane(*lds_next) : 32;
        store_x(xa, cur); PUSH_FENCE();
        load_x(xa, ta); PUSH_FENCE();
        PUSH_PUBLISH(8, xflag(cur));
      }
      if (tb >= 32) break;
      {
        const int cur = tb;
        if (ta < 32) PUSH_DRAW();
        go(xticket(cur));
        tb = ta < 32 ? __builtin_amdgcn_readfirstlane(*lds_next) : 32;
        store_x(xb, cur); PUSH_FENCE();
        load_x(xb, tb); PUSH_FENCE();
        PUSH_PUBLISH(8, xflag(cur));
      }
    }
  }
  const unsigned call = __builtin_amdgcn_readfirstlane(call_v) + 1u;
  const size_t stage_off = WS_STAGE + (size_t)(call & 1u) * PAR_BYTES;

  const int idx = (idx_in - NPUSH) & 31;
  const int mt = idx >> 3, nt = idx & 7;
  const int tm = (int)g * 4 + mt;
  auto* const lds_sel = (__attribute__((address_space(3))) int*)(smem + NSTAGE * STAGE);
  const unsigned* myflag = nullptr;
  if (lane < 8) myflag = (const unsigned*)(myws + WS_XFLAG + (size_t)tm * 128) + lane;
  else if (lane < 24) myflag = (const unsigned*)(myws + WS_WFLAG + (size_t)(nt * 2 + ((lane - 8) >> 3)) * 128) + (lane & 7);
  unsigned rem = 0xffu, ready = 0u;
  auto select = [&]() {
    const unsigned long long t0 = __builtin_amdgcn_s_memrealtime();
    while ((ready & rem) == 0u) {
      unsigned v = call;
      if (myflag) v = ld_sys(myflag);
      const unsigned m = (unsigned)__builtin_amdgcn_ballot_w64(v >= call);
      ready = m & (m >> 8) & (m >> 16) & 0xffu;
      if (ready & rem) break;
      if (ld_agent(tmo) == call || __builtin_amdgcn_s_memrealtime() - t0 > DEADLINE_TICKS) {
        if (lane == 0) st_agent(tmo, call);
        ready = 0xffu;
        break;
      }
      __builtin_amdgcn_s_sleep(4);
    }
    const unsigned cand = ready & rem;
    const int sel = (cand >> rank) & 1u ? rank : __builtin_ctz(cand);
    rem &= ~(1u << sel);
    if (lane == 0) *lds_sel = sel;
  };
  if (wave == 0) {
    select();
    __builtin_amdgcn_fence(__ATOMIC_ACQUIRE, "");
    asm volatile("s_waitcnt vmcnt(0) lgkmcnt(0)" ::: "memory");
  }
  __builtin_amdgcn_s_barrier();
  asm volatile("" ::: "memory");
  int src = __builtin_amdgcn_readfirstlane(*lds_sel);

  const int wm = wave >> 2, wn = wave & 3;
  v4f acc[4][4];
#pragma unroll
  for (int i = 0; i < 4; ++i)
#pragma unroll
    for (int j = 0; j < 4; ++j) acc[i][j] = v4f{0.f, 0.f, 0.f, 0.f};
  const float s = p.sx[0] * p.sw[0];
  unsigned ticket = 0;

  {
    uint8_t* const sbase = myws + stage_off;
    const unsigned va = (unsigned)tm * (NKB * A_STAGE) + wave * 8192 + lane * 16;
    const unsigned vb = (unsigned)XS_BYTES + (unsigned)nt * (NKB * B_STAGE) + wave * 4096 + lane * 16;
    auto issue1 = [&](unsigned nrec, int kb, int boff, int c) {
      const __amdgpu_buffer_rsrc_t rs = make_rsrc(sbase, nrec);
      if (c < 2) bufld16(rs, smem + boff + wave * 2048 + c * 1024, va, (kb >> 2) * (4 * A_STAGE) + (kb & 3) * 2048 + c * 1024, 0);
      else bufld16(rs, smem + boff + A_STAGE + wave * 4096 + (c - 2) * 1024, vb, kb * B_STAGE + (c - 2) * 1024, 0);
    };
#pragma unroll
    for (int c = 0; c < 6; ++c) issue1((unsigned)PAR_BYTES, src * 4, 0, c);
#pragma unroll
    for (int c = 0; c < 6; ++c) issue1((unsigned)PAR_BYTES, src * 4 + 1, STAGE, c);
    int boff = 0;
    const int a_rd = wm * 4 * 2048 + lane * 16;
    const int b_rd = A_STAGE + wn * 4 * 2048 + lane * 16;
    auto step = [&](unsigned nrec, int pkb) {
      int nb = boff + 2 * STAGE;
      if (nb >= NSTAGE * STAGE) nb -= NSTAGE * STAGE;
      const uint8_t* sa = smem + boff + a_rd;
      const uint8_t* sb = smem + boff + b_rd;
      v8i af[4], bf[4];
      auto rd = [&](const uint8_t* q) {
        const v4i lo = *(const v4i*)q, hi = *(const v4i*)(q + 1024);
        return __builtin_shufflevector(lo, hi, 0, 1, 2, 3, 4, 5, 6, 7);
      };
      af[0] = rd(sa);
#pragma unroll
      for (int j = 0; j < 4; ++j) bf[j] = rd(sb + j * 2048);
#pragma unroll
      for (int i = 1; i < 4; ++i) af[i] = rd(sa + i * 2048);
      __builtin_amdgcn_sched_barrier(0);
#pragma unroll
      for (int i = 0; i < 4; ++i) {
#pragma unroll
        for (int j = 0; j < 4; ++j) {
          acc[i][j] = __builtin_amdgcn_mfma_scale_f32_16x16x128_f8f6f4(af[i], bf[j], acc[i][j], 0  , 1  , 0, 0, 0, 0);
          if ((j & 1) && i < 3) {
            __builtin_amdgcn_sched_barrier(0);
            issue1(nrec, pkb, nb, i * 2 + (j >> 1));
            __builtin_amdgcn_sched_barrier(0);
          }
        }
      }
      __builtin_amdgcn_s_waitcnt(0xC07F);
      boff += STAGE;
      if (boff >= NSTAGE * STAGE) boff = 0;
    };
#define GEMM_SYNC() do { asm volatile("s_waitcnt vmcnt(6)" ::: "memory"); __builtin_amdgcn_s_barrier(); asm volatile("" ::: "memory"); } while (0)
    __builtin_amdgcn_s_waitcnt(0xC07F);
    for (int slot = 0; slot < WORLD; ++slot) {
      const bool more = slot + 1 < WORLD;
      const unsigned nrec = more ? (unsigned)PAR_BYTES : 0u;
      GEMM_SYNC();
      step((unsigned)PAR_BYTES, src * 4 + 2);
      GEMM_SYNC();
      step((unsigned)PAR_BYTES, src * 4 + 3);
      if (more && wave == 0) {
        select();
        if (slot == WORLD - 2) {
          if (lane == 0 && g != (unsigned)rank)
            ticket = __hip_atomic_fetch_add((unsigned*)(myws + WS_QTICK + g * 128), 1u, __ATOMIC_RELAXED, __HIP_MEMORY_SCOPE_AGENT);
        }
        asm volatile("s_waitcnt lgkmcnt(0)" ::: "memory");
      }
      GEMM_SYNC();
      const int nsrc = more ? __builtin_amdgcn_readfirstlane(*lds_sel) : 0;
      __builtin_amdgcn_s_waitcnt(0xC07F);
      step(nrec, nsrc * 4);
      GEMM_SYNC();
      step(nrec, nsrc * 4 + 1);
      src = nsrc;
    }
  }
  const unsigned tmo_v = ld_agent(tmo);
  asm volatile("s_waitcnt vmcnt(1)" ::: "memory");

  const __amdgpu_buffer_rsrc_t ro = make_rsrc(p.out[rank], (unsigned)OUT_BYTES);
  const unsigned row_off = (unsigned)(mt * BM + wm * 64 + gq * 4) * GN * 4u;
  const unsigned col_off = (unsigned)(nt * BN + wn * 64 + c16 * 4) * 4u;

  if (g == (unsigned)rank) {
    const bool bad = tmo_v == call;
#pragma unroll
    for (int i = 0; i < 4; ++i)
#pragma unroll
      for (int r = 0; r < 4; ++r) {
        v4f v;
#pragma unroll
        for (int j = 0; j < 4; ++j) {
          const float y = acc[i][j][r] * s;
          v[j] = (y <= 0.f) ? 0.f : y;
          if (bad) v[j] = __builtin_nanf("");
        }
        __builtin_amdgcn_raw_buffer_store_b128(__builtin_bit_cast(v4u, v), ro,
                                               row_off + col_off + (unsigned)(rank * NS * 4) + (unsigned)(i * 16 + r) * (GN * 4), 0, AUX_SYS);
      }
  } else {
    asm volatile("s_waitcnt lgkmcnt(0)" ::: "memory");
    __builtin_amdgcn_s_barrier();
    asm volatile("" ::: "memory");
    uint8_t* const pk = smem + wave * PK_SLOT;
    const unsigned pk_lds = (unsigned)(unsigned long)((__attribute__((address_space(3))) uint8_t*)smem) + wave * PK_SLOT;
    unsigned nxt = pk_lds + 512u;
    unsigned bits[2] = {0u, 0u};
#pragma unroll
    for (int i = 0; i < 4; ++i)
#pragma unroll
      for (int r = 0; r < 4; ++r) {
        float y0, y1, y2, y3;
        unsigned t0, t1, t2, t3, n, b1, b2, b3;
        asm volatile(
            "v_mul_f32 %[y0], %[s], %[a0]\n\t"
            "v_mul_f32 %[y1], %[s], %[a1]\n\t"
            "v_mul_f32 %[y2], %[s], %[a2]\n\t"
            "v_mul_f32 %[y3], %[s], %[a3]\n\t"
            "v_cmp_nge_f32_e64 s[84:85], 0, %[y0]\n\t"
            "v_cmp_nge_f32_e64 s[86:87], 0, %[y1]\n\t"
            "v_cmp_nge_f32_e64 s[88:89], 0, %[y2]\n\t"
            "v_cmp_nge_f32_e64 s[90:91], 0, %[y3]\n\t"
            "s_bcnt1_i32_b64 %[n], s[84:85]\n\t"
            "s_lshl2_add_u32 %[b1], %[n], %[nx]\n\t"
            "s_bcnt1_i32_b64 %[n], s[86:87]\n\t"
            "s_lshl2_add_u32 %[b2], %[n], %[b1]\n\t"
            "s_bcnt1_i32_b64 %[n], s[88:89]\n\t"
            "s_lshl2_add_u32 %[b3], %[n], %[b2]\n\t"
            "v_mbcnt_lo_u32_b32 %[t0], s84, 0\n\t"
            "v_mbcnt_lo_u32_b32 %[t1], s86, 0\n\t"
            "v_mbcnt_lo_u32_b32 %[t2], s88, 0\n\t"
            "v_mbcnt_lo_u32_b32 %[t3], s90, 0\n\t"
            "v_mbcnt_hi_u32_b32 %[t0], s85, %[t0]\n\t"
            "v_mbcnt_hi_u32_b32 %[t1], s87, %[t1]\n\t"
            "v_mbcnt_hi_u32_b32 %[t2], s89, %[t2]\n\t"
            "v_mbcnt_hi_u32_b32 %[t3], s91, %[t3]\n\t"
            "v_lshl_add_u32 %[t0], %[t0], 2, %[nx]\n\t"
            "v_lshl_add_u32 %[t1], %[t1], 2, %[b1]\n\t"
            "v_lshl_add_u32 %[t2], %[t2], 2, %[b2]\n\t"
            "v_lshl_add_u32 %[t3], %[t3], 2, %[b3]\n\t"
            "s_bcnt1_i32_b64 %[n], s[90:91]\n\t"
            "s_lshl2_add_u32 %[nx], %[n], %[b3]\n\t"
            "v_addc_co_u32_e64 %[bt], vcc, %[bt], %[bt], s[84:85]\n\t"
            "v_addc_co_u32_e64 %[bt], vcc, %[bt], %[bt], s[86:87]\n\t"
            "v_addc_co_u32_e64 %[bt], vcc, %[bt], %[bt], s[88:89]\n\t"
            "v_addc_co_u32_e64 %[bt], vcc, %[bt], %[bt], s[90:91]\n\t"
            "s_mov_b64 exec, s[84:85]\n\t"
            "ds_write_b32 %[t0], %[y0]\n\t"
            "s_mov_b64 exec, s[86:87]\n\t"
            "ds_write_b32 %[t1], %[y1]\n\t"
            "s_mov_b64 exec, s[88:89]\n\t"
            "ds_write_b32 %[t2], %[y2]\n\t"
            "s_mov_b64 exec, s[90:91]\n\t"
            "ds_write_b32 %[t3], %[y3]\n\t"
            "s_mov_b64 exec, -1"
            : [y0] "=&v"(y0), [y1] "=&v"(y1), [y2] "=&v"(y2), [y3] "=&v"(y3), [t0] "=&v"(t0), [t1] "=&v"(t1), [t2] "=&v"(t2),
              [t3] "=&v"(t3), [n] "=&s"(n), [b1] "=&s"(b1), [b2] "=&s"(b2), [b3] "=&s"(b3), [nx] "+s"(nxt), [bt] "+v"(bits[i >> 1])
            : [s] "s"(s), [a0] "v"(acc[i][0][r]), [a1] "v"(acc[i][1][r]), [a2] "v"(acc[i][2][r]), [a3] "v"(acc[i][3][r])
            : "vcc", "scc", "s84", "s85", "s86", "s87", "s88", "s89", "s90", "s91", "memory");
      }
    unsigned base = nxt - pk_lds;
    if (tmo_v == call) {
      bits[0] = bits[1] = 0xffffffffu;
      base = 512u + 16384u;
      for (int o = 0; o < 64; ++o) *(float*)(pk + 512 + o * 256 + lane * 4) = __builtin_nanf("");
    }
    *(uint2*)(pk + lane * 8) = make_uint2(bits[0], bits[1]);
    const unsigned bytes = (base + 63u) & ~63u;

    if (tid == 0) link_wait((call - 1u) * 80u + 48u + (ticket - (call - 1u) * 32u), call);
    asm volatile("s_waitcnt lgkmcnt(0)" ::: "memory");
    __builtin_amdgcn_s_barrier();
    asm volatile("" ::: "memory");
    {
      const __amdgpu_buffer_rsrc_t rp = make_rsrc(p.ws[g] + WS_PK, (unsigned)PK_BYTES);
      const unsigned slot = (unsigned)((rank * 32 + idx_in) * 8 + wave) * PK_SLOT;
#pragma unroll
      for (int o = 0; o < 17; ++o) {
        const unsigned off = o * 1024u + lane * 16u;
        if (o * 1024u < bytes) {
          const v4u v = *(const v4u*)(pk + off);
          if (off < bytes) __builtin_amdgcn_raw_buffer_store_b128(v, rp, slot + off, 0, AUX_SYS);
        }
        if (o == REL_AT) link_release();
      }
    }
    asm volatile("s_waitcnt vmcnt(0)" ::: "memory");
    __builtin_amdgcn_s_barrier();
    if (tid == 0) st_sys((unsigned*)(p.ws[g] + WS_OFLAG + ((size_t)rank * 32 + idx_in) * 128), call);

    if (wave == 0) {
      const unsigned* f = lane == 0 ? (const unsigned*)(myws + WS_OFLAG + ((size_t)g * 32 + idx_in) * 128) : nullptr;
      wait_flag(f, call, tmo);
    }
    asm volatile("" ::: "memory");
    __builtin_amdgcn_s_barrier();
    asm volatile("" ::: "memory");
    const bool bad2 = ld_agent(tmo) == call;
    {
      const __amdgpu_buffer_rsrc_t rq = make_rsrc(myws + WS_PK, (unsigned)PK_BYTES);
      const unsigned src = (unsigned)((g * 32 + idx_in) * 8 + wave) * PK_SLOT;
#pragma unroll
      for (int o = 0; o < 17; ++o) bufld16(rq, pk + o * 1024, lane * 16, src + o * 1024, AUX_SYS);
      asm volatile("s_waitcnt vmcnt(0)" ::: "memory");
      const uint2 mm = *(const uint2*)(pk + lane * 8);
      *(float*)(pk + 17152u + lane * 4u) = 0.f;
      const unsigned zero_at = pk_lds + 17152u + lane * 4u;
      unsigned rb = pk_lds + 512u;
      unsigned mb[2] = {mm.x, mm.y};
#define DEC4(v0, v1, v2, v3) \
            "v_add_co_u32_e64 %[mb], s[84:85], %[mb], %[mb]\n\t" \
            "v_add_co_u32_e64 %[mb], s[86:87], %[mb], %[mb]\n\t" \
            "v_add_co_u32_e64 %[mb], s[88:89], %[mb], %[mb]\n\t" \
            "v_add_co_u32_e64 %[mb], s[90:91], %[mb], %[mb]\n\t" \
            "s_bcnt1_i32_b64 %[n], s[84:85]\n\t" \
            "s_lshl2_add_u32 %[b1], %[n], %[rb]\n\t" \
            "s_bcnt1_i32_b64 %[n], s[86:87]\n\t" \
            "s_lshl2_add_u32 %[b2], %[n], %[b1]\n\t" \
            "s_bcnt1_i32_b64 %[n], s[88:89]\n\t" \
            "s_lshl2_add_u32 %[b3], %[n], %[b2]\n\t" \
            "v_mbcnt_lo_u32_b32 " v0 ", s84, 0\n\t" \
            "v_mbcnt_lo_u32_b32 " v1 ", s86, 0\n\t" \
            "v_mbcnt_lo_u32_b32 " v2 ", s88, 0\n\t" \
            "v_mbcnt_lo_u32_b32 " v3 ", s90, 0\n\t" \
            "v_mbcnt_hi_u32_b32 " v0 ", s85, " v0 "\n\t" \
            "v_mbcnt_hi_u32_b32 " v1 ", s87, " v1 "\n\t" \
            "v_mbcnt_hi_u32_b32 " v2 ", s89, " v2 "\n\t" \
            "v_mbcnt_hi_u32_b32 " v3 ", s91, " v3 "\n\t" \
            "v_lshl_add_u32 " v0 ", " v0 ", 2, %[rb]\n\t" \
            "v_lshl_add_u32 " v1 ", " v1 ", 2, %[b1]\n\t" \
            "v_lshl_add_u32 " v2 ", " v2 ", 2, %[b2]\n\t" \
            "v_lshl_add_u32 " v3 ", " v3 ", 2, %[b3]\n\t" \
            "s_bcnt1_i32_b64 %[n], s[90:91]\n\t" \
            "s_lshl2_add_u32 %[rb], %[n], %[b3]\n\t" \
            "v_cndmask_b32_e64 " v0 ", %[z], " v0 ", s[84:85]\n\t" \
            "v_cndmask_b32_e64 " v1 ", %[z], " v1 ", s[86:87]\n\t" \
            "v_cndmask_b32_e64 " v2 ", %[z], " v2 ", s[88:89]\n\t" \
            "v_cndmask_b32_e64 " v3 ", %[z], " v3 ", s[90:91]\n\t" \
            "ds_read_b32 " v0 ", " v0 "\n\t" \
            "ds_read_b32 " v1 ", " v1 "\n\t" \
            "ds_read_b32 " v2 ", " v2 "\n\t" \
            "ds_read_b32 " v3 ", " v3 "\n\t"
#pragma unroll
      for (int i = 0; i < 4; ++i) {
        unsigned e[16], n, b1, b2, b3;
        asm volatile(
            DEC4("%[e0]", "%[e1]", "%[e2]", "%[e3]")
            DEC4("%[e4]", "%[e5]", "%[e6]", "%[e7]")
            DEC4("%[e8]", "%[e9]", "%[e10]", "%[e11]")
            DEC4("%[e12]", "%[e13]", "%[e14]", "%[e15]")
            "s_waitcnt lgkmcnt(0)"
            : [e0] "=&v"(e[0]), [e1] "=&v"(e[1]), [e2] "=&v"(e[2]), [e3] "=&v"(e[3]), [e4] "=&v"(e[4]), [e5] "=&v"(e[5]),
              [e6] "=&v"(e[6]), [e7] "=&v"(e[7]), [e8] "=&v"(e[8]), [e9] "=&v"(e[9]), [e10] "=&v"(e[10]), [e11] "=&v"(e[11]),
              [e12] "=&v"(e[12]), [e13] "=&v"(e[13]), [e14] "=&v"(e[14]), [e15] "=&v"(e[15]),
              [n] "=&s"(n), [b1] "=&s"(b1), [b2] "=&s"(b2), [b3] "=&s"(b3), [rb] "+s"(rb), [mb] "+v"(mb[i >> 1])
            : [z] "v"(zero_at)
            : "scc", "s84", "s85", "s86", "s87", "s88", "s89", "s90", "s91", "memory");
#pragma unroll
        for (int r = 0; r < 4; ++r) {
          v4u v = {e[r * 4], e[r * 4 + 1], e[r * 4 + 2], e[r * 4 + 3]};
          if (bad2) v = v4u{0x7fc00000u, 0x7fc00000u, 0x7fc00000u, 0x7fc00000u};
          __builtin_amdgcn_raw_buffer_store_b128(v, ro, row_off + col_off + g * (NS * 4) + (unsigned)(i * 16 + r) * (GN * 4), 0, AUX_SYS);
        }
      }
    }
  }
  if (tid == 0) {
    const unsigned t = __hip_atomic_fetch_add((unsigned*)(myws + WS_EXIT), 1u, __ATOMIC_RELAXED, __HIP_MEMORY_SCOPE_AGENT);
    if ((t + 1u) % NGRID == 0u) {
      for (int l = 0; l < WORLD; ++l) st_agent((unsigned*)(myws + WS_CLAIM + l * 128), 0u);
      st_agent((unsigned*)(myws + WS_CALL), call);
    }
  }
}

}

extern "C" void kernel_launch(int rank, int nranks, ncclComm_t comm,
                              hipStream_t stream,
                              void* const* d_in, const int* in_sizes, int n_in,
                              void* d_out, int out_size,
                              const int* in_dtypes, int out_dtype,
                              void* const* d_out_all,
                              void* const* d_ws_all, size_t ws_bytes) {
  (void)comm; (void)in_dtypes; (void)out_dtype; (void)d_out;
  if (nranks != WORLD || n_in != 4 || ws_bytes < WS_NEED || (unsigned)rank >= (unsigned)WORLD ||
      in_sizes[0] != GM * KS || in_sizes[1] != KS * GN || out_size != MB * GN) {
    fprintf(stderr, "kernel_launch: unexpected problem shape\n");
    return;
  }
  Params p;
  p.x = (const uint8_t*)d_in[0];
  p.w = (const uint8_t*)d_in[1];
  p.sx = (const float*)d_in[2];
  p.sw = (const float*)d_in[3];
  for (int r = 0; r < WORLD; ++r) {
    p.out[r] = (float*)d_out_all[r];
    p.ws[r] = (uint8_t*)d_ws_all[r];
  }
  p.rank = rank;
  {
    const unsigned group1 = 0xB2u;
    unsigned lo = 0, hi = 4;
    p.label_map = 0;
    for (int r = 0; r < WORLD; ++r) {
      const bool same = ((group1 >> r) & 1u) == ((group1 >> rank) & 1u);
      p.label_map |= (unsigned)r << (4u * (same ? lo++ : hi++));
    }
  }
  gemm_rs_kernel<<<dim3(NGRID), dim3(NTHREADS), 0, stream>>>(p);
  HIP_CHECK(hipGetLastError());
}
```

```cpp
#include <hip/hip_runtime.h>
#include <rccl/rccl.h>
#include <cstdio>
#include <cstdint>

#define HIP_CHECK(expr) do { hipError_t _e = (expr); if (_e != hipSuccess) { \
    fprintf(stderr, "HIP error %s at %s:%d\n", hipGetErrorString(_e), __FILE__, __LINE__); \
    return; } } while (0)

namespace {

constexpr int WORLD = 8;
constexpr int GM = 4096, GK = 4096, GN = 16384;
constexpr int KS = GK / WORLD;
constexpr int NS = GN / WORLD;
constexpr int MB = GM / WORLD;
constexpr int BM = 128, BN = 256, BK = 128;
constexpr int NKB = GK / BK;
constexpr int A_STAGE = BM * BK;
constexpr int B_STAGE = BN * BK;
constexpr int STAGE = A_STAGE + B_STAGE;
constexpr int NSTAGE = 3;
constexpr int NTHREADS = 512;
constexpr int NGRID = 256;
constexpr int NPUSH = 8;
constexpr unsigned NTOK = 2;
constexpr int REL_AT = 4;

constexpr size_t WS_CALL = 0;
constexpr size_t WS_EXIT = 128;
constexpr size_t WS_TMO = 256;
constexpr size_t WS_QTICK = 1024;
constexpr size_t WS_QDONE = 2048;
constexpr size_t WS_CLAIM = 3072;
constexpr size_t WS_XFLAG = 4096;
constexpr size_t WS_WFLAG = 8192;
constexpr size_t WS_PQ = 12288;
constexpr size_t WS_OFLAG = 16384;
constexpr size_t WS_STAGE = 1u << 20;
constexpr size_t XS_BYTES = (size_t)GM * GK;
constexpr size_t WT_BYTES = (size_t)GK * NS;
constexpr size_t PAR_BYTES = XS_BYTES + WT_BYTES;
constexpr unsigned PK_SLOT = 17408;
constexpr size_t WS_PK = WS_STAGE + 2 * PAR_BYTES;
constexpr size_t PK_BYTES = (size_t)WORLD * 32 * 8 * PK_SLOT;
constexpr size_t WS_NEED = WS_PK + PK_BYTES;
constexpr size_t OUT_BYTES = (size_t)MB * GN * 4;

constexpr unsigned long long DEADLINE_TICKS = 50000000ull;

typedef int v4i __attribute__((ext_vector_type(4)));
typedef unsigned v4u __attribute__((ext_vector_type(4)));
typedef int v8i __attribute__((ext_vector_type(8)));
typedef float v4f __attribute__((ext_vector_type(4)));

struct Params {
  const uint8_t* x;
  const uint8_t* w;
  const float* sx;
  const float* sw;
  float* out[WORLD];
  uint8_t* ws[WORLD];
  int rank;
  unsigned label_map;
};

#define AUX_SYS 17

__device__ __forceinline__ __amdgpu_buffer_rsrc_t make_rsrc(void* p, unsigned bytes) {
  return __builtin_amdgcn_make_buffer_rsrc(p, 0, bytes, 0x00020000);
}
#define bufld16(rs, l, voff, soff, aux) \
  __builtin_amdgcn_raw_ptr_buffer_load_lds(rs, (__attribute__((address_space(3))) void*)(l), 16, voff, soff, 0, aux)
__device__ __forceinline__ unsigned ld_sys(const unsigned* p) {
  return __hip_atomic_load(p, __ATOMIC_RELAXED, __HIP_MEMORY_SCOPE_SYSTEM);
}
__device__ __forceinline__ unsigned ld_agent(const unsigned* p) {
  return __hip_atomic_load(p, __ATOMIC_RELAXED, __HIP_MEMORY_SCOPE_AGENT);
}
__device__ __forceinline__ void st_sys(unsigned* p, unsigned v) {
  __hip_atomic_store(p, v, __ATOMIC_RELAXED, __HIP_MEMORY_SCOPE_SYSTEM);
}
__device__ __forceinline__ void st_agent(unsigned* p, unsigned v) {
  __hip_atomic_store(p, v, __ATOMIC_RELAXED, __HIP_MEMORY_SCOPE_AGENT);
}

__device__ __forceinline__ void wait_flag(const unsigned* flag, unsigned call, unsigned* tmo) {
  const unsigned long long t0 = __builtin_amdgcn_s_memrealtime();
  for (unsigned it = 1;; ++it) {
    unsigned v = call;
    if (flag) v = ld_sys(flag);
    if (__all(v >= call)) break;
    if ((it & 7u) == 0u && (ld_agent(tmo) == call || __builtin_amdgcn_s_memrealtime() - t0 > DEADLINE_TICKS)) {
      st_agent(tmo, call);
      break;
    }
    __builtin_amdgcn_s_sleep(2);
  }
  asm volatile("s_waitcnt vmcnt(0)" ::: "memory");
}

__global__ void __launch_bounds__(NTHREADS) gemm_rs_kernel(Params p) {
  __shared__ __attribute__((aligned(16))) uint8_t smem[NSTAGE * STAGE + 64];

  const int tid = threadIdx.x;
  const int lane = tid & 63;
  const int wave = __builtin_amdgcn_readfirstlane(tid >> 6);
  const int rank = p.rank;
  uint8_t* const myws = p.ws[rank];
  unsigned* const tmo = (unsigned*)(myws + WS_TMO);
  const unsigned call_v = ld_agent((const unsigned*)(myws + WS_CALL));

  const int c16 = lane & 15, gq = lane >> 4;

  const int idx_guess = blockIdx.x >> 3;
  const unsigned xsrc = (unsigned)(wave * 16 + c16) * KS + gq * 16;
  auto load_x = [&](v4u (&b)[8], int tmq) {
    const __amdgpu_buffer_rsrc_t rx = make_rsrc((void*)p.x, (unsigned)tmq < 32u ? (unsigned)(GM * KS) : 0u);
#pragma unroll
    for (int u = 0; u < 8; ++u) b[u] = __builtin_amdgcn_raw_buffer_load_b128(rx, xsrc, (tmq & 31) * (BM * KS) + u * 64, 0);
  };
  v4u xa[8];
  load_x(xa, idx_guess < NPUSH ? idx_guess * 4 : 32);
  asm volatile("" ::: "memory");

  auto* const lds_slot = (__attribute__((address_space(3))) unsigned*)(smem + NSTAGE * STAGE + 16);
  if (tid == 0) {
    const unsigned xcc = __builtin_amdgcn_s_getreg(20   | (0 << 6) | ((4 - 1) << 11)) & 7u;
    unsigned slot = 0;
    for (unsigned k = 0; k < 256u; ++k) {
      const unsigned lab = (p.label_map >> (((xcc + (k >> 5)) & 7u) * 4u)) & 7u;
      const unsigned i = ((unsigned)idx_guess + k) & 31u;
      const unsigned old = __hip_atomic_fetch_or((unsigned*)(myws + WS_CLAIM + lab * 128), 1u << i, __ATOMIC_RELAXED, __HIP_MEMORY_SCOPE_AGENT);
      slot = lab | (i << 3);
      if (!((old >> i) & 1u)) break;
    }
    *lds_slot = slot;
  }
  asm volatile("s_waitcnt lgkmcnt(0)" ::: "memory");
  __builtin_amdgcn_s_barrier();
  asm volatile("" ::: "memory");
  const unsigned slot_v = __builtin_amdgcn_readfirstlane(*lds_slot);
  const unsigned g = slot_v & 7u;
  const int idx_in = (int)(slot_v >> 3) & 31;

  const bool remote = g != (unsigned)rank;

  unsigned* const qdone = (unsigned*)(myws + WS_QDONE + g * 128);
  auto link_wait = [&](unsigned ticket, unsigned call) {
    const unsigned long long t0 = __builtin_amdgcn_s_memrealtime();
    for (unsigned it = 1; (int)(ld_agent(qdone) + NTOK - ticket) <= 0; ++it) {
      if ((it & 15u) == 0u && (ld_agent(tmo) == call || __builtin_amdgcn_s_memrealtime() - t0 > DEADLINE_TICKS)) {
        st_agent(tmo, call);
        break;
      }
      __builtin_amdgcn_s_sleep(2);
    }
  };
  auto link_release = [&]() {
    asm volatile("" ::: "memory");
    __builtin_amdgcn_s_barrier();
    if (tid == 0 && remote) __hip_atomic_fetch_add(qdone, 1u, __ATOMIC_RELAXED, __HIP_MEMORY_SCOPE_AGENT);
    asm volatile("" ::: "memory");
  };

  if (idx_in < NPUSH) {
    uint8_t* const dws = p.ws[g];
    auto* const lds_next = (__attribute__((address_space(3))) unsigned*)smem;
    const int wn = wave & 3;
    const uint8_t* const wsrc = p.w + (size_t)((wave >> 2) * 128 + (idx_in & 1) * 256 + gq * 16) * GN + (size_t)g * NS + (idx_in >> 1) * BN + wn * 64 + c16 * 4;
    const unsigned wdst = (unsigned)XS_BYTES + (unsigned)(idx_in >> 1) * (NKB * B_STAGE) + (unsigned)(rank * 4 + (idx_in & 1) * 2 + (wave >> 2)) * B_STAGE +
                          (unsigned)wn * 4 * 2048 + lane * 16;
    const unsigned xdst = (unsigned)(rank * 4) * A_STAGE + (unsigned)wave * 8192 + lane * 16;
    unsigned* const pq = (unsigned*)(myws + WS_PQ + g * 128);

    auto load_w = [&](unsigned (&b)[32], int half) {
#pragma unroll
      for (int t = 0; t < 32; ++t) b[t] = *(const unsigned*)(wsrc + half * (4 * BN) + (size_t)((t >> 4) * 64 + (t & 15)) * GN);
    };
    v4u xb[8];
    unsigned wa[32], wb[32];
    if (idx_in != idx_guess) load_x(xa, idx_in * 4);
    asm volatile("" ::: "memory");
    const unsigned call = __builtin_amdgcn_readfirstlane(call_v) + 1u;
    const __amdgpu_buffer_rsrc_t rs = make_rsrc(dws + WS_STAGE + (size_t)(call & 1u) * PAR_BYTES, (unsigned)PAR_BYTES);
    const unsigned pq_base = (call - 1u) * (24u + NPUSH);

    auto store_x = [&](const v4u (&b)[8], int tmq) {
      const unsigned dst = xdst + (unsigned)tmq * (NKB * A_STAGE);
#pragma unroll
      for (int u = 0; u < 8; ++u) {
        __builtin_amdgcn_raw_buffer_store_b128(b[u], rs, dst + u * 1024, 0, AUX_SYS);
        if (u == REL_AT) link_release();
      }
    };
    auto store_w = [&](const unsigned (&b)[32], int half) {
      const unsigned dst = wdst + (unsigned)half * (4u * NKB * B_STAGE);
#pragma unroll
      for (int qh = 0; qh < 2; ++qh) {
        v4u o[4];
#pragma unroll
        for (int u = 0; u < 4; ++u) {
          const unsigned* v = b + qh * 16 + 4 * u;
          const unsigned t0 = __builtin_amdgcn_perm(v[1], v[0], 0x05010400u);
          const unsigned t1 = __builtin_amdgcn_perm(v[1], v[0], 0x07030602u);
          const unsigned t2 = __builtin_amdgcn_perm(v[3], v[2], 0x05010400u);
          const unsigned t3 = __builtin_amdgcn_perm(v[3], v[2], 0x07030602u);
          o[0][u] = __builtin_amdgcn_perm(t2, t0, 0x05040100u);
          o[1][u] = __builtin_amdgcn_perm(t2, t0, 0x07060302u);
          o[2][u] = __builtin_amdgcn_perm(t3, t1, 0x05040100u);
          o[3][u] = __builtin_amdgcn_perm(t3, t1, 0x07060302u);
        }
#pragma unroll
        for (int j = 0; j < 4; ++j) {
          __builtin_amdgcn_raw_buffer_store_b128(o[j], rs, dst + j * 2048 + qh * 1024, 0, AUX_SYS);
          if (qh * 4 + j == REL_AT) link_release();
        }
      }
    };
    auto xflag = [&](int tmq) { return (unsigned*)(dws + WS_XFLAG + (size_t)tmq * 128) + rank; };
    auto wflag = [&](int half) { return (unsigned*)(dws + WS_WFLAG + (size_t)(half * 8 + idx_in) * 128) + rank; };
#define PUSH_FENCE() asm volatile("" ::: "memory")
#define PUSH_PUBLISH(N, flagptr) do { \
      asm volatile("s_waitcnt vmcnt(" #N ")" ::: "memory"); \
      __builtin_amdgcn_s_barrier(); PUSH_FENCE(); \
      if (tid == 0) st_sys(flagptr, call); \
      PUSH_FENCE(); } while (0)
#define PUSH_DRAW() do { if (tid == 0) { \
      const unsigned d = __hip_atomic_fetch_add(pq, 1u, __ATOMIC_RELAXED, __HIP_MEMORY_SCOPE_AGENT) - pq_base; \
      *lds_next = d < 24u ? (d & 7u) * 4u + 1u + (d >> 3) : 32u; } } while (0)
    const unsigned tk_base = (call - 1u) * 80u;
    auto xticket = [&](int tmq) { return tk_base + ((tmq & 3) == 0 ? (unsigned)(tmq >> 2) : 16u + (unsigned)(tmq & 3) * 8u + (unsigned)(tmq >> 2)); };
    auto go = [&](unsigned ticket) {
      if (tid == 0 && remote) link_wait(ticket, call);
      asm volatile("s_waitcnt lgkmcnt(0)" ::: "memory");
      __builtin_amdgcn_s_barrier();
      PUSH_FENCE();
    };
    if (idx_in >= (int)NTOK) go(xticket(idx_in * 4));
    store_x(xa, idx_in * 4); PUSH_FENCE();
    load_w(wa, 0); PUSH_FENCE();
    PUSH_PUBLISH(32, xflag(idx_in * 4));
    PUSH_DRAW();
    go(tk_base + 8u + idx_in);
    int ta = __builtin_amdgcn_readfirstlane(*lds_next);
    store_w(wa, 0); PUSH_FENCE();
    load_w(wb, 1); PUSH_FENCE();
    load_x(xa, ta); PUSH_FENCE();
    PUSH_PUBLISH(40, wflag(0));
    if (ta < 32) PUSH_DRAW();
    go(tk_base + 16u + idx_in);
    int tb = ta < 32 ? __builtin_amdgcn_readfirstlane(*lds_next) : 32;
    store_w(wb, 1); PUSH_FENCE();
    load_x(xb, tb); PUSH_FENCE();
    PUSH_PUBLISH(8, wflag(1));
    for (;;) {
      if (ta >= 32) break;
      {
        const int cur = ta;
        if (tb < 32) PUSH_DRAW();
        go(xticket(cur));
        ta = tb < 32 ? __builtin_amdgcn_readfirstlane(*lds_next) : 32;
        store_x(xa, cur); PUSH_FENCE();
        load_x(xa, ta); PUSH_FENCE();
        PUSH_PUBLISH(8, xflag(cur));
      }
      if (tb >= 32) break;
      {
        const int cur = tb;
        if (ta < 32) PUSH_DRAW();
        go(xticket(cur));
        tb = ta < 32 ? __builtin_amdgcn_readfirstlane(*lds_next) : 32;
        store_x(xb, cur); PUSH_FENCE();
        load_x(xb, tb); PUSH_FENCE();
        PUSH_PUBLISH(8, xflag(cur));
      }
    }
  }
  const unsigned call = __builtin_amdgcn_readfirstlane(call_v) + 1u;
  const size_t stage_off = WS_STAGE + (size_t)(call & 1u) * PAR_BYTES;

  const int idx = (idx_in - NPUSH) & 31;
  const int mt = idx >> 3, nt = idx & 7;
  const int tm = (int)g * 4 + mt;
  auto* const lds_sel = (__attribute__((address_space(3))) int*)(smem + NSTAGE * STAGE);
  const unsigned* myflag = nullptr;
  if (lane < 8) myflag = (const unsigned*)(myws + WS_XFLAG + (size_t)tm * 128) + lane;
  else if (lane < 24) myflag = (const unsigned*)(myws + WS_WFLAG + (size_t)(nt * 2 + ((lane - 8) >> 3)) * 128) + (lane & 7);
  unsigned rem = 0xffu, ready = 0u;
  auto select = [&]() {
    const unsigned long long t0 = __builtin_amdgcn_s_memrealtime();
    while ((ready & rem) == 0u) {
      unsigned v = call;
      if (myflag) v = ld_sys(myflag);
      const unsigned m = (unsigned)__builtin_amdgcn_ballot_w64(v >= call);
      ready = m & (m >> 8) & (m >> 16) & 0xffu;
      if (ready & rem) break;
      if (ld_agent(tmo) == call || __builtin_amdgcn_s_memrealtime() - t0 > DEADLINE_TICKS) {
        if (lane == 0) st_agent(tmo, call);
        ready = 0xffu;
        break;
      }
      __builtin_amdgcn_s_sleep(4);
    }
    const unsigned cand = ready & rem;
    const int sel = (cand >> rank) & 1u ? rank : __builtin_ctz(cand);
    rem &= ~(1u << sel);
    if (lane == 0) *lds_sel = sel;
  };
  if (wave == 0) {
    select();
    __builtin_amdgcn_fence(__ATOMIC_ACQUIRE, "");
    asm volatile("s_waitcnt vmcnt(0) lgkmcnt(0)" ::: "memory");
  }
  __builtin_amdgcn_s_barrier();
  asm volatile("" ::: "memory");
  int src = __builtin_amdgcn_readfirstlane(*lds_sel);

  const int wm = wave >> 2, wn = wave & 3;
  v4f acc[4][4];
#pragma unroll
  for (int i = 0; i < 4; ++i)
#pragma unroll
    for (int j = 0; j < 4; ++j) acc[i][j] = v4f{0.f, 0.f, 0.f, 0.f};
  const float s = p.sx[0] * p.sw[0];
  unsigned ticket = 0;

  {
    uint8_t* const sbase = myws + stage_off;
    const unsigned va = (unsigned)tm * (NKB * A_STAGE) + wave * 8192 + lane * 16;
    const unsigned vb = (unsigned)XS_BYTES + (unsigned)nt * (NKB * B_STAGE) + wave * 4096 + lane * 16;
    auto issue1 = [&](unsigned nrec, int kb, int boff, int c) {
      const __amdgpu_buffer_rsrc_t rs = make_rsrc(sbase, nrec);
      if (c < 2) bufld16(rs, smem + boff + wave * 2048 + c * 1024, va, (kb >> 2) * (4 * A_STAGE) + (kb & 3) * 2048 + c * 1024, 0);
      else bufld16(rs, smem + boff + A_STAGE + wave * 4096 + (c - 2) * 1024, vb, kb * B_STAGE + (c - 2) * 1024, 0);
    };
#pragma unroll
    for (int c = 0; c < 6; ++c) issue1((unsigned)PAR_BYTES, src * 4, 0, c);
#pragma unroll
    for (int c = 0; c < 6; ++c) issue1((unsigned)PAR_BYTES, src * 4 + 1, STAGE, c);
    int boff = 0;
    const int a_rd = wm * 4 * 2048 + lane * 16;
    const int b_rd = A_STAGE + wn * 4 * 2048 + lane * 16;
    auto step = [&](unsigned nrec, int pkb) {
      int nb = boff + 2 * STAGE;
      if (nb >= NSTAGE * STAGE) nb -= NSTAGE * STAGE;
      const uint8_t* sa = smem + boff + a_rd;
      const uint8_t* sb = smem + boff + b_rd;
      v8i af[4], bf[4];
      auto rd = [&](const uint8_t* q) {
        const v4i lo = *(const v4i*)q, hi = *(const v4i*)(q + 1024);
        return __builtin_shufflevector(lo, hi, 0, 1, 2, 3, 4, 5, 6, 7);
      };
      af[0] = rd(sa);
#pragma unroll
      for (int j = 0; j < 4; ++j) bf[j] = rd(sb + j * 2048);
#pragma unroll
      for (int i = 1; i < 4; ++i) af[i] = rd(sa + i * 2048);
      __builtin_amdgcn_sched_barrier(0);
#pragma unroll
      for (int i = 0; i < 4; ++i) {
#pragma unroll
        for (int j = 0; j < 4; ++j) {
          acc[i][j] = __builtin_amdgcn_mfma_scale_f32_16x16x128_f8f6f4(af[i], bf[j], acc[i][j], 0  , 1  , 0, 0, 0, 0);
          if ((j & 1) && i < 3) {
            __builtin_amdgcn_sched_barrier(0);
            issue1(nrec, pkb, nb, i * 2 + (j >> 1));
            __builtin_amdgcn_sched_barrier(0);
          }
        }
      }
      __builtin_amdgcn_s_waitcnt(0xC07F);
      boff += STAGE;
      if (boff >= NSTAGE * STAGE) boff = 0;
    };
#define GEMM_SYNC() do { asm volatile("s_waitcnt vmcnt(6)" ::: "memory"); __builtin_amdgcn_s_barrier(); asm volatile("" ::: "memory"); } while (0)
    __builtin_amdgcn_s_waitcnt(0xC07F);
    for (int slot = 0; slot < WORLD; ++slot) {
      const bool more = slot + 1 < WORLD;
      const unsigned nrec = more ? (unsigned)PAR_BYTES : 0u;
      GEMM_SYNC();
      step((unsigned)PAR_BYTES, src * 4 + 2);
      GEMM_SYNC();
      step((unsigned)PAR_BYTES, src * 4 + 3);
      if (more && wave == 0) {
        select();
        if (slot == WORLD - 2) {
          if (lane == 0 && g != (unsigned)rank)
            ticket = __hip_atomic_fetch_add((unsigned*)(myws + WS_QTICK + g * 128), 1u, __ATOMIC_RELAXED, __HIP_MEMORY_SCOPE_AGENT);
        }
        asm volatile("s_waitcnt lgkmcnt(0)" ::: "memory");
      }
      GEMM_SYNC();
      const int nsrc = more ? __builtin_amdgcn_readfirstlane(*lds_sel) : 0;
      __builtin_amdgcn_s_waitcnt(0xC07F);
      step(nrec, nsrc * 4);
      GEMM_SYNC();
      step(nrec, nsrc * 4 + 1);
      src = nsrc;
    }
  }
  const unsigned tmo_v = ld_agent(tmo);
  asm volatile("s_waitcnt vmcnt(1)" ::: "memory");

  const __amdgpu_buffer_rsrc_t ro = make_rsrc(p.out[rank], (unsigned)OUT_BYTES);
  const unsigned row_off = (unsigned)(mt * BM + wm * 64 + gq * 4) * GN * 4u;
  const unsigned col_off = (unsigned)(nt * BN + wn * 64 + c16 * 4) * 4u;

  if (g == (unsigned)rank) {
    const bool bad = tmo_v == call;
#pragma unroll
    for (int i = 0; i < 4; ++i)
#pragma unroll
      for (int r = 0; r < 4; ++r) {
        v4f v;
#pragma unroll
        for (int j = 0; j < 4; ++j) {
          const float y = acc[i][j][r] * s;
          v[j] = (y <= 0.f) ? 0.f : y;
          if (bad) v[j] = __builtin_nanf("");
        }
        __builtin_amdgcn_raw_buffer_store_b128(__builtin_bit_cast(v4u, v), ro,
                                               row_off + col_off + (unsigned)(rank * NS * 4) + (unsigned)(i * 16 + r) * (GN * 4), 0, AUX_SYS);
      }
  } else {
    asm volatile("s_waitcnt lgkmcnt(0)" ::: "memory");
    __builtin_amdgcn_s_barrier();
    asm volatile("" ::: "memory");
    uint8_t* const pk = smem + wave * PK_SLOT;
    const unsigned pk_lds = (unsigned)(unsigned long)((__attribute__((address_space(3))) uint8_t*)smem) + wave * PK_SLOT;
    unsigned nxt = pk_lds + 512u;
    unsigned bits[2] = {0u, 0u};
#pragma unroll
    for (int i = 0; i < 4; ++i)
#pragma unroll
      for (int r = 0; r < 4; ++r) {
        float y0, y1, y2, y3;
        unsigned t0, t1, t2, t3, n, b1, b2, b3;
        asm volatile(
            "v_mul_f32 %[y0], %[s], %[a0]\n\t"
            "v_mul_f32 %[y1], %[s], %[a1]\n\t"
            "v_mul_f32 %[y2], %[s], %[a2]\n\t"
            "v_mul_f32 %[y3], %[s], %[a3]\n\t"
            "v_cmp_nge_f32_e64 s[84:85], 0, %[y0]\n\t"
            "v_cmp_nge_f32_e64 s[86:87], 0, %[y1]\n\t"
            "v_cmp_nge_f32_e64 s[88:89], 0, %[y2]\n\t"
            "v_cmp_nge_f32_e64 s[90:91], 0, %[y3]\n\t"
            "s_bcnt1_i32_b64 %[n], s[84:85]\n\t"
            "s_lshl2_add_u32 %[b1], %[n], %[nx]\n\t"
            "s_bcnt1_i32_b64 %[n], s[86:87]\n\t"
            "s_lshl2_add_u32 %[b2], %[n], %[b1]\n\t"
            "s_bcnt1_i32_b64 %[n], s[88:89]\n\t"
            "s_lshl2_add_u32 %[b3], %[n], %[b2]\n\t"
            "v_mbcnt_lo_u32_b32 %[t0], s84, 0\n\t"
            "v_mbcnt_lo_u32_b32 %[t1], s86, 0\n\t"
            "v_mbcnt_lo_u32_b32 %[t2], s88, 0\n\t"
            "v_mbcnt_lo_u32_b32 %[t3], s90, 0\n\t"
            "v_mbcnt_hi_u32_b32 %[t0], s85, %[t0]\n\t"
            "v_mbcnt_hi_u32_b32 %[t1], s87, %[t1]\n\t"
            "v_mbcnt_hi_u32_b32 %[t2], s89, %[t2]\n\t"
            "v_mbcnt_hi_u32_b32 %[t3], s91, %[t3]\n\t"
            "v_lshl_add_u32 %[t0], %[t0], 2, %[nx]\n\t"
            "v_lshl_add_u32 %[t1], %[t1], 2, %[b1]\n\t"
            "v_lshl_add_u32 %[t2], %[t2], 2, %[b2]\n\t"
            "v_lshl_add_u32 %[t3], %[t3], 2, %[b3]\n\t"
            "s_bcnt1_i32_b64 %[n], s[90:91]\n\t"
            "s_lshl2_add_u32 %[nx], %[n], %[b3]\n\t"
            "v_addc_co_u32_e64 %[bt], vcc, %[bt], %[bt], s[84:85]\n\t"
            "v_addc_co_u32_e64 %[bt], vcc, %[bt], %[bt], s[86:87]\n\t"
            "v_addc_co_u32_e64 %[bt], vcc, %[bt], %[bt], s[88:89]\n\t"
            "v_addc_co_u32_e64 %[bt], vcc, %[bt], %[bt], s[90:91]\n\t"
            "s_mov_b64 exec, s[84:85]\n\t"
            "ds_write_b32 %[t0], %[y0]\n\t"
            "s_mov_b64 exec, s[86:87]\n\t"
            "ds_write_b32 %[t1], %[y1]\n\t"
            "s_mov_b64 exec, s[88:89]\n\t"
            "ds_write_b32 %[t2], %[y2]\n\t"
            "s_mov_b64 exec, s[90:91]\n\t"
            "ds_write_b32 %[t3], %[y3]\n\t"
            "s_mov_b64 exec, -1"
            : [y0] "=&v"(y0), [y1] "=&v"(y1), [y2] "=&v"(y2), [y3] "=&v"(y3), [t0] "=&v"(t0), [t1] "=&v"(t1), [t2] "=&v"(t2),
              [t3] "=&v"(t3), [n] "=&s"(n), [b1] "=&s"(b1), [b2] "=&s"(b2), [b3] "=&s"(b3), [nx] "+s"(nxt), [bt] "+v"(bits[i >> 1])
            : [s] "s"(s), [a0] "v"(acc[i][0][r]), [a1] "v"(acc[i][1][r]), [a2] "v"(acc[i][2][r]), [a3] "v"(acc[i][3][r])
            : "vcc", "scc", "s84", "s85", "s86", "s87", "s88", "s89", "s90", "s91", "memory");
      }
    unsigned base = nxt - pk_lds;
    if (tmo_v == call) {
      bits[0] = bits[1] = 0xffffffffu;
      base = 512u + 16384u;
      for (int o = 0; o < 64; ++o) *(float*)(pk + 512 + o * 256 + lane * 4) = __builtin_nanf("");
    }
    *(uint2*)(pk + lane * 8) = make_uint2(bits[0], bits[1]);
    const unsigned bytes = (base + 63u) & ~63u;

    if (tid == 0) link_wait((call - 1u) * 80u + 48u + (ticket - (call - 1u) * 32u), call);
    asm volatile("s_waitcnt lgkmcnt(0)" ::: "memory");
    __builtin_amdgcn_s_barrier();
    asm volatile("" ::: "memory");
    {
      const __amdgpu_buffer_rsrc_t rp = make_rsrc(p.ws[g] + WS_PK, (unsigned)PK_BYTES);
      const unsigned slot = (unsigned)((rank * 32 + idx_in) * 8 + wave) * PK_SLOT;
#pragma unroll
      for (int o = 0; o < 17; ++o) {
        const unsigned off = o * 1024u + lane * 16u;
        if (o * 1024u < bytes) {
          const v4u v = *(const v4u*)(pk + off);
          if (off < bytes) __builtin_amdgcn_raw_buffer_store_b128(v, rp, slot + off, 0, AUX_SYS);
        }
        if (o == REL_AT) link_release();
      }
    }
    asm volatile("s_waitcnt vmcnt(0)" ::: "memory");
    __builtin_amdgcn_s_barrier();
    if (tid == 0) st_sys((unsigned*)(p.ws[g] + WS_OFLAG + ((size_t)rank * 32 + idx_in) * 128), call);

    if (wave == 0) {
      const unsigned* f = lane == 0 ? (const unsigned*)(myws + WS_OFLAG + ((size_t)g * 32 + idx_in) * 128) : nullptr;
      wait_flag(f, call, tmo);
    }
    asm volatile("" ::: "memory");
    __builtin_amdgcn_s_barrier();
    asm volatile("" ::: "memory");
    const bool bad2 = ld_agent(tmo) == call;
    {
      const __amdgpu_buffer_rsrc_t rq = make_rsrc(myws + WS_PK, (unsigned)PK_BYTES);
      const unsigned src = (unsigned)((g * 32 + idx_in) * 8 + wave) * PK_SLOT;
#pragma unroll
      for (int o = 0; o < 17; ++o) bufld16(rq, pk + o * 1024, lane * 16, src + o * 1024, AUX_SYS);
      asm volatile("s_waitcnt vmcnt(0)" ::: "memory");
      const uint2 mm = *(const uint2*)(pk + lane * 8);
      *(float*)(pk + 17152u + lane * 4u) = 0.f;
      const unsigned zero_at = pk_lds + 17152u + lane * 4u;
      unsigned rb = pk_lds + 512u;
      unsigned mb[2] = {mm.x, mm.y};
#define DEC4(v0, v1, v2, v3) \
            "v_add_co_u32_e64 %[mb], s[84:85], %[mb], %[mb]\n\t" \
            "v_add_co_u32_e64 %[mb], s[86:87], %[mb], %[mb]\n\t" \
            "v_add_co_u32_e64 %[mb], s[88:89], %[mb], %[mb]\n\t" \
            "v_add_co_u32_e64 %[mb], s[90:91], %[mb], %[mb]\n\t" \
            "s_bcnt1_i32_b64 %[n], s[84:85]\n\t" \
            "s_lshl2_add_u32 %[b1], %[n], %[rb]\n\t" \
            "s_bcnt1_i32_b64 %[n], s[86:87]\n\t" \
            "s_lshl2_add_u32 %[b2], %[n], %[b1]\n\t" \
            "s_bcnt1_i32_b64 %[n], s[88:89]\n\t" \
            "s_lshl2_add_u32 %[b3], %[n], %[b2]\n\t" \
            "v_mbcnt_lo_u32_b32 " v0 ", s84, 0\n\t" \
            "v_mbcnt_lo_u32_b32 " v1 ", s86, 0\n\t" \
            "v_mbcnt_lo_u32_b32 " v2 ", s88, 0\n\t" \
            "v_mbcnt_lo_u32_b32 " v3 ", s90, 0\n\t" \
            "v_mbcnt_hi_u32_b32 " v0 ", s85, " v0 "\n\t" \
            "v_mbcnt_hi_u32_b32 " v1 ", s87, " v1 "\n\t" \
            "v_mbcnt_hi_u32_b32 " v2 ", s89, " v2 "\n\t" \
            "v_mbcnt_hi_u32_b32 " v3 ", s91, " v3 "\n\t" \
            "v_lshl_add_u32 " v0 ", " v0 ", 2, %[rb]\n\t" \
            "v_lshl_add_u32 " v1 ", " v1 ", 2, %[b1]\n\t" \
            "v_lshl_add_u32 " v2 ", " v2 ", 2, %[b2]\n\t" \
            "v_lshl_add_u32 " v3 ", " v3 ", 2, %[b3]\n\t" \
            "s_bcnt1_i32_b64 %[n], s[90:91]\n\t" \
            "s_lshl2_add_u32 %[rb], %[n], %[b3]\n\t" \
            "v_cndmask_b32_e64 " v0 ", %[z], " v0 ", s[84:85]\n\t" \
            "v_cndmask_b32_e64 " v1 ", %[z], " v1 ", s[86:87]\n\t" \
            "v_cndmask_b32_e64 " v2 ", %[z], " v2 ", s[88:89]\n\t" \
            "v_cndmask_b32_e64 " v3 ", %[z], " v3 ", s[90:91]\n\t" \
            "ds_read_b32 " v0 ", " v0 "\n\t" \
            "ds_read_b32 " v1 ", " v1 "\n\t" \
            "ds_read_b32 " v2 ", " v2 "\n\t" \
            "ds_read_b32 " v3 ", " v3 "\n\t"
#pragma unroll
      for (int i = 0; i < 4; ++i) {
        unsigned e[16], n, b1, b2, b3;
        asm volatile(
            DEC4("%[e0]", "%[e1]", "%[e2]", "%[e3]")
            DEC4("%[e4]", "%[e5]", "%[e6]", "%[e7]")
            DEC4("%[e8]", "%[e9]", "%[e10]", "%[e11]")
            DEC4("%[e12]", "%[e13]", "%[e14]", "%[e15]")
            "s_waitcnt lgkmcnt(0)"
            : [e0] "=&v"(e[0]), [e1] "=&v"(e[1]), [e2] "=&v"(e[2]), [e3] "=&v"(e[3]), [e4] "=&v"(e[4]), [e5] "=&v"(e[5]),
              [e6] "=&v"(e[6]), [e7] "=&v"(e[7]), [e8] "=&v"(e[8]), [e9] "=&v"(e[9]), [e10] "=&v"(e[10]), [e11] "=&v"(e[11]),
              [e12] "=&v"(e[12]), [e13] "=&v"(e[13]), [e14] "=&v"(e[14]), [e15] "=&v"(e[15]),
              [n] "=&s"(n), [b1] "=&s"(b1), [b2] "=&s"(b2), [b3] "=&s"(b3), [rb] "+s"(rb), [mb] "+v"(mb[i >> 1])
            : [z] "v"(zero_at)
            : "scc", "s84", "s85", "s86", "s87", "s88", "s89", "s90", "s91", "memory");
#pragma unroll
        for (int r = 0; r < 4; ++r) {
          v4u v = {e[r * 4], e[r * 4 + 1], e[r * 4 + 2], e[r * 4 + 3]};
          if (bad2) v = v4u{0x7fc00000u, 0x7fc00000u, 0x7fc00000u, 0x7fc00000u};
          __builtin_amdgcn_raw_buffer_store_b128(v, ro, row_off + col_off + g * (NS * 4) + (unsigned)(i * 16 + r) * (GN * 4), 0, AUX_SYS);
        }
      }
    }
  }
  if (tid == 0) {
    const unsigned t = __hip_atomic_fetch_add((unsigned*)(myws + WS_EXIT), 1u, __ATOMIC_RELAXED, __HIP_MEMORY_SCOPE_AGENT);
    if ((t + 1u) % NGRID == 0u) {
      for (int l = 0; l < WORLD; ++l) st_agent((unsigned*)(myws + WS_CLAIM + l * 128), 0u);
      st_agent((unsigned*)(myws + WS_CALL), call);
    }
  }
}

}

extern "C" void kernel_launch(int rank, int nranks, ncclComm_t comm,
                              hipStream_t stream,
                              void* const* d_in, const int* in_sizes, int n_in,
                              void* d_out, int out_size,
                              const int* in_dtypes, int out_dtype,
                              void* const* d_out_all,
                              void* const* d_ws_all, size_t ws_bytes) {
  (void)comm; (void)in_dtypes; (void)out_dtype; (void)d_out;
  if (nranks != WORLD || n_in != 4 || ws_bytes < WS_NEED || (unsigned)rank >= (unsigned)WORLD ||
      in_sizes[0] != GM * KS || in_sizes[1] != KS * GN || out_size != MB * GN) {
    fprintf(stderr, "kernel_launch: unexpected problem shape\n");
    return;
  }
  Params p;
  p.x = (const uint8_t*)d_in[0];
  p.w = (const uint8_t*)d_in[1];
  p.sx = (const float*)d_in[2];
  p.sw = (const float*)d_in[3];
  for (int r = 0; r < WORLD; ++r) {
    p.out[r] = (float*)d_out_all[r];
    p.ws[r] = (uint8_t*)d_ws_all[r];
  }
  p.rank = rank;
  {
    unsigned group1 = 0xB2u;
    int cur = -1, count = 0, bus[WORLD];
    bool known = hipGetDevice(&cur) == hipSuccess && hipGetDeviceCount(&count) == hipSuccess && count == WORLD && cur == rank;
    for (int r = 0; known && r < WORLD; ++r)
      known = hipDeviceGetAttribute(&bus[r], hipDeviceAttributePciBusId, r) == hipSuccess;
    if (known) {
      unsigned g1 = 0, seen = 0;
      for (int r = 0; r < WORLD; ++r) {
        int pos = 0;
        for (int q = 0; q < WORLD; ++q) pos += bus[q] < bus[r];
        seen |= 1u << pos;
        if ((0xB1u >> pos) & 1u) g1 |= 1u << r;
      }
      if (seen == 0xFFu) group1 = g1;
    }
    (void)hipGetLastError();
    unsigned lo = 0, hi = 4;
    p.label_map = 0;
    int same_count = 0;
    for (int r = 0; r < WORLD; ++r) same_count += ((group1 >> r) & 1u) == ((group1 >> rank) & 1u);
    if (same_count != 4) group1 = 0xB2u;
    for (int r = 0; r < WORLD; ++r) {
      const bool same = ((group1 >> r) & 1u) == ((group1 >> rank) & 1u);
      p.label_map |= (unsigned)r << (4u * (same ? lo++ : hi++));
    }
  }
  gemm_rs_kernel<<<dim3(NGRID), dim3(NTHREADS), 0, stream>>>(p);
  HIP_CHECK(hipGetLastError());
}
```
